```python
import numpy as np

kernel_name = "gpu_a2a_gemm_m4096_k4096_n16384_bf16_reluqe4m3"


ABSMAX_RTOL = 0.09

COMPUTE_DTYPE = "bfloat16"
EPILOGUE = "relu_quant_e4m3"
IN_QUANT = None
OUT_QUANT = "e4m3"
IN_DTYPES = {"x": "bfloat16", "w_mat": "bfloat16"}
WORLD_SIZE = 8

def _to_bf16(x):
    u = np.ascontiguousarray(x, dtype=np.float32).view(np.uint32)
    return ((u + 0x7FFF + ((u >> 16) & 1)) & 0xFFFF0000).view(np.float32)

def setup_inputs():

    x = _to_bf16(np.random.randn(4096, 4096).astype(np.float32))
    w_mat = _to_bf16((np.random.randn(4096, 16384) / np.sqrt(4096)).astype(np.float32))
    return {"x": x, "w_mat": w_mat}

def _snap_fp8(x, mant_bits, max_abs, min_normal):
    a = np.abs(x.astype(np.float64))
    _m, _e = np.frexp(a)
    q = np.where(a >= min_normal, np.ldexp(1.0, _e - 1 - mant_bits),
                 min_normal * 2.0 ** -mant_bits)
    snapped = np.minimum(np.round(a / q) * q, max_abs)
    return (np.sign(x) * snapped).astype(np.float32)
def _quant_dequant(y, amax):
    scale = amax / 448.0
    q = _snap_fp8(y / scale, 3, 448.0, 2.0 ** -6)
    return (q * scale).astype(np.float32)
def _epilogue(y):
    y = np.maximum(y, 0.0)
    return _quant_dequant(y, np.abs(y).max())

def reference(x, w_mat):
    y = (x @ w_mat).astype(np.float32)
    return _epilogue(y)
```

```cpp
#include <hip/hip_runtime.h>
#include <rccl/rccl.h>
#include <cstdio>
#include <cstdint>

#define HIP_CHECK(expr) do { hipError_t _e = (expr); if (_e != hipSuccess) { \
    fprintf(stderr, "HIP error %s at %s:%d\n", hipGetErrorString(_e), __FILE__, __LINE__); \
    return; } } while (0)

namespace a2ag {

typedef __bf16 bf16x4 __attribute__((ext_vector_type(4)));
typedef __bf16 bf16x8 __attribute__((ext_vector_type(8)));
typedef float f32x4 __attribute__((ext_vector_type(4)));
typedef unsigned u32x4 __attribute__((ext_vector_type(4)));
typedef unsigned long long u64;
#define LAS __attribute__((address_space(3)))
#define DI __device__ __forceinline__

constexpr int NR = 8;
constexpr int MLOC = 512, KDIM = 4096, NDIM = 16384, KSL = 512;
constexpr int BM = 256, BN = 128, BK = 64;
constexpr int NT = KDIM / BK;
constexpr int NWG = (MLOC / BM) * (NDIM / BN);
constexpr int NTHREADS = 576;
constexpr unsigned A_STAGE = BM * BK * 2, B_STAGE = BK * BN * 2;
constexpr unsigned A_RING = 3, B_RING = 4;
constexpr unsigned LDS_B0 = A_RING * A_STAGE;
constexpr unsigned LDS_BYTES = LDS_B0 + B_RING * B_STAGE;
constexpr unsigned SLOT_BYTES = MLOC * KSL * 2;
constexpr unsigned X_BYTES = NR * SLOT_BYTES;
constexpr unsigned W_BYTES = (unsigned)KDIM * NDIM * 2;
constexpr int NCHUNK = 4, NPART = 32;
constexpr unsigned WS_XFLAG = X_BYTES;
constexpr unsigned WS_GRAN = WS_XFLAG + 8192;
constexpr unsigned WS_SEQ = WS_GRAN + NR * NWG * 8;
constexpr unsigned WS_TMO = WS_SEQ + NWG * 64;
constexpr unsigned WS_DBG = WS_TMO + 4096;
constexpr unsigned WS_NEED = WS_DBG + NWG * 64 * 8;
constexpr u64 DEADLINE_TICKS = 40000000ull;
constexpr unsigned OOB = 0x40000000u;

struct Args {
    const void* x; const void* w; void* out;
    char* ws[NR];
    int rank;
};

DI void dma16(__amdgpu_buffer_rsrc_t srd, unsigned voff, unsigned soff, unsigned lds_addr) {
    asm volatile("s_mov_b32 m0, %3\n\ts_nop 0\n\tbuffer_load_dwordx4 %0, %1, %2 offen lds"
                 :: "v"(voff), "s"(srd), "s"(soff), "s"(lds_addr) : "memory");
}
#define WAIT_V(n) asm volatile("s_waitcnt vmcnt(" #n ")" ::: "memory")
#define WAIT_L(n) asm volatile("s_waitcnt lgkmcnt(" #n ")" ::: "memory")
#define BAR() __builtin_amdgcn_s_barrier()
#define SCHED() __builtin_amdgcn_sched_barrier(0)

#define STAMP(i) do {} while (0)
#define CSTAMP(i) do {} while (0)
#define STAMP_DECL
#define STAMP_FLUSH(lo, hi) do {} while (0)

DI unsigned ktile_of(int t, int rank) {
    if (t < 8) return (unsigned)(rank * 8 + t);
    const int i = t - 8, c = i / 14, r = i - c * 14;
    return (unsigned)((((rank + 1 + (r >> 1)) & 7) << 3) + 2 * c + (r & 1));
}

DI bool timed_out(unsigned* tmo, unsigned call, u64 t0) {
    if (__hip_atomic_load(tmo, __ATOMIC_RELAXED, __HIP_MEMORY_SCOPE_AGENT) == call
        || __builtin_amdgcn_s_memrealtime() - t0 > DEADLINE_TICKS) {
        __hip_atomic_store(tmo, call, __ATOMIC_RELAXED, __HIP_MEMORY_SCOPE_AGENT);
        return true;
    }
    return false;
}

DI bool wait_xflags(__amdgpu_buffer_rsrc_t wsr, int c, unsigned call, int lane, int rank, unsigned* tmo) {
    const u64 t0 = __builtin_amdgcn_s_memrealtime();
    for (;;) {
        asm volatile("" ::: "memory");
        const u32x4 v = __builtin_amdgcn_raw_buffer_load_b128(wsr, WS_XFLAG + c * 1024 + lane * 16, 0, 17);
        const bool ok = (lane >> 3) == rank || ((int)(v.x - call) >= 0 && (int)(v.y - call) >= 0 && (int)(v.z - call) >= 0 && (int)(v.w - call) >= 0);
        if (__all(ok)) break;
        if (__builtin_amdgcn_readfirstlane(timed_out(tmo, call, t0))) return false;
        __builtin_amdgcn_s_sleep(1);
    }
    __builtin_amdgcn_fence(__ATOMIC_ACQUIRE, "");
    return true;
}

DI float wave_max(float v) {
#pragma unroll
    for (int o = 32; o >= 1; o >>= 1) v = fmaxf(v, __shfl_xor(v, o));
    return v;
}

__global__ __launch_bounds__(NTHREADS) void a2a_gemm_kernel(Args a) {
    extern __shared__ __attribute__((aligned(16))) char smem[];
    LAS char* lds = (LAS char*)smem;
    const int tid = threadIdx.x, lane = tid & 63;
    const int wid = __builtin_amdgcn_readfirstlane(tid >> 6);
    const int wg = blockIdx.x, rank = a.rank;
    char* wsl = a.ws[0];
#pragma unroll
    for (int r = 1; r < NR; ++r) wsl = (rank == r) ? a.ws[r] : wsl;
    unsigned* seqp = (unsigned*)(wsl + WS_SEQ + wg * 64);
    unsigned* tmo = (unsigned*)(wsl + WS_TMO);
    STAMP_DECL;
    if (wid == 0) STAMP(0);
    if (wid == 8) STAMP(32);

    if (wid == 8) {
        if (wg < 7 * NPART) {
            const int j = wg % 7 + 1, part = wg / 7, dst = (rank + j) & 7;
            char* wsd = a.ws[0];
#pragma unroll
            for (int r = 1; r < NR; ++r) wsd = (dst == r) ? a.ws[r] : wsd;
            const __amdgpu_buffer_rsrc_t xs = __builtin_amdgcn_make_buffer_rsrc((void*)a.x, 0, X_BYTES, 0x00020000);
            const __amdgpu_buffer_rsrc_t ds = __builtin_amdgcn_make_buffer_rsrc((void*)wsd, 0, WS_GRAN, 0x00020000);
            const unsigned voff = (unsigned)(part * 16 + (lane >> 4)) * 1024u + (unsigned)(lane & 15) * 16u;
            u32x4 v[NCHUNK][4];
#pragma unroll
            for (int c = 0; c < NCHUNK; ++c)
#pragma unroll
                for (int i = 0; i < 4; ++i)
                    v[c][i] = __builtin_amdgcn_raw_buffer_load_b128(xs, voff + c * 256 + i * 4096, dst * SLOT_BYTES, 0);
            const unsigned call = __hip_atomic_load(seqp, __ATOMIC_RELAXED, __HIP_MEMORY_SCOPE_AGENT) + 1u;
#pragma unroll
            for (int c = 0; c < NCHUNK; ++c)
#pragma unroll
                for (int i = 0; i < 4; ++i)
                    __builtin_amdgcn_raw_buffer_store_b128(v[c][i], ds, voff + c * 256 + i * 4096, rank * SLOT_BYTES, 17);
            STAMP(33);
            WAIT_V(0);
            STAMP(34);
            if (lane < NCHUNK)
                __hip_atomic_store((unsigned*)(wsd + WS_XFLAG + lane * 1024 + rank * 128 + part * 4), call, __ATOMIC_RELAXED, __HIP_MEMORY_SCOPE_SYSTEM);
            STAMP_FLUSH(32, 64);
        }
        return;
    }

    const int xcd = wg & 7, idx = wg >> 3;
    const int ntile = xcd * 16 + (idx >> 1), mtile = idx & 1;
    const int wm = wid & 3, wn = wid >> 2;
    const __amdgpu_buffer_rsrc_t xs = __builtin_amdgcn_make_buffer_rsrc((void*)a.x, 0, X_BYTES, 0x00020000);
    const __amdgpu_buffer_rsrc_t ss = __builtin_amdgcn_make_buffer_rsrc((void*)wsl, 0, X_BYTES, 0x00020000);
    const __amdgpu_buffer_rsrc_t wsr = __builtin_amdgcn_make_buffer_rsrc((void*)wsl, 0, WS_SEQ, 0x00020000);
    const __amdgpu_buffer_rsrc_t wr = __builtin_amdgcn_make_buffer_rsrc((void*)a.w, 0, W_BYTES, 0x00020000);
    const unsigned voffA = (unsigned)(8 * wid + (lane >> 3)) * 1024u + (unsigned)(((lane & 7) ^ (4 * (wid & 1) + (lane >> 4))) << 4);
    const unsigned voffB = (unsigned)(8 * wid + (lane >> 4)) * 32768u + (unsigned)(((lane & 15) ^ (((wid & 1) << 3) | (lane >> 4))) << 4);
    const unsigned ldsbase = (unsigned)(size_t)lds;
    const unsigned ldsA_w = ldsbase + (unsigned)wid * 1024u, ldsB_w = ldsbase + LDS_B0 + (unsigned)wid * 2048u;
    const unsigned soA0 = (unsigned)mtile * (BM * 1024u), soB0 = (unsigned)ntile * (BN * 2u);
    const int fr = lane & 15, g = lane >> 4, q = fr >> 2, p = lane & 3;
    unsigned aoff[2], boff[4];
#pragma unroll
    for (int kk = 0; kk < 2; ++kk) aoff[kk] = (unsigned)((wm * 64 + fr) * 128 + (((kk * 4 + g) ^ (fr >> 1)) << 4));
#pragma unroll
    for (int f = 0; f < 4; ++f)
        boff[f] = LDS_B0 + (unsigned)((8 * g + q) * 256 + (((wn * 8 + 2 * p + (f >> 1)) ^ (q | ((g & 1) << 3))) << 4) + ((((f & 1) ^ (p & 1))) << 3));

    auto soffA = [&](int t) -> unsigned { if (t >= NT) return OOB; const unsigned kt = ktile_of(t, rank); return (kt >> 3) * SLOT_BYTES + (kt & 7) * 128u + soA0; };
    auto soffB = [&](int t) -> unsigned { if (t >= NT) return OOB; return ktile_of(t, rank) * (BK * 32768u) + soB0; };
#define DMA_A(t, st, so, j) dma16((t) < 8 ? xs : ss, voffA, (so) + (j) * 65536u, ldsA_w + (st) * A_STAGE + (j) * 8192u)
#define DMA_B(st, so, j) dma16(wr, voffB, (so) + (j) * 131072u, ldsB_w + (st) * B_STAGE + (j) * 1024u)

    {
        const unsigned a0 = soffA(0), a1 = soffA(1), b0 = soffB(0), b1 = soffB(1), b2 = soffB(2);
        DMA_A(0, 0, a0, 0); DMA_A(0, 0, a0, 1); DMA_A(0, 0, a0, 2); DMA_A(0, 0, a0, 3);
        DMA_B(0, b0, 0); DMA_B(0, b0, 1);
        DMA_B(1, b1, 0); DMA_B(1, b1, 1);
        DMA_A(1, 1, a1, 0); DMA_A(1, 1, a1, 1); DMA_A(1, 1, a1, 2); DMA_A(1, 1, a1, 3);
        DMA_B(2, b2, 0); DMA_B(2, b2, 1);
    }
    const unsigned call = __builtin_amdgcn_readfirstlane(__hip_atomic_load(seqp, __ATOMIC_RELAXED, __HIP_MEMORY_SCOPE_AGENT)) + 1u;
    bool good = true;

    f32x4 acc[4][4];
#pragma unroll
    for (int m = 0; m < 4; ++m)
#pragma unroll
        for (int f = 0; f < 4; ++f) acc[m][f] = (f32x4){0.f, 0.f, 0.f, 0.f};
    bf16x8 xf[4][2], wf[2][2];

    WAIT_V(8);
    if (wid == 0) STAMP(1);
    if (wn == 1) BAR();
    BAR();
    if (wid == 0) { STAMP(2); CSTAMP(30); }

    unsigned sa = 0, sb = 0;
    for (int t = 0; t < NT; ++t) {
        const unsigned sa2 = (sa == 0) ? 2u : sa - 1u;
        const unsigned sb3 = (sb + 3u) & 3u;
        const LAS char* la = lds + sa * A_STAGE;
        const LAS char* lb = lds + sb * B_STAGE;
        if (t >= 6 && (t - 6) % 14 == 0 && t < 62) {
            if (wid == 0) STAMP(3 + 2 * ((t - 6) / 14));
            good = wait_xflags(wsr, (t - 6) / 14, call, lane, rank, tmo) && good;
            if (wid == 0) STAMP(4 + 2 * ((t - 6) / 14));
        }
        const unsigned an = soffA(t + 2), bn = soffB(t + 3);
#pragma unroll
        for (int f = 0; f < 2; ++f)
#pragma unroll
            for (int kk = 0; kk < 2; ++kk) {
                const bf16x4 lo = __builtin_amdgcn_ds_read_tr16_b64_v4bf16((LAS bf16x4*)(lb + boff[f] + kk * 8192));
                const bf16x4 hi = __builtin_amdgcn_ds_read_tr16_b64_v4bf16((LAS bf16x4*)(lb + boff[f] + kk * 8192 + 1024));
                wf[f][kk] = __builtin_shufflevector(lo, hi, 0, 1, 2, 3, 4, 5, 6, 7);
            }
#pragma unroll
        for (int m = 0; m < 4; ++m)
#pragma unroll
            for (int kk = 0; kk < 2; ++kk) xf[m][kk] = *(const LAS bf16x8*)(la + aoff[kk] + m * 2048);
        DMA_A(t + 2, sa2, an, 0); DMA_A(t + 2, sa2, an, 1);
        BAR(); WAIT_L(0);
        __builtin_amdgcn_s_setprio(1);
#pragma unroll
        for (int kk = 0; kk < 2; ++kk)
#pragma unroll
            for (int m = 0; m < 4; ++m)
#pragma unroll
                for (int f = 0; f < 2; ++f) acc[m][f] = __builtin_amdgcn_mfma_f32_16x16x32_bf16(wf[f][kk], xf[m][kk], acc[m][f], 0, 0, 0);
        __builtin_amdgcn_s_setprio(0);
        BAR(); SCHED();
#pragma unroll
        for (int f = 0; f < 2; ++f)
#pragma unroll
            for (int kk = 0; kk < 2; ++kk) {
                const bf16x4 lo = __builtin_amdgcn_ds_read_tr16_b64_v4bf16((LAS bf16x4*)(lb + boff[2 + f] + kk * 8192));
                const bf16x4 hi = __builtin_amdgcn_ds_read_tr16_b64_v4bf16((LAS bf16x4*)(lb + boff[2 + f] + kk * 8192 + 1024));
                wf[f][kk] = __builtin_shufflevector(lo, hi, 0, 1, 2, 3, 4, 5, 6, 7);
            }
        DMA_A(t + 2, sa2, an, 2); DMA_A(t + 2, sa2, an, 3);
        DMA_B(sb3, bn, 0); DMA_B(sb3, bn, 1);
        WAIT_V(8);
        BAR(); WAIT_L(0);
        __builtin_amdgcn_s_setprio(1);
#pragma unroll
        for (int kk = 0; kk < 2; ++kk)
#pragma unroll
            for (int m = 0; m < 4; ++m)
#pragma unroll
                for (int f = 0; f < 2; ++f) acc[m][2 + f] = __builtin_amdgcn_mfma_f32_16x16x32_bf16(wf[f][kk], xf[m][kk], acc[m][2 + f], 0, 0, 0);
        __builtin_amdgcn_s_setprio(0);
        BAR(); SCHED();
        sa = (sa == 2) ? 0u : sa + 1u;
        sb = (sb + 1u) & 3u;
    }
    WAIT_V(0);
    if (wn == 0) BAR();
    BAR();
    if (wid == 0) { STAMP(11); CSTAMP(31); }

    float mx = 0.f;
#pragma unroll
    for (int m = 0; m < 4; ++m)
#pragma unroll
        for (int f = 0; f < 4; ++f)
#pragma unroll
            for (int r = 0; r < 4; ++r) mx = fmaxf(mx, acc[m][f][r]);
    mx = wave_max(mx);
    LAS unsigned* red = (LAS unsigned*)lds;
    if (lane == 0) red[wid] = good ? __float_as_uint(mx) : 0xffffffffu;
    __syncthreads();
    if (wid == 0) {
        unsigned wmx = 0;
#pragma unroll
        for (int i = 0; i < 8; ++i) wmx = max(wmx, red[i]);
        char* pw = a.ws[0];
#pragma unroll
        for (int r = 1; r < NR; ++r) pw = (lane == r) ? a.ws[r] : pw;
        if (lane < NR)
            __hip_atomic_store((u64*)(pw + WS_GRAN) + rank * NWG + wg, ((u64)call << 32) | wmx, __ATOMIC_RELAXED, __HIP_MEMORY_SCOPE_SYSTEM);
        if (lane == 0) __hip_atomic_store(seqp, call, __ATOMIC_RELAXED, __HIP_MEMORY_SCOPE_AGENT);
    }
    unsigned gm = 0;
    if (wid == 0) STAMP(12);
    {
        const u64 t0 = __builtin_amdgcn_s_memrealtime();
        for (;;) {
            asm volatile("" ::: "memory");
            const u32x4 v0 = __builtin_amdgcn_raw_buffer_load_b128(wsr, WS_GRAN + tid * 16, 0, 17);
            const u32x4 v1 = __builtin_amdgcn_raw_buffer_load_b128(wsr, WS_GRAN + 8192 + tid * 16, 0, 17);
            const bool ok = v0.y == call && v0.w == call && v1.y == call && v1.w == call;
            gm = max(max(v0.x, v0.z), max(v1.x, v1.z));
            if (__all(ok)) break;
            if (__builtin_amdgcn_readfirstlane(timed_out(tmo, call, t0))) { gm = 0xffffffffu; break; }
            __builtin_amdgcn_s_sleep(1);
        }
    }
#pragma unroll
    for (int o = 32; o >= 1; o >>= 1) gm = max(gm, (unsigned)__shfl_xor((int)gm, o));
    if (lane == 0) red[16 + wid] = gm;
    __syncthreads();
#pragma unroll
    for (int i = 0; i < 8; ++i) gm = max(gm, red[16 + i]);
    const bool bad = gm == 0xffffffffu;
    if (wid == 0) STAMP(13);
    const float amax = __uint_as_float(gm);
    const float inv = amax > 0.f ? 448.0f / amax : 0.f;
    char* outp = (char*)a.out + (size_t)(mtile * BM + wm * 64 + fr) * NDIM + ntile * BN + wn * 64 + g * 16;
    const bool odd = g & 1;
#pragma unroll
    for (int m = 0; m < 4; ++m) {
        unsigned d[4];
#pragma unroll
        for (int f = 0; f < 4; ++f) {
            const f32x4 y = acc[m][f];
            const float y0 = __builtin_amdgcn_fmed3f(y[0] * inv, 0.f, 448.f), y1 = __builtin_amdgcn_fmed3f(y[1] * inv, 0.f, 448.f);
            const float y2 = __builtin_amdgcn_fmed3f(y[2] * inv, 0.f, 448.f), y3 = __builtin_amdgcn_fmed3f(y[3] * inv, 0.f, 448.f);
            int w = __builtin_amdgcn_cvt_pk_fp8_f32(y0, y1, 0, false);
            w = __builtin_amdgcn_cvt_pk_fp8_f32(y2, y3, w, true);
            d[f] = bad ? 0x7f7f7f7fu : (unsigned)w;
        }
        u32x4 o;
        o.x = odd ? d[1] : d[0]; o.y = odd ? d[0] : d[1]; o.z = odd ? d[3] : d[2]; o.w = odd ? d[2] : d[3];
        *(u32x4*)(outp + (size_t)m * 16 * NDIM) = o;
    }
}

}

extern "C" void kernel_launch(int rank, int nranks, ncclComm_t comm,
                              hipStream_t stream,
                              void* const* d_in, const int* in_sizes, int n_in,
                              void* d_out, int out_size,
                              const int* in_dtypes, int out_dtype,
                              void* const* d_out_all,
                              void* const* d_ws_all, size_t ws_bytes) {
    using namespace a2ag;
    if (nranks != NR || (unsigned)rank >= (unsigned)NR || n_in < 2 || ws_bytes < WS_NEED
        || in_sizes[0] != KDIM * KSL || in_sizes[1] != KDIM * NDIM || out_size != MLOC * NDIM) {
        fprintf(stderr, "kernel_launch: unexpected problem geometry\n");
        return;
    }
    HIP_CHECK(hipFuncSetAttribute((const void*)a2a_gemm_kernel, hipFuncAttributeMaxDynamicSharedMemorySize, LDS_BYTES));
    Args a;
    a.x = d_in[0]; a.w = d_in[1]; a.out = d_out; a.rank = rank;
    for (int r = 0; r < NR; ++r) a.ws[r] = (char*)d_ws_all[r];
    hipLaunchKernelGGL(a2a_gemm_kernel, dim3(NWG), dim3(NTHREADS), LDS_BYTES, stream, a);
    HIP_CHECK(hipGetLastError());
}
```

```cpp
#include <hip/hip_runtime.h>
#include <rccl/rccl.h>
#include <cstdio>
#include <cstdint>

#define HIP_CHECK(expr) do { hipError_t _e = (expr); if (_e != hipSuccess) { \
    fprintf(stderr, "HIP error %s at %s:%d\n", hipGetErrorString(_e), __FILE__, __LINE__); \
    return; } } while (0)

#define PUSH_PACE 3
#define PUSH_SPIN 5
#define POL_W 0

#pragma clang diagnostic ignored "-Wint-to-pointer-cast"

namespace a2ag {

typedef __bf16 bf16x4 __attribute__((ext_vector_type(4)));
typedef __bf16 bf16x8 __attribute__((ext_vector_type(8)));
typedef float f32x4 __attribute__((ext_vector_type(4)));
typedef unsigned u32x4 __attribute__((ext_vector_type(4)));
typedef unsigned long long u64;
#define LAS __attribute__((address_space(3)))
#define DI __device__ __forceinline__

constexpr int NR = 8;
constexpr int MLOC = 512, KDIM = 4096, NDIM = 16384, KSL = 512;
constexpr int BM = 256, BN = 128, BK = 64;
constexpr int NT = KDIM / BK;
constexpr int NWG = (MLOC / BM) * (NDIM / BN);
constexpr int NTHREADS = 768;
constexpr unsigned A_STAGE = BM * BK * 2, B_STAGE = BK * BN * 2;
constexpr int RA = 3, RB = 4;
constexpr unsigned LDS_B0 = RA * A_STAGE;
constexpr unsigned LDS_BYTES = LDS_B0 + RB * B_STAGE;
constexpr unsigned SLOT_BYTES = MLOC * KSL * 2;
constexpr unsigned X_BYTES = NR * SLOT_BYTES;
constexpr unsigned W_BYTES = (unsigned)KDIM * NDIM * 2;
constexpr int NCHUNK = 8, NPART = 32;
constexpr unsigned WS_XFLAG = X_BYTES;
constexpr unsigned WS_GRAN = WS_XFLAG + NCHUNK * 1024;
constexpr unsigned WS_SEQ = WS_GRAN + NR * NWG * 8;
constexpr unsigned WS_TMO = WS_SEQ + NWG * 64;
constexpr unsigned WS_DBG = WS_TMO + 4096;
constexpr unsigned WS_NEED = WS_DBG + NWG * 64 * 8;
constexpr u64 DEADLINE_TICKS = 40000000ull;
constexpr unsigned OOB = 0x40000000u;
constexpr unsigned SRD3 = 0x00020000u;

struct Args {
    const void* x; const void* w; void* out;
    char* ws[NR];
    int rank;
};

template <int POL> DI void dma16(__amdgpu_buffer_rsrc_t srd, unsigned voff, unsigned soff, unsigned lds_addr) {
    if (POL == 0) asm volatile("s_mov_b32 m0, %3\n\ts_nop 0\n\tbuffer_load_dwordx4 %0, %1, %2 offen lds" :: "v"(voff), "s"(srd), "s"(soff), "s"(lds_addr) : "memory");
    if (POL == 1) asm volatile("s_mov_b32 m0, %3\n\ts_nop 0\n\tbuffer_load_dwordx4 %0, %1, %2 offen nt lds" :: "v"(voff), "s"(srd), "s"(soff), "s"(lds_addr) : "memory");
    if (POL == 2) asm volatile("s_mov_b32 m0, %3\n\ts_nop 0\n\tbuffer_load_dwordx4 %0, %1, %2 offen sc1 lds" :: "v"(voff), "s"(srd), "s"(soff), "s"(lds_addr) : "memory");
}
template <int N> DI void wait_v() { asm volatile("s_waitcnt vmcnt(%0)" :: "n"(N) : "memory"); }
#define WAIT_L(n) asm volatile("s_waitcnt lgkmcnt(" #n ")" ::: "memory")
#define BAR() __builtin_amdgcn_s_barrier()
#define SCHED() __builtin_amdgcn_sched_barrier(0)
#define PRIO(n) __builtin_amdgcn_s_setprio(n)
#define MFMA(a, b, c) __builtin_amdgcn_mfma_f32_16x16x32_bf16(a, b, c, 0, 0, 0)

#define STAMP(i) do {} while (0)
#define CSTAMP(i) do {} while (0)
#define STAMP_DECL
#define STAMP_FLUSH(lo, hi) do {} while (0)

DI unsigned ktile_of(int t, int rank) {
    const int i = t - 8, c = (i * 37) >> 8, j = i - c * 7;
    return t < 8 ? (unsigned)(rank * 8 + t) : (unsigned)((((rank + 1 + j) & 7) << 3) + c);
}

DI bool timed_out(unsigned* tmo, unsigned call, u64 t0, unsigned& polls) {
    if ((++polls & 31u) != 0u) return false;
    if (__builtin_amdgcn_readfirstlane(__hip_atomic_load(tmo, __ATOMIC_RELAXED, __HIP_MEMORY_SCOPE_AGENT)) == call
        || __builtin_amdgcn_s_memrealtime() - t0 > DEADLINE_TICKS) {
        __hip_atomic_store(tmo, call, __ATOMIC_RELAXED, __HIP_MEMORY_SCOPE_AGENT);
        return true;
    }
    return false;
}
DI bool flags_ok(const u32x4 v, unsigned call, bool own) {
    return __all(own || ((int)(v.x - call) >= 0 && (int)(v.y - call) >= 0 && (int)(v.z - call) >= 0 && (int)(v.w - call) >= 0));
}
DI unsigned vm_outstanding() {
    const unsigned s = __builtin_amdgcn_s_getreg(7 | (0 << 6) | (31 << 11));
    return (s & 15u) | ((s >> 18) & 48u);
}
DI unsigned wave_umax(unsigned v) {
    v = max(v, (unsigned)__builtin_amdgcn_update_dpp(0, (int)v, 0x111, 0xf, 0xf, false));
    v = max(v, (unsigned)__builtin_amdgcn_update_dpp(0, (int)v, 0x112, 0xf, 0xf, false));
    v = max(v, (unsigned)__builtin_amdgcn_update_dpp(0, (int)v, 0x114, 0xf, 0xf, false));
    v = max(v, (unsigned)__builtin_amdgcn_update_dpp(0, (int)v, 0x118, 0xf, 0xf, false));
    const unsigned a = __builtin_amdgcn_readlane(v, 15), b = __builtin_amdgcn_readlane(v, 31);
    const unsigned c = __builtin_amdgcn_readlane(v, 47), d = __builtin_amdgcn_readlane(v, 63);
    return max(max(a, b), max(c, d));
}

__global__ __launch_bounds__(NTHREADS) void a2a_gemm_kernel(Args a) {
    extern __shared__ __attribute__((aligned(16))) char smem[];
    LAS char* lds = (LAS char*)smem;
    const int tid = threadIdx.x, lane = tid & 63;
    const int wid = __builtin_amdgcn_readfirstlane(tid >> 6);
    const int wg = blockIdx.x, rank = a.rank;
    const int xcd = wg & 7, idx = wg >> 3;
    const int ntile = xcd * 16 + (idx >> 1), mtile = idx & 1;
    const unsigned ldsbase = (unsigned)(size_t)lds;
    LAS unsigned* red = (LAS unsigned*)lds;
    char* wsl = a.ws[0];
#pragma unroll
    for (int r = 1; r < NR; ++r) wsl = (rank == r) ? a.ws[r] : wsl;
    unsigned* seqp = (unsigned*)(wsl + WS_SEQ + wg * 64);
    unsigned* tmo = (unsigned*)(wsl + WS_TMO);
    STAMP_DECL;

    if (wid == 10) {
        PRIO(3);
        const __amdgpu_buffer_rsrc_t wr = __builtin_amdgcn_make_buffer_rsrc((void*)a.w, 0, W_BYTES, SRD3);
        unsigned vo[2];
#pragma unroll
        for (int e = 0; e < 2; ++e) vo[e] = (unsigned)(lane >> 4) * 32768u + (unsigned)(((lane & 15) ^ ((e << 3) | (lane >> 4))) << 4);
        const unsigned so0 = (unsigned)ntile * (BN * 2u), l0 = ldsbase + LDS_B0;
        auto issue = [&](int t, unsigned st) {
            const unsigned so = t < NT ? ktile_of(t, rank) * (BK * 32768u) + so0 : OOB;
            const unsigned la = l0 + st * B_STAGE;
#pragma unroll
            for (int p = 0; p < 16; ++p) dma16<POL_W>(wr, vo[(p >> 1) & 1], so + p * 131072u, la + p * 1024u);
        };
#pragma unroll 1
        for (int t = 0; t < RB - 1; ++t) issue(t, t);
        wait_v<(RB - 2) * 16>(); issue(RB - 1, RB - 1); BAR();
        unsigned st = 0;
#pragma unroll 1
        for (int t = 0; t < NT; ++t) {
            wait_v<(RB - 2) * 16>(); BAR();
            issue(t + RB, st); st = (st + 1u) & (RB - 1);
        }
        wait_v<0>(); BAR();
        return;
    }
    if (wid == 8 || wid == 9) {
        PRIO(3);
        const int half = wid - 8;
        const __amdgpu_buffer_rsrc_t xs = __builtin_amdgcn_make_buffer_rsrc((void*)a.x, 0, X_BYTES, SRD3);
        const __amdgpu_buffer_rsrc_t ss = __builtin_amdgcn_make_buffer_rsrc((void*)wsl, 0, WS_GRAN, SRD3);
        unsigned vo[2], vs[2];
#pragma unroll
        for (int e = 0; e < 2; ++e) {
            const unsigned ch = (unsigned)(((lane & 7) ^ (4 * e + (lane >> 4))) << 4);
            vo[e] = (unsigned)(lane >> 3) * 1024u + ch;
            vs[e] = (unsigned)(lane >> 3) * 128u + ch;
        }
        const unsigned row0 = (unsigned)(mtile * BM + half * 128);
        const unsigned l0 = ldsbase + (unsigned)half * 16384u;
        unsigned st = 0;
        auto issue = [&](int t) {
            const bool mine = t < 8;
            const unsigned so = mine ? (unsigned)rank * SLOT_BYTES + row0 * 1024u + (unsigned)t * 128u
                                     : (t < NT ? ktile_of(t, rank) * 65536u + row0 * 128u : OOB);
            const unsigned ps = mine ? 8192u : 1024u, la = l0 + st * A_STAGE;
            const unsigned v0 = mine ? vo[0] : vs[0], v1 = mine ? vo[1] : vs[1];
            const __amdgpu_buffer_rsrc_t srd = mine ? xs : ss;
#pragma unroll
            for (int p = 0; p < 16; ++p) dma16<2>(srd, (p & 1) ? v1 : v0, so + p * ps, la + p * 1024u);
            st = (st == RA - 1) ? 0u : st + 1u;
        };
        unsigned seq;
        asm volatile("global_load_dword %0, %1, off sc1\n\ts_nop 0" : "=&v"(seq) : "v"(seqp) : "memory");
        issue(0); issue(1); issue(2);
        asm volatile("s_waitcnt vmcnt(%1)" : "+v"(seq) : "n"((RA - 1) * 16) : "memory");
        BAR();
        const unsigned call = __builtin_amdgcn_readfirstlane(seq) + 1u;
        const bool own = (lane >> 3) == rank;
        const unsigned fvo = WS_XFLAG + (unsigned)lane * 16u;
        bool good = true;
        int tn = RA;
#define XSTEP() do { wait_v<(RA - 2) * 16>(); BAR(); issue(tn); ++tn; } while (0)
#pragma unroll 1
        while (tn < 7) XSTEP();
#pragma unroll 1
        for (int chunk = 0; chunk < NCHUNK; ++chunk) {
            u32x4 fl;
            wait_v<(RA - 2) * 16>(); BAR();
            asm volatile("buffer_load_dwordx4 %0, %1, %2, %3 offen sc0 sc1" : "=&v"(fl) : "v"(fvo), "s"(ss), "s"((unsigned)chunk * 1024u) : "memory");
            issue(tn); ++tn;
            asm volatile("s_waitcnt vmcnt(%1)" : "+v"(fl) : "n"((RA - 2) * 16) : "memory");
            BAR();
            if (!flags_ok(fl, call, own)) {
                if (half == 0) STAMP(40 + chunk);
                const u64 t0 = __builtin_amdgcn_s_memrealtime();
                unsigned polls = 0;
                for (;;) {
                    u32x4 f2;
                    asm volatile("buffer_load_dwordx4 %0, %1, %2, %3 offen sc0 sc1\n\ts_waitcnt vmcnt(0)" : "=&v"(f2) : "v"(fvo), "s"(ss), "s"((unsigned)chunk * 1024u) : "memory");
                    if (flags_ok(f2, call, own)) break;
                    if (timed_out(tmo, call, t0, polls)) { good = false; break; }
                }
                if (half == 0) STAMP(48 + chunk);
            }
            issue(tn); ++tn;
#pragma unroll 1
            for (int i = 0; i < 5; ++i) XSTEP();
        }
#pragma unroll 1
        while (tn < NT + RA) XSTEP();
        wait_v<0>(); BAR();
        if (lane == 0) red[8 + half] = good ? 0u : 0xffffffffu;
        WAIT_L(0); BAR();
        if (half == 0) STAMP_FLUSH(40, 56);
        return;
    }
    if (wid == 11) {
        if (wg >= 7 * NPART) return;
        STAMP(32);
        const int j = wg % 7 + 1, part = wg / 7, dst = (rank + j) & 7;
        char* wsd = a.ws[0];
#pragma unroll
        for (int r = 1; r < NR; ++r) wsd = (dst == r) ? a.ws[r] : wsd;
        const __amdgpu_buffer_rsrc_t xs = __builtin_amdgcn_make_buffer_rsrc((void*)a.x, 0, X_BYTES, SRD3);
        const __amdgpu_buffer_rsrc_t ds = __builtin_amdgcn_make_buffer_rsrc((void*)wsd, 0, WS_GRAN, SRD3);
        const unsigned vsrc = (unsigned)(part * 16 + (lane >> 3)) * 1024u + (unsigned)(lane & 7) * 16u;
        const unsigned vdst = (unsigned)part * 2048u + (unsigned)lane * 16u;
        u32x4 v[NCHUNK][2];
#pragma unroll
        for (int c = 0; c < NCHUNK; ++c)
#pragma unroll
            for (int i = 0; i < 2; ++i)
                v[c][i] = __builtin_amdgcn_raw_buffer_load_b128(xs, vsrc + c * 128 + i * 8192, dst * SLOT_BYTES, 0);
        const unsigned call = __hip_atomic_load(seqp, __ATOMIC_RELAXED, __HIP_MEMORY_SCOPE_AGENT) + 1u;
        unsigned cv = call;
        asm volatile("s_waitcnt vmcnt(0)" : "+v"(cv) :: "memory");
        STAMP(33);
        const unsigned fdst = WS_XFLAG + (unsigned)rank * 128u + (unsigned)part * 4u;
        int nb = 0;
#pragma unroll
        for (int c = 0; c < NCHUNK; ++c) {
#pragma unroll 1
            while (nb < PUSH_PACE * c) { BAR(); ++nb; }
            asm volatile("" ::: "memory");
#pragma unroll
            for (int i = 0; i < 2; ++i)
                __builtin_amdgcn_raw_buffer_store_b128(v[c][i], ds, vdst + i * 1024, rank * SLOT_BYTES + c * 65536, 17);
            asm volatile("" ::: "memory");
            int spin = 0;
#pragma unroll 1
            while (vm_outstanding() != 0u) {
                if (++spin > PUSH_SPIN && nb < 7 + 7 * c) { BAR(); ++nb; spin = 0; }
                else __builtin_amdgcn_s_sleep(1);
            }
            asm volatile("" ::: "memory");
            if (lane == 0) __builtin_amdgcn_raw_buffer_store_b32(cv, ds, fdst, c * 1024, 17);
            asm volatile("" ::: "memory");
            if (c == 0) STAMP(34);
        }
        STAMP(35);
        STAMP_FLUSH(32, 40);
        return;
    }

    if (wid == 0) STAMP(0);
    const int wm = wid & 3, wn = wid >> 2;
    const int fr = lane & 15, g = lane >> 4, q = fr >> 2, p = lane & 3;
    unsigned aoff[2], boff[4];
#pragma unroll
    for (int kk = 0; kk < 2; ++kk) aoff[kk] = ldsbase + (unsigned)((wm * 64 + fr) * 128 + (((kk * 4 + g) ^ (fr >> 1)) << 4));
#pragma unroll
    for (int f = 0; f < 4; ++f)
        boff[f] = ldsbase + LDS_B0 + (unsigned)((8 * g + q) * 256 + (((wn * 8 + 2 * p + (f >> 1)) ^ (q | ((g & 1) << 3))) << 4) + ((((f & 1) ^ (p & 1))) << 3));
    asm volatile("" : "+v"(boff[0]), "+v"(boff[1]), "+v"(boff[2]), "+v"(boff[3]), "+v"(aoff[0]), "+v"(aoff[1]));
    const unsigned call = __builtin_amdgcn_readfirstlane(__hip_atomic_load(seqp, __ATOMIC_RELAXED, __HIP_MEMORY_SCOPE_AGENT)) + 1u;

    f32x4 acc[4][4];
#pragma unroll
    for (int m = 0; m < 4; ++m)
#pragma unroll
        for (int f = 0; f < 4; ++f) acc[m][f] = (f32x4){0.f, 0.f, 0.f, 0.f};
    bf16x8 x0[4], w0[4], x1[4], w1[4];
    bf16x4 tl[4];
    unsigned bcur[4], acur = aoff[0];
#pragma unroll
    for (int f = 0; f < 4; ++f) bcur[f] = boff[f];
#define RD_WL(f, kk) do { tl[f] = __builtin_amdgcn_ds_read_tr16_b64_v4bf16((LAS bf16x4*)(bcur[f] + (kk) * 8192)); } while (0)
#define RD_WH(dst, f, kk) do { const bf16x4 hi_ = __builtin_amdgcn_ds_read_tr16_b64_v4bf16((LAS bf16x4*)(bcur[f] + (kk) * 8192 + 1024)); \
        dst = __builtin_shufflevector(tl[f], hi_, 0, 1, 2, 3, 4, 5, 6, 7); } while (0)
#define RD_XA(dst, m) do { dst = *(const LAS bf16x8*)(acur + (m) * 2048); } while (0)
    asm volatile("" ::: "memory");
    BAR();
    asm volatile("" ::: "memory");
    if (wid == 0) { STAMP(2); CSTAMP(30); }
#pragma unroll
    for (int i = 0; i < 4; ++i) { RD_WL(i, 0); RD_WH(w0[i], i, 0); RD_XA(x0[i], i); }
    acur = aoff[1];
#define STEP(P0, P1, cw, cx, nw, nx, kk, X0, X1) do { \
        PRIO(P0); \
        acc[0][0] = MFMA(cw[0], cx[0], acc[0][0]); RD_WL(0, kk); SCHED(); \
        acc[0][1] = MFMA(cw[1], cx[0], acc[0][1]); RD_WH(nw[0], 0, kk); SCHED(); \
        acc[0][2] = MFMA(cw[2], cx[0], acc[0][2]); RD_XA(nx[0], 0); SCHED(); \
        acc[0][3] = MFMA(cw[3], cx[0], acc[0][3]); RD_WL(1, kk); SCHED(); \
        acc[1][3] = MFMA(cw[3], cx[1], acc[1][3]); RD_WH(nw[1], 1, kk); SCHED(); \
        acc[1][2] = MFMA(cw[2], cx[1], acc[1][2]); RD_WL(2, kk); SCHED(); \
        acc[1][1] = MFMA(cw[1], cx[1], acc[1][1]); RD_WH(nw[2], 2, kk); SCHED(); \
        acc[1][0] = MFMA(cw[0], cx[1], acc[1][0]); RD_WL(3, kk); SCHED(); \
        PRIO(P1); \
        acc[2][0] = MFMA(cw[0], cx[2], acc[2][0]); RD_WH(nw[3], 3, kk); SCHED(); \
        acc[2][1] = MFMA(cw[1], cx[2], acc[2][1]); RD_XA(nx[1], 1); SCHED(); \
        acc[2][2] = MFMA(cw[2], cx[2], acc[2][2]); RD_XA(nx[2], 2); SCHED(); \
        acc[2][3] = MFMA(cw[3], cx[2], acc[2][3]); RD_XA(nx[3], 3); SCHED(); \
        acc[3][3] = MFMA(cw[3], cx[3], acc[3][3]); X0; SCHED(); \
        acc[3][2] = MFMA(cw[2], cx[3], acc[3][2]); X1; SCHED(); \
        acc[3][1] = MFMA(cw[1], cx[3], acc[3][1]); SCHED(); \
        acc[3][0] = MFMA(cw[0], cx[3], acc[3][0]); SCHED(); \
    } while (0)
    {
        unsigned sa = 0, sb = 0;
#pragma unroll 1
        for (int t = 0; t < NT; ++t) {
            const unsigned san = (sa == RA - 1) ? 0u : sa + 1u, sbn = (sb + 1u) & (RB - 1);
            SCHED();
            STEP(1, 0, w0, x0, w1, x1, 1,
                 { bcur[0] = boff[0] + sbn * B_STAGE; bcur[1] = boff[1] + sbn * B_STAGE; acur = aoff[0] + san * A_STAGE; },
                 { bcur[2] = boff[2] + sbn * B_STAGE; bcur[3] = boff[3] + sbn * B_STAGE; });
            WAIT_L(0); BAR(); SCHED();
            STEP(3, 2, w1, x1, w0, x0, 0, { acur = aoff[1] + san * A_STAGE; }, {});
            sa = san; sb = sbn;
        }
    }
    float mx = 0.f;
#pragma unroll
    for (int m = 0; m < 4; ++m)
#pragma unroll
        for (int f = 0; f < 4; ++f) mx = fmaxf(fmaxf(mx, fmaxf(acc[m][f][0], acc[m][f][1])), fmaxf(acc[m][f][2], acc[m][f][3]));
    const unsigned wmax = wave_umax(__float_as_uint(mx));
    WAIT_L(0); BAR();
    if (wid == 0) { STAMP(11); CSTAMP(31); }
    if (lane == 0) red[wid] = wmax;
    WAIT_L(0); BAR();
    asm volatile("" ::: "memory");
    const __amdgpu_buffer_rsrc_t wsr = __builtin_amdgcn_make_buffer_rsrc((void*)wsl, 0, WS_SEQ, SRD3);
    if (wid == 0) {
        const unsigned wmx = wave_umax(lane < 10 ? red[lane] : 0u);
        char* pw = a.ws[0];
#pragma unroll
        for (int r = 1; r < NR; ++r) pw = (lane == r) ? a.ws[r] : pw;
        if (lane < NR)
            __hip_atomic_store((u64*)(pw + WS_GRAN) + rank * NWG + wg, ((u64)call << 32) | wmx, __ATOMIC_RELAXED, __HIP_MEMORY_SCOPE_SYSTEM);
        if (lane == 0) __hip_atomic_store(seqp, call, __ATOMIC_RELAXED, __HIP_MEMORY_SCOPE_AGENT);
        STAMP(12);
    }
    unsigned gm = 0;
    {
        const u64 t0 = __builtin_amdgcn_s_memrealtime();
        unsigned polls = 0;
        for (;;) {
            asm volatile("" ::: "memory");
            const u32x4 v0 = __builtin_amdgcn_raw_buffer_load_b128(wsr, WS_GRAN + tid * 16, 0, 17);
            const u32x4 v1 = __builtin_amdgcn_raw_buffer_load_b128(wsr, WS_GRAN + 8192 + tid * 16, 0, 17);
            const bool ok = v0.y == call && v0.w == call && v1.y == call && v1.w == call;
            gm = max(max(v0.x, v0.z), max(v1.x, v1.z));
            if (__all(ok)) break;
            if (timed_out(tmo, call, t0, polls)) { gm = 0xffffffffu; break; }
        }
    }
    gm = wave_umax(gm);
    if (lane == 0) red[16 + wid] = gm;
    WAIT_L(0); BAR();
    asm volatile("" ::: "memory");
    gm = wave_umax(lane < 8 ? red[16 + lane] : 0u);
    if (wid == 0) STAMP(13);
    const bool bad = gm == 0xffffffffu;
    const float amax = __uint_as_float(gm);
    const float inv = amax > 0.f ? 448.0f / amax : 0.f;
    char* outp = (char*)a.out + (size_t)(mtile * BM + wm * 64 + fr) * NDIM + ntile * BN + wn * 64 + g * 16;
    const bool odd = g & 1;
#pragma unroll
    for (int m = 0; m < 4; ++m) {
        unsigned d[4];
#pragma unroll
        for (int f = 0; f < 4; ++f) {
            const f32x4 y = acc[m][f];
            const float y0 = __builtin_amdgcn_fmed3f(y[0] * inv, 0.f, 448.f), y1 = __builtin_amdgcn_fmed3f(y[1] * inv, 0.f, 448.f);
            const float y2 = __builtin_amdgcn_fmed3f(y[2] * inv, 0.f, 448.f), y3 = __builtin_amdgcn_fmed3f(y[3] * inv, 0.f, 448.f);
            int w = __builtin_amdgcn_cvt_pk_fp8_f32(y0, y1, 0, false);
            w = __builtin_amdgcn_cvt_pk_fp8_f32(y2, y3, w, true);
            d[f] = bad ? 0x7f7f7f7fu : (unsigned)w;
        }
        u32x4 o;
        o.x = odd ? d[1] : d[0]; o.y = odd ? d[0] : d[1]; o.z = odd ? d[3] : d[2]; o.w = odd ? d[2] : d[3];
        *(u32x4*)(outp + (size_t)m * 16 * NDIM) = o;
    }
}

}

extern "C" void kernel_launch(int rank, int nranks, ncclComm_t comm,
                              hipStream_t stream,
                              void* const* d_in, const int* in_sizes, int n_in,
                              void* d_out, int out_size,
                              const int* in_dtypes, int out_dtype,
                              void* const* d_out_all,
                              void* const* d_ws_all, size_t ws_bytes) {
    using namespace a2ag;
    if (nranks != NR || (unsigned)rank >= (unsigned)NR || n_in < 2 || ws_bytes < WS_NEED
        || in_sizes[0] != KDIM * KSL || in_sizes[1] != KDIM * NDIM || out_size != MLOC * NDIM) {
        fprintf(stderr, "kernel_launch: unexpected problem geometry\n");
        return;
    }
    HIP_CHECK(hipFuncSetAttribute((const void*)a2a_gemm_kernel, hipFuncAttributeMaxDynamicSharedMemorySize, LDS_BYTES));
    Args a;
    a.x = d_in[0]; a.w = d_in[1]; a.out = d_out; a.rank = rank;
    for (int r = 0; r < NR; ++r) a.ws[r] = (char*)d_ws_all[r];
    hipLaunchKernelGGL(a2a_gemm_kernel, dim3(NWG), dim3(NTHREADS), LDS_BYTES, stream, a);
    HIP_CHECK(hipGetLastError());
}
```

```cpp
#include <hip/hip_runtime.h>
#include <rccl/rccl.h>
#include <cstdio>
#include <cstdint>
#include <type_traits>

#define HIP_CHECK(expr) do { hipError_t _e = (expr); if (_e != hipSuccess) { \
    fprintf(stderr, "HIP error %s at %s:%d\n", hipGetErrorString(_e), __FILE__, __LINE__); \
    return; } } while (0)

#define PUSH_PACE 4
#define XPOLL_SLEEP 4
#define POLL_SLEEP 4
#define PUSH0 1
#define PUSH0_SPREAD 2
#define PUSH_DELAY 8
#define PUSH_SPIN 3

#pragma clang diagnostic ignored "-Wint-to-pointer-cast"

namespace a2ag {

typedef __bf16 bf16x4 __attribute__((ext_vector_type(4)));
typedef __bf16 bf16x8 __attribute__((ext_vector_type(8)));
typedef float f32x4 __attribute__((ext_vector_type(4)));
typedef unsigned u32x4 __attribute__((ext_vector_type(4)));
typedef unsigned long long u64;
#define LAS __attribute__((address_space(3)))
#define DI __device__ __forceinline__

constexpr int NR = 8;
constexpr int MLOC = 512, KDIM = 4096, NDIM = 16384, KSL = 512;
constexpr int BM = 256, BN = 128, BK = 64;
constexpr int NT = KDIM / BK;
constexpr int NWG = (MLOC / BM) * (NDIM / BN);
constexpr int NTHREADS = 704;
constexpr unsigned A_STAGE = BM * BK * 2, B_STAGE = BK * BN * 2;
constexpr int RA = 3, RB = 4;
constexpr unsigned LDS_B0 = RA * A_STAGE;
constexpr unsigned LDS_BYTES = LDS_B0 + RB * B_STAGE;
constexpr unsigned SLOT_BYTES = MLOC * KSL * 2;
constexpr unsigned X_BYTES = NR * SLOT_BYTES;
constexpr unsigned W_BYTES = (unsigned)KDIM * NDIM * 2;
constexpr int NCHUNK = 8, NPART = 32;
constexpr unsigned WS_XFLAG = X_BYTES;
constexpr unsigned WS_GRAN = WS_XFLAG + NCHUNK * 1024;
constexpr unsigned WS_SEQ = WS_GRAN + NR * NWG * 8;
constexpr unsigned WS_TMO = WS_SEQ + NWG * 64;
constexpr unsigned WS_DBG = WS_TMO + 4096;
constexpr unsigned WS_NEED = WS_DBG + NWG * 64 * 8;
constexpr u64 DEADLINE_TICKS = 40000000ull;
constexpr unsigned OOB = 0x40000000u;
constexpr unsigned SRD3 = 0x00020000u;

struct Args {
    const void* x; const void* w; void* out;
    char* ws[NR];
    int rank;
};

DI void set_m0(unsigned v) { asm volatile("s_mov_b32 m0, %0\n\ts_nop 0" :: "s"(v) : "memory"); }
template <int OFF> DI void dma_x(__amdgpu_buffer_rsrc_t srd, unsigned voff, unsigned soff) {
    asm volatile("buffer_load_dwordx4 %0, %1, %2 offen offset:%3 sc1 lds" :: "v"(voff), "s"(srd), "s"(soff), "n"(OFF) : "memory");
}
template <int OFF> DI void dma_w(__amdgpu_buffer_rsrc_t srd, unsigned voff, unsigned soff) {
    asm volatile("buffer_load_dwordx4 %0, %1, %2 offen offset:%3 lds" :: "v"(voff), "s"(srd), "s"(soff), "n"(OFF) : "memory");
}
template <int N> DI void wait_v() { asm volatile("s_waitcnt vmcnt(%0)" :: "n"(N) : "memory"); }
#define WAIT_L(n) asm volatile("s_waitcnt lgkmcnt(" #n ")" ::: "memory")
#define BAR() __builtin_amdgcn_s_barrier()
#define SCHED() __builtin_amdgcn_sched_barrier(0)
#define PRIO(n) __builtin_amdgcn_s_setprio(n)
#define MFMA(a, b, c) __builtin_amdgcn_mfma_f32_16x16x32_bf16(a, b, c, 0, 0, 0)

#define STAMP(i) do {} while (0)
#define CSTAMP(i) do {} while (0)
#define STAMP_DECL
#define STAMP_FLUSH(lo, hi) do {} while (0)

DI unsigned ktile_of(int t, int rank) {
    const int i = t - 8, c = (i * 37) >> 8, j = i - c * 7;
    return t < 8 ? (unsigned)(rank * 8 + t) : (unsigned)((((rank + 1 + j) & 7) << 3) + c);
}

DI bool timed_out(unsigned* tmo, unsigned call, u64 t0, unsigned& polls) {
    if ((++polls & 31u) != 0u) return false;
    if (__builtin_amdgcn_readfirstlane(__hip_atomic_load(tmo, __ATOMIC_RELAXED, __HIP_MEMORY_SCOPE_AGENT)) == call
        || __builtin_amdgcn_s_memrealtime() - t0 > DEADLINE_TICKS) {
        __hip_atomic_store(tmo, call, __ATOMIC_RELAXED, __HIP_MEMORY_SCOPE_AGENT);
        return true;
    }
    return false;
}
DI unsigned vm_outstanding() {
    const unsigned s = __builtin_amdgcn_s_getreg(7 | (0 << 6) | (31 << 11));
    return (s & 15u) | ((s >> 18) & 48u);
}
DI unsigned wave_umax(unsigned v) {
    v = max(v, (unsigned)__builtin_amdgcn_update_dpp(0, (int)v, 0x111, 0xf, 0xf, false));
    v = max(v, (unsigned)__builtin_amdgcn_update_dpp(0, (int)v, 0x112, 0xf, 0xf, false));
    v = max(v, (unsigned)__builtin_amdgcn_update_dpp(0, (int)v, 0x114, 0xf, 0xf, false));
    v = max(v, (unsigned)__builtin_amdgcn_update_dpp(0, (int)v, 0x118, 0xf, 0xf, false));
    const unsigned a = __builtin_amdgcn_readlane(v, 15), b = __builtin_amdgcn_readlane(v, 31);
    const unsigned c = __builtin_amdgcn_readlane(v, 47), d = __builtin_amdgcn_readlane(v, 63);
    return max(max(a, b), max(c, d));
}

__global__ __launch_bounds__(NTHREADS) void a2a_gemm_kernel(Args a) {
    extern __shared__ __attribute__((aligned(16))) char smem[];
    LAS char* lds = (LAS char*)smem;
    const int tid = threadIdx.x, lane = tid & 63;
    const int wid = __builtin_amdgcn_readfirstlane(tid >> 6);
    const int wg = blockIdx.x, rank = a.rank;
    const int xcd = wg & 7, idx = wg >> 3;
    const int ntile = xcd * 16 + (idx >> 1), mtile = idx & 1;
    const unsigned ldsbase = (unsigned)(size_t)lds;
    LAS unsigned* red = (LAS unsigned*)(lds + LDS_B0 + 3 * B_STAGE);
    char* wsl = a.ws[0];
#pragma unroll
    for (int r = 1; r < NR; ++r) wsl = (rank == r) ? a.ws[r] : wsl;
    unsigned* seqp = (unsigned*)(wsl + WS_SEQ + wg * 64);
    unsigned* tmo = (unsigned*)(wsl + WS_TMO);
    STAMP_DECL;

    if (wid == 8 || wid == 9) {
        PRIO(3);
        const int half = wid - 8;
        const __amdgpu_buffer_rsrc_t ss = __builtin_amdgcn_make_buffer_rsrc((void*)wsl, 0, WS_TMO, SRD3);
        const __amdgpu_buffer_rsrc_t wr = __builtin_amdgcn_make_buffer_rsrc((void*)a.w, 0, W_BYTES, SRD3);
        unsigned vs[2], vw[2];
#pragma unroll
        for (int e = 0; e < 2; ++e) {
            vs[e] = (unsigned)(lane >> 3) * 128u + (unsigned)(((lane & 7) ^ (4 * e + (lane >> 4))) << 4);
            vw[e] = (unsigned)(lane >> 4) * 32768u + (unsigned)(((lane & 15) ^ ((e << 3) | (lane >> 4))) << 4);
        }
        const unsigned row0 = (unsigned)(mtile * BM + half * 128);
        const unsigned lA = ldsbase + (unsigned)half * 16384u, lB = ldsbase + LDS_B0 + (unsigned)half * 8192u;
        const unsigned soB = (unsigned)ntile * (BN * 2u) + (unsigned)half * (32u * 32768u);
        const unsigned soOwn = (unsigned)rank * SLOT_BYTES + row0 * 1024u;
        const char* xb = (const char*)a.x;
        unsigned dx = 7168u, v0 = vs[0] + (unsigned)(lane >> 3) * 896u, v1 = vs[1] + (unsigned)(lane >> 3) * 896u;
        unsigned la = lA;
        auto issueA = [&](unsigned so) {
            const __amdgpu_buffer_rsrc_t srd = __builtin_amdgcn_make_buffer_rsrc((void*)xb, 0, X_BYTES, SRD3);
#pragma unroll
            for (int p = 0; p < 16; p += 4) {
                const unsigned s4 = so + p * (dx + 1024u);
                set_m0(la + p * 1024u);
                dma_x<0>(srd, v0, s4); dma_x<1024>(srd, v1, s4 + dx); dma_x<2048>(srd, v0, s4 + 2u * dx); dma_x<3072>(srd, v1, s4 + 3u * dx);
            }
            la = (la == lA + (RA - 1) * A_STAGE) ? lA : la + A_STAGE;
        };
        auto issueAs = [&](unsigned so, unsigned l) {
#pragma unroll
            for (int p = 0; p < 16; p += 4) {
                set_m0(l + p * 1024u);
                dma_x<0>(ss, vs[0], so + p * 1024u); dma_x<1024>(ss, vs[1], so + p * 1024u); dma_x<2048>(ss, vs[0], so + p * 1024u); dma_x<3072>(ss, vs[1], so + p * 1024u);
            }
        };
        auto wstage = [&](int t) -> unsigned { return lB + ((unsigned)t & (RB - 1)) * B_STAGE; };
        auto issueW = [&](unsigned so, unsigned lb, auto lo, auto hi) {
            constexpr int LO = decltype(lo)::value, HI = decltype(hi)::value;
            if constexpr (LO == 0) {
                set_m0(lb);
                dma_w<0>(wr, vw[0], so); dma_w<1024>(wr, vw[0], so + 130048u); dma_w<2048>(wr, vw[1], so + 260096u); dma_w<3072>(wr, vw[1], so + 390144u);
            }
            set_m0(lb + 4096u);
            if constexpr (LO <= 4) { dma_w<0>(wr, vw[0], so + 524288u); dma_w<1024>(wr, vw[0], so + 654336u); }
            if constexpr (HI == 8) { dma_w<2048>(wr, vw[1], so + 784384u); dma_w<3072>(wr, vw[1], so + 914432u); }
        };
        using I0 = std::integral_constant<int, 0>; using I6 = std::integral_constant<int, 6>; using I8 = std::integral_constant<int, 8>;
        const unsigned fvo = WS_XFLAG - 1024u + (unsigned)lane * 16u;
#define POLL_ISSUE(n) asm volatile("buffer_load_dwordx4 v[160:163], %0, %1, %2 offen sc0 sc1" :: "v"(fvo), "s"(ss), "s"((n) * 1024u) : "v160", "v161", "v162", "v163", "memory")
        const unsigned soW0 = (unsigned)rank * (8u * BK * 32768u) + soB;
        issueA(soOwn); issueW(soW0, wstage(0), I0{}, I8{});
        asm volatile("buffer_load_dword v164, off, %0, %1 sc1" :: "s"(ss), "s"(WS_SEQ + (unsigned)wg * 64u) : "v164", "memory");
        issueW(soW0 + BK * 32768u, wstage(1), I0{}, I8{}); issueA(soOwn + 128u);
        unsigned tabA, tabW;
        {
            const int t = lane + 3;
            const unsigned kt = ktile_of(t, rank);
            const unsigned first = (int)(kt >> 3) == ((rank + 1) & 7) ? 16u : 0u;
            tabA = t < 8 ? soOwn + (unsigned)t * 128u : (t < NT ? kt * 65536u + row0 * 128u + first + (kt & 7u) + 1u : OOB);
            const unsigned n0 = t < 8 ? 0u : (kt & 7u) + 1u, n2 = t + 2 < 8 ? 0u : (ktile_of(t + 2, rank) & 7u) + 1u;
            tabW = t < NT ? kt * (BK * 32768u) + soB + ((t + 2 < NT && n2 > n0) ? n2 : 0u) : OOB;
        }
        wait_v<24>(); BAR();
        issueW(soW0 + 2u * BK * 32768u, wstage(2), I0{}, I8{}); issueA(soOwn + 256u);
        if (mtile == 0) issueW(soW0 + 3u * BK * 32768u, wstage(3), I0{}, I6{});
        unsigned call;
        asm volatile("v_readfirstlane_b32 %0, v164" : "=s"(call) :: "memory");
        call += 1u;
        const bool own = (lane >> 3) == rank;
        bool good = true;
        unsigned prog = 0;
        auto flags_ok = [&]() -> bool {
            u32x4 f;
            asm volatile("v_mov_b32 %0, v160\n\tv_mov_b32 %1, v161\n\tv_mov_b32 %2, v162\n\tv_mov_b32 %3, v163" : "=v"(f.x), "=v"(f.y), "=v"(f.z), "=v"(f.w) :: "memory");
            return __all(own || (int)((f.x - call) | (f.y - call) | (f.z - call) | (f.w - call)) >= 0);
        };
        auto await = [&](unsigned need) {
            if (__builtin_expect(prog < need, 0)) {
                if (half == 0) STAMP(40 + need - 1);
                const u64 t0 = __builtin_amdgcn_s_memrealtime();
                unsigned polls = 0;
                for (;;) {
                    POLL_ISSUE(need); wait_v<0>();
                    if (flags_ok()) break;
                    if (timed_out(tmo, call, t0, polls)) { good = false; break; }
                    __builtin_amdgcn_s_sleep(XPOLL_SLEEP);
                }
                prog = need;
                if (half == 0) STAMP(48 + need - 1);
            }
        };
        auto pick_x = [&](int i) {
            const bool mine = i < 5;
            xb = mine ? (const char*)a.x : (const char*)wsl;
            dx = mine ? 7168u : 0u;
            const unsigned dd = mine ? 896u : 0u;
            v0 = vs[0] + (unsigned)(lane >> 3) * dd; v1 = vs[1] + (unsigned)(lane >> 3) * dd;
        };
        unsigned pc = 0;
        int i = 0;
#pragma unroll 1
        for (; i < 5 + mtile; ++i) {
            pick_x(i);
            const unsigned ta = __builtin_amdgcn_readlane(tabA, i), w3 = __builtin_amdgcn_readlane(tabW, i), w4 = __builtin_amdgcn_readlane(tabW, i + 1);
            if ((i ^ mtile) & 1) {
                wait_v<24>(); BAR();
                pc = w3 & 15u;
                if (pc != 0u) POLL_ISSUE(pc);
                issueA(ta & ~127u);
                issueW(w3 & ~255u, wstage(i + 3), I0{}, I8{});
                issueW(w4 & ~255u, wstage(i + 4), I0{}, I6{});
            } else {
                wait_v<30>(); BAR();
                issueW(w3 & ~255u, wstage(i + 3), I6{}, I8{});
                if (pc != 0u) { if (flags_ok()) prog = pc; }
                await(ta & 15u);
                issueA(ta & ~127u);
            }
        }
        unsigned aH, aL, wH, wE, laH, laL, lbH, lbE, nH, nL, rareH, rareL;
        auto plan = [&](int h) {
            const unsigned t0 = __builtin_amdgcn_readlane(tabA, h & 63), t1 = __builtin_amdgcn_readlane(tabA, (h + 1) & 63);
            const unsigned u0 = __builtin_amdgcn_readlane(tabW, h & 63), u1 = __builtin_amdgcn_readlane(tabW, (h + 1) & 63);
            aH = t0 & ~127u; nH = t0 & 15u; aL = t1 & ~127u; nL = t1 & 15u; wH = u0 & ~255u; pc = u0 & 15u; wE = u1 & ~255u;
            laH = la; laL = (la == lA + (RA - 1) * A_STAGE) ? lA : la + A_STAGE; la = (laL == lA + (RA - 1) * A_STAGE) ? lA : laL + A_STAGE;
            lbH = wstage(h + 3); lbE = wstage(h + 4);
            rareH = pc | (t0 & 16u); rareL = pc | (t1 & 16u);
            asm volatile("" : "+s"(aH), "+s"(aL), "+s"(wH), "+s"(wE), "+s"(laH), "+s"(laL), "+s"(lbH), "+s"(lbE), "+s"(rareH), "+s"(rareL));
        };
        plan(i);
#pragma unroll 1
        for (int n = 0; n < 29; ++n) {
            wait_v<24>(); BAR();
            if (__builtin_expect(rareH != 0u, 0)) { await(nH); if (pc != 0u) POLL_ISSUE(pc); }
            issueAs(aH, laH);
            issueW(wH, lbH, I0{}, I8{});
            issueW(wE, lbE, I0{}, I6{});
            wait_v<30>(); BAR();
            issueW(wE, lbE, I6{}, I8{});
            if (__builtin_expect(rareL != 0u, 0)) { if (pc != 0u && flags_ok()) prog = pc; await(nL); }
            issueAs(aL, laL);
            i += 2;
            plan(i);
        }
        if (mtile == 0) BAR();
        if (lane == 0) red[8 + half] = good ? 0u : 0xffffffffu;
        WAIT_L(0); BAR();
        if (half == 0) { STAMP_FLUSH(16, 24); STAMP_FLUSH(40, 56); }
        return;
    }
    if (wid == 10) {
        if (wg >= 7 * NPART) return;
        STAMP(32);
        PRIO(3);
        const int j = wg % 7 + 1, part = wg / 7, dst = (rank + j) & 7;
        char* wsd = a.ws[0];
#pragma unroll
        for (int r = 1; r < NR; ++r) wsd = (dst == r) ? a.ws[r] : wsd;
        const __amdgpu_buffer_rsrc_t xs = __builtin_amdgcn_make_buffer_rsrc((void*)a.x, 0, X_BYTES, SRD3);
        const __amdgpu_buffer_rsrc_t ds = __builtin_amdgcn_make_buffer_rsrc((void*)wsd, 0, WS_GRAN, SRD3);
        const unsigned vsrc = (unsigned)(part * 16 + (lane >> 3)) * 1024u + (unsigned)(lane & 7) * 16u;
        const unsigned vdst = (unsigned)part * 2048u + (unsigned)lane * 16u;
        __builtin_amdgcn_s_sleep(PUSH_DELAY);
        u32x4 v[NCHUNK][2];
        const unsigned call = __hip_atomic_load(seqp, __ATOMIC_RELAXED, __HIP_MEMORY_SCOPE_AGENT) + 1u;
#pragma unroll
        for (int c = 0; c < NCHUNK; ++c)
#pragma unroll
            for (int i = 0; i < 2; ++i)
                v[c][i] = __builtin_amdgcn_raw_buffer_load_b128(xs, vsrc + c * 128 + i * 8192, dst * SLOT_BYTES, 0);
        STAMP(33);
        const unsigned fdst = WS_XFLAG + (unsigned)rank * 128u + (unsigned)part * 4u;
        int nb = 0;
#pragma unroll
        for (int c = 0; c < NCHUNK; ++c) {
            const int slot = c == 0 ? PUSH0 + part % PUSH0_SPREAD : PUSH_PACE * c + (part & (PUSH_PACE - 1));
#pragma unroll 1
            while (nb < slot) { BAR(); ++nb; }
            asm volatile("" ::: "memory");
#pragma unroll
            for (int i = 0; i < 2; ++i)
                __builtin_amdgcn_raw_buffer_store_b128(v[c][i], ds, vdst + i * 1024, rank * SLOT_BYTES + c * 65536, 17);
            asm volatile("" ::: "memory");
            int spin = 0;
#pragma unroll 1
            while (vm_outstanding() != 0u) {
                if (++spin > (c == 0 ? PUSH_SPIN : 0) && nb < 7 + 7 * c) { BAR(); ++nb; spin = 0; }
                else __builtin_amdgcn_s_sleep(1);
            }
            asm volatile("" ::: "memory");
            if (lane == 0) __builtin_amdgcn_raw_buffer_store_b32(call, ds, fdst, c * 1024, 17);
            asm volatile("" ::: "memory");
            if (c == 0) STAMP(34);
        }
        STAMP(35);
        STAMP_FLUSH(32, 40);
        return;
    }

    if (wid == 0) STAMP(0);
    const int wm = wid & 3, wn = wid >> 2;
    const int fr = lane & 15, g = lane >> 4, q = fr >> 2, p = lane & 3;
    unsigned aoff[2], boff[4];
#pragma unroll
    for (int kk = 0; kk < 2; ++kk) aoff[kk] = ldsbase + (unsigned)((wm * 64 + fr) * 128 + (((kk * 4 + g) ^ (fr >> 1)) << 4));
#pragma unroll
    for (int f = 0; f < 4; ++f)
        boff[f] = ldsbase + LDS_B0 + (unsigned)((8 * g + q) * 256 + (((wn * 8 + 2 * p + (f >> 1)) ^ (q | ((g & 1) << 3))) << 4) + ((((f & 1) ^ (p & 1))) << 3));
    asm volatile("" : "+v"(boff[0]), "+v"(boff[1]), "+v"(boff[2]), "+v"(boff[3]), "+v"(aoff[0]), "+v"(aoff[1]));
    if (wid < 2) {
        const __amdgpu_buffer_rsrc_t xs = __builtin_amdgcn_make_buffer_rsrc((void*)a.x, 0, X_BYTES, SRD3);
        const unsigned po = (unsigned)(idx * 16 + fr) * 1024u + (unsigned)(wid == 0 ? 3 + g : 7) * 128u;
        asm volatile("buffer_load_dword v161, %0, %1, %2 offen" :: "v"(po), "s"(xs), "s"((unsigned)rank * SLOT_BYTES) : "v161", "memory");
    }
    const unsigned call = __builtin_amdgcn_readfirstlane(__hip_atomic_load(seqp, __ATOMIC_RELAXED, __HIP_MEMORY_SCOPE_AGENT)) + 1u;

    f32x4 acc[4][4];
#pragma unroll
    for (int m = 0; m < 4; ++m)
#pragma unroll
        for (int f = 0; f < 4; ++f) acc[m][f] = (f32x4){0.f, 0.f, 0.f, 0.f};
    bf16x8 x0[4], w0[4], x1[4], w1[4];
    bf16x4 tl[4];
    unsigned bcur[4], acur = aoff[0];
#pragma unroll
    for (int f = 0; f < 4; ++f) bcur[f] = boff[f];
#define RD_WL(f, kk) do { tl[f] = __builtin_amdgcn_ds_read_tr16_b64_v4bf16((LAS bf16x4*)(bcur[f] + (kk) * 8192)); } while (0)
#define RD_WH(dst, f, kk) do { const bf16x4 hi_ = __builtin_amdgcn_ds_read_tr16_b64_v4bf16((LAS bf16x4*)(bcur[f] + (kk) * 8192 + 1024)); \
        dst = __builtin_shufflevector(tl[f], hi_, 0, 1, 2, 3, 4, 5, 6, 7); } while (0)
#define RD_XA(dst, m) do { dst = *(const LAS bf16x8*)(acur + (m) * 2048); } while (0)
    asm volatile("" ::: "memory");
    BAR();
    asm volatile("" ::: "memory");
    if (wid == 0) { STAMP(2); CSTAMP(30); }
#pragma unroll
    for (int i = 0; i < 4; ++i) { RD_WL(i, 0); RD_WH(w0[i], i, 0); RD_XA(x0[i], i); }
    acur = aoff[1];
#define STEP(P0, P1, cw, cx, nw, nx, kk, X0, X1) do { \
        PRIO(P0); \
        acc[0][0] = MFMA(cw[0], cx[0], acc[0][0]); RD_WL(0, kk); SCHED(); \
        acc[0][1] = MFMA(cw[1], cx[0], acc[0][1]); RD_WH(nw[0], 0, kk); SCHED(); \
        acc[0][2] = MFMA(cw[2], cx[0], acc[0][2]); RD_XA(nx[0], 0); SCHED(); \
        acc[0][3] = MFMA(cw[3], cx[0], acc[0][3]); RD_WL(1, kk); SCHED(); \
        acc[1][3] = MFMA(cw[3], cx[1], acc[1][3]); RD_WH(nw[1], 1, kk); SCHED(); \
        acc[1][2] = MFMA(cw[2], cx[1], acc[1][2]); RD_WL(2, kk); SCHED(); \
        acc[1][1] = MFMA(cw[1], cx[1], acc[1][1]); RD_WH(nw[2], 2, kk); SCHED(); \
        acc[1][0] = MFMA(cw[0], cx[1], acc[1][0]); RD_WL(3, kk); SCHED(); \
        PRIO(P1); \
        acc[2][0] = MFMA(cw[0], cx[2], acc[2][0]); RD_WH(nw[3], 3, kk); SCHED(); \
        acc[2][1] = MFMA(cw[1], cx[2], acc[2][1]); RD_XA(nx[1], 1); SCHED(); \
        acc[2][2] = MFMA(cw[2], cx[2], acc[2][2]); RD_XA(nx[2], 2); SCHED(); \
        acc[2][3] = MFMA(cw[3], cx[2], acc[2][3]); RD_XA(nx[3], 3); SCHED(); \
        acc[3][3] = MFMA(cw[3], cx[3], acc[3][3]); X0; SCHED(); \
        acc[3][2] = MFMA(cw[2], cx[3], acc[3][2]); X1; SCHED(); \
        acc[3][1] = MFMA(cw[1], cx[3], acc[3][1]); SCHED(); \
        acc[3][0] = MFMA(cw[0], cx[3], acc[3][0]); SCHED(); \
    } while (0)
    {
        unsigned sa = 0, sb = 0;
#pragma unroll 1
        for (int t = 0; t < NT; ++t) {
            const unsigned san = (sa == RA - 1) ? 0u : sa + 1u, sbn = (sb + 1u) & (RB - 1);
            SCHED();
            STEP(1, 0, w0, x0, w1, x1, 1,
                 { bcur[0] = boff[0] + sbn * B_STAGE; bcur[1] = boff[1] + sbn * B_STAGE; acur = aoff[0] + san * A_STAGE; },
                 { bcur[2] = boff[2] + sbn * B_STAGE; bcur[3] = boff[3] + sbn * B_STAGE; });
            WAIT_L(0); BAR(); SCHED();
            STEP(3, 2, w1, x1, w0, x0, 0, { acur = aoff[1] + san * A_STAGE; }, {});
            sa = san; sb = sbn;
        }
    }
    float mx = 0.f;
#pragma unroll
    for (int m = 0; m < 4; ++m)
#pragma unroll
        for (int f = 0; f < 4; ++f) mx = fmaxf(fmaxf(mx, fmaxf(acc[m][f][0], acc[m][f][1])), fmaxf(acc[m][f][2], acc[m][f][3]));
    const unsigned wmax = wave_umax(__float_as_uint(mx));
    if (wid == 0) { STAMP(11); CSTAMP(31); }
    if (lane == 0) red[wid] = wmax;
    WAIT_L(0); BAR();
    asm volatile("" ::: "memory");
    const __amdgpu_buffer_rsrc_t wsr = __builtin_amdgcn_make_buffer_rsrc((void*)wsl, 0, WS_SEQ, SRD3);
    if (wid == 0) {
        const unsigned wmx = wave_umax(lane < 10 ? red[lane] : 0u);
        char* pw = a.ws[0];
#pragma unroll
        for (int r = 1; r < NR; ++r) pw = (lane == r) ? a.ws[r] : pw;
        if (lane < NR)
            __hip_atomic_store((u64*)(pw + WS_GRAN) + rank * NWG + wg, ((u64)call << 32) | wmx, __ATOMIC_RELAXED, __HIP_MEMORY_SCOPE_SYSTEM);
        if (lane == 0) __hip_atomic_store(seqp, call, __ATOMIC_RELAXED, __HIP_MEMORY_SCOPE_AGENT);
        STAMP(12);
    }
    asm volatile("" ::: "memory");
    BAR();
    asm volatile("" ::: "memory");
    unsigned gm = 0;
    {
        const u64 t0 = __builtin_amdgcn_s_memrealtime();
        unsigned polls = 0;
        bool need0 = true, need1 = true;
        for (;;) {
            asm volatile("" ::: "memory");
            u32x4 v0 = {0u, 0u, 0u, 0u}, v1 = {0u, 0u, 0u, 0u};
            if (need0) v0 = __builtin_amdgcn_raw_buffer_load_b128(wsr, WS_GRAN + tid * 16, 0, 17);
            if (need1) v1 = __builtin_amdgcn_raw_buffer_load_b128(wsr, WS_GRAN + 8192 + tid * 16, 0, 17);
            if (need0 && v0.y == call && v0.w == call) { need0 = false; gm = max(gm, max(v0.x, v0.z)); }
            if (need1 && v1.y == call && v1.w == call) { need1 = false; gm = max(gm, max(v1.x, v1.z)); }
            if (!__any(need0 || need1)) break;
            if (timed_out(tmo, call, t0, polls)) { gm = 0xffffffffu; break; }
            __builtin_amdgcn_s_sleep(POLL_SLEEP);
        }
    }
    gm = wave_umax(gm);
    if (lane == 0) red[16 + wid] = gm;
    WAIT_L(0); BAR();
    asm volatile("" ::: "memory");
    gm = wave_umax(lane < 8 ? red[16 + lane] : 0u);
    if (wid == 0) STAMP(13);
    const bool bad = gm == 0xffffffffu;
    const float amax = __uint_as_float(gm);
    const float inv = amax > 0.f ? 448.0f / amax : 0.f;
    char* outp = (char*)a.out + (size_t)(mtile * BM + wm * 64 + fr) * NDIM + ntile * BN + wn * 64 + g * 16;
    const bool odd = g & 1;
#pragma unroll
    for (int m = 0; m < 4; ++m) {
        unsigned d[4];
#pragma unroll
        for (int f = 0; f < 4; ++f) {
            const f32x4 y = acc[m][f];
            const float y0 = __builtin_amdgcn_fmed3f(y[0] * inv, 0.f, 448.f), y1 = __builtin_amdgcn_fmed3f(y[1] * inv, 0.f, 448.f);
            const float y2 = __builtin_amdgcn_fmed3f(y[2] * inv, 0.f, 448.f), y3 = __builtin_amdgcn_fmed3f(y[3] * inv, 0.f, 448.f);
            int w = __builtin_amdgcn_cvt_pk_fp8_f32(y0, y1, 0, false);
            w = __builtin_amdgcn_cvt_pk_fp8_f32(y2, y3, w, true);
            d[f] = bad ? 0x7f7f7f7fu : (unsigned)w;
        }
        u32x4 o;
        o.x = odd ? d[1] : d[0]; o.y = odd ? d[0] : d[1]; o.z = odd ? d[3] : d[2]; o.w = odd ? d[2] : d[3];
        *(u32x4*)(outp + (size_t)m * 16 * NDIM) = o;
    }
}

}

extern "C" void kernel_launch(int rank, int nranks, ncclComm_t comm,
                              hipStream_t stream,
                              void* const* d_in, const int* in_sizes, int n_in,
                              void* d_out, int out_size,
                              const int* in_dtypes, int out_dtype,
                              void* const* d_out_all,
                              void* const* d_ws_all, size_t ws_bytes) {
    using namespace a2ag;
    if (nranks != NR || (unsigned)rank >= (unsigned)NR || n_in < 2 || ws_bytes < WS_NEED
        || in_sizes[0] != KDIM * KSL || in_sizes[1] != KDIM * NDIM || out_size != MLOC * NDIM) {
        fprintf(stderr, "kernel_launch: unexpected problem geometry\n");
        return;
    }
    HIP_CHECK(hipFuncSetAttribute((const void*)a2a_gemm_kernel, hipFuncAttributeMaxDynamicSharedMemorySize, LDS_BYTES));
    Args a;
    a.x = d_in[0]; a.w = d_in[1]; a.out = d_out; a.rank = rank;
    for (int r = 0; r < NR; ++r) a.ws[r] = (char*)d_ws_all[r];
    hipLaunchKernelGGL(a2a_gemm_kernel, dim3(NWG), dim3(NTHREADS), LDS_BYTES, stream, a);
    HIP_CHECK(hipGetLastError());
}
```

```cpp
#include <hip/hip_runtime.h>
#include <rccl/rccl.h>
#include <cstdio>
#include <cstdint>
#include <type_traits>

#define HIP_CHECK(expr) do { hipError_t _e = (expr); if (_e != hipSuccess) { \
    fprintf(stderr, "HIP error %s at %s:%d\n", hipGetErrorString(_e), __FILE__, __LINE__); \
    return; } } while (0)

#define PUSH_PACE 2
#define W_NT 0
#define POLL_AHEAD 0
#define XPF 0
#define XPOLL_SLEEP 4
#define POLL_SLEEP 4
#define PUSH0 1
#define PUSH0_SPREAD 2
#define PUSH_DELAY 8
#define PUSH_SPIN 3

#pragma clang diagnostic ignored "-Wint-to-pointer-cast"

namespace a2ag {

typedef __bf16 bf16x4 __attribute__((ext_vector_type(4)));
typedef __bf16 bf16x8 __attribute__((ext_vector_type(8)));
typedef float f32x4 __attribute__((ext_vector_type(4)));
typedef unsigned u32x4 __attribute__((ext_vector_type(4)));
typedef unsigned long long u64;
#define LAS __attribute__((address_space(3)))
#define DI __device__ __forceinline__

constexpr int NR = 8;
constexpr int MLOC = 512, KDIM = 4096, NDIM = 16384, KSL = 512;
constexpr int BM = 256, BN = 128, BK = 64;
constexpr int NT = KDIM / BK;
constexpr int NWG = (MLOC / BM) * (NDIM / BN);
constexpr int NTHREADS = 704;
constexpr unsigned A_STAGE = BM * BK * 2, B_STAGE = BK * BN * 2;
constexpr int RA = 3, RB = 4;
constexpr unsigned LDS_B0 = RA * A_STAGE;
constexpr unsigned LDS_BYTES = LDS_B0 + RB * B_STAGE;
constexpr unsigned SLOT_BYTES = MLOC * KSL * 2;
constexpr unsigned X_BYTES = NR * SLOT_BYTES;
constexpr unsigned W_BYTES = (unsigned)KDIM * NDIM * 2;
constexpr int NCHUNK = 8, NPART = 32;
constexpr unsigned WS_XFLAG = X_BYTES;
constexpr unsigned WS_GRAN = WS_XFLAG + NCHUNK * 1024;
constexpr unsigned WS_SEQ = WS_GRAN + NR * NWG * 8;
constexpr unsigned WS_TMO = WS_SEQ + NWG * 64;
constexpr unsigned WS_DBG = WS_TMO + 4096;
constexpr unsigned WS_NEED = WS_DBG + NWG * 64 * 8;
constexpr u64 DEADLINE_TICKS = 40000000ull;
constexpr unsigned OOB = 0x40000000u;
constexpr unsigned SRD3 = 0x00020000u;

struct Args {
    const void* x; const void* w; void* out;
    char* ws[NR];
    int rank;
};

DI void set_m0(unsigned v) { asm volatile("s_mov_b32 m0, %0\n\ts_nop 0" :: "s"(v) : "memory"); }
template <int OFF> DI void dma_x(__amdgpu_buffer_rsrc_t srd, unsigned voff, unsigned soff) {
    asm volatile("buffer_load_dwordx4 %0, %1, %2 offen offset:%3 sc1 lds" :: "v"(voff), "s"(srd), "s"(soff), "n"(OFF) : "memory");
}
template <int OFF, bool NT_> DI void dma_w(__amdgpu_buffer_rsrc_t srd, unsigned voff, unsigned soff) {
    if constexpr (NT_) asm volatile("buffer_load_dwordx4 %0, %1, %2 offen offset:%3 nt lds" :: "v"(voff), "s"(srd), "s"(soff), "n"(OFF) : "memory");
    else asm volatile("buffer_load_dwordx4 %0, %1, %2 offen offset:%3 lds" :: "v"(voff), "s"(srd), "s"(soff), "n"(OFF) : "memory");
}
template <int N> DI void wait_v() { asm volatile("s_waitcnt vmcnt(%0)" :: "n"(N) : "memory"); }
#define WAIT_L(n) asm volatile("s_waitcnt lgkmcnt(" #n ")" ::: "memory")
#define BAR() __builtin_amdgcn_s_barrier()
#define SCHED() __builtin_amdgcn_sched_barrier(0)
#define PRIO(n) __builtin_amdgcn_s_setprio(n)
#define MFMA(a, b, c) __builtin_amdgcn_mfma_f32_16x16x32_bf16(a, b, c, 0, 0, 0)

#define STAMP(i) do {} while (0)
#define CSTAMP(i) do {} while (0)
#define STAMP_DECL
#define STAMP_FLUSH(lo, hi) do {} while (0)

DI unsigned ktile_of(int t, int rank) {
    const int i = t - 8, c = (i * 37) >> 8, j = i - c * 7;
    return t < 8 ? (unsigned)(rank * 8 + t) : (unsigned)((((rank + 1 + j) & 7) << 3) + c);
}

DI bool timed_out(unsigned* tmo, unsigned call, u64 t0, unsigned& polls) {
    if ((++polls & 31u) != 0u) return false;
    if (__builtin_amdgcn_readfirstlane(__hip_atomic_load(tmo, __ATOMIC_RELAXED, __HIP_MEMORY_SCOPE_AGENT)) == call
        || __builtin_amdgcn_s_memrealtime() - t0 > DEADLINE_TICKS) {
        __hip_atomic_store(tmo, call, __ATOMIC_RELAXED, __HIP_MEMORY_SCOPE_AGENT);
        return true;
    }
    return false;
}
DI unsigned vm_outstanding() {
    const unsigned s = __builtin_amdgcn_s_getreg(7 | (0 << 6) | (31 << 11));
    return (s & 15u) | ((s >> 18) & 48u);
}
DI unsigned wave_umax(unsigned v) {
    v = max(v, (unsigned)__builtin_amdgcn_update_dpp(0, (int)v, 0x111, 0xf, 0xf, false));
    v = max(v, (unsigned)__builtin_amdgcn_update_dpp(0, (int)v, 0x112, 0xf, 0xf, false));
    v = max(v, (unsigned)__builtin_amdgcn_update_dpp(0, (int)v, 0x114, 0xf, 0xf, false));
    v = max(v, (unsigned)__builtin_amdgcn_update_dpp(0, (int)v, 0x118, 0xf, 0xf, false));
    const unsigned a = __builtin_amdgcn_readlane(v, 15), b = __builtin_amdgcn_readlane(v, 31);
    const unsigned c = __builtin_amdgcn_readlane(v, 47), d = __builtin_amdgcn_readlane(v, 63);
    return max(max(a, b), max(c, d));
}

__global__ __launch_bounds__(NTHREADS) void a2a_gemm_kernel(Args a) {
    extern __shared__ __attribute__((aligned(16))) char smem[];
    LAS char* lds = (LAS char*)smem;
    const int tid = threadIdx.x, lane = tid & 63;
    const int wid = __builtin_amdgcn_readfirstlane(tid >> 6);
    const int wg = blockIdx.x, rank = a.rank;
    const int xcd = wg & 7, idx = wg >> 3;
    const int ntile = xcd * 16 + (idx >> 1), mtile = idx & 1;
    const unsigned ldsbase = (unsigned)(size_t)lds;
    LAS unsigned* red = (LAS unsigned*)(lds + LDS_B0 + 3 * B_STAGE);
    char* wsl = a.ws[0];
#pragma unroll
    for (int r = 1; r < NR; ++r) wsl = (rank == r) ? a.ws[r] : wsl;
    unsigned* seqp = (unsigned*)(wsl + WS_SEQ + wg * 64);
    unsigned* tmo = (unsigned*)(wsl + WS_TMO);
    STAMP_DECL;

    if (wid == 8 || wid == 9) {
        PRIO(3);
        const int half = wid - 8;
        const __amdgpu_buffer_rsrc_t ss = __builtin_amdgcn_make_buffer_rsrc((void*)wsl, 0, WS_TMO, SRD3);
        const __amdgpu_buffer_rsrc_t wr = __builtin_amdgcn_make_buffer_rsrc((void*)a.w, 0, W_BYTES, SRD3);
        unsigned vs[2], vw[2];
#pragma unroll
        for (int e = 0; e < 2; ++e) {
            vs[e] = (unsigned)(lane >> 3) * 128u + (unsigned)(((lane & 7) ^ (4 * e + (lane >> 4))) << 4);
            vw[e] = (unsigned)(lane >> 4) * 32768u + (unsigned)(((lane & 15) ^ ((e << 3) | (lane >> 4))) << 4);
        }
        const unsigned row0 = (unsigned)(mtile * BM + half * 128);
        const unsigned lA = ldsbase + (unsigned)half * 16384u, lB = ldsbase + LDS_B0 + (unsigned)half * 8192u;
        const unsigned soB = (unsigned)ntile * (BN * 2u) + (unsigned)half * (32u * 32768u);
        const unsigned soOwn = (unsigned)rank * SLOT_BYTES + row0 * 1024u;
        const char* xb = (const char*)a.x;
        unsigned dx = 7168u, v0 = vs[0] + (unsigned)(lane >> 3) * 896u, v1 = vs[1] + (unsigned)(lane >> 3) * 896u;
        unsigned la = lA;
        auto issueA = [&](unsigned so) {
            const __amdgpu_buffer_rsrc_t srd = __builtin_amdgcn_make_buffer_rsrc((void*)xb, 0, X_BYTES, SRD3);
#pragma unroll
            for (int p = 0; p < 16; p += 4) {
                const unsigned s4 = so + p * (dx + 1024u);
                set_m0(la + p * 1024u);
                dma_x<0>(srd, v0, s4); dma_x<1024>(srd, v1, s4 + dx); dma_x<2048>(srd, v0, s4 + 2u * dx); dma_x<3072>(srd, v1, s4 + 3u * dx);
            }
            la = (la == lA + (RA - 1) * A_STAGE) ? lA : la + A_STAGE;
        };
        auto issueAs = [&](unsigned so, unsigned l) {
#pragma unroll
            for (int p = 0; p < 16; p += 4) {
                set_m0(l + p * 1024u);
                dma_x<0>(ss, vs[0], so + p * 1024u); dma_x<1024>(ss, vs[1], so + p * 1024u); dma_x<2048>(ss, vs[0], so + p * 1024u); dma_x<3072>(ss, vs[1], so + p * 1024u);
            }
        };
        auto wstage = [&](int t) -> unsigned { return lB + ((unsigned)t & (RB - 1)) * B_STAGE; };
        auto issueW = [&](unsigned so, unsigned lb, auto lo, auto hi) {
            constexpr int LO = decltype(lo)::value, HI = decltype(hi)::value;
            constexpr bool N = (HI - LO == 8) ? (W_NT & 2) != 0 : (W_NT & 1) != 0;
            if constexpr (LO == 0) {
                set_m0(lb);
                dma_w<0, N>(wr, vw[0], so); dma_w<1024, N>(wr, vw[0], so + 130048u); dma_w<2048, N>(wr, vw[1], so + 260096u); dma_w<3072, N>(wr, vw[1], so + 390144u);
            }
            set_m0(lb + 4096u);
            if constexpr (LO <= 4) { dma_w<0, N>(wr, vw[0], so + 524288u); dma_w<1024, N>(wr, vw[0], so + 654336u); }
            if constexpr (HI == 8) { dma_w<2048, N>(wr, vw[1], so + 784384u); dma_w<3072, N>(wr, vw[1], so + 914432u); }
        };
        using I0 = std::integral_constant<int, 0>; using I6 = std::integral_constant<int, 6>; using I8 = std::integral_constant<int, 8>;
        const unsigned fvo = WS_XFLAG - 1024u + (unsigned)lane * 16u;
#define POLL_ISSUE(n) asm volatile("buffer_load_dwordx4 v[160:163], %0, %1, %2 offen sc0 sc1" :: "v"(fvo), "s"(ss), "s"((n) * 1024u) : "v160", "v161", "v162", "v163", "memory")
        const unsigned soW0 = (unsigned)rank * (8u * BK * 32768u) + soB;
        if (half == 0) STAMP(56);
        issueA(soOwn); issueW(soW0, wstage(0), I0{}, I8{});
        if (half == 0) STAMP(57);
        asm volatile("buffer_load_dword v164, off, %0, %1 sc1" :: "s"(ss), "s"(WS_SEQ + (unsigned)wg * 64u) : "v164", "memory");
        issueW(soW0 + BK * 32768u, wstage(1), I0{}, I8{}); issueA(soOwn + 128u);
        unsigned tabA, tabW;
        {
            const int t = lane + 3;
            const unsigned kt = ktile_of(t, rank);
            const unsigned first = (int)(kt >> 3) == ((rank + 1) & 7) ? 16u : 0u;
            tabA = t < 8 ? soOwn + (unsigned)t * 128u : (t < NT ? kt * 65536u + row0 * 128u + first + (kt & 7u) + 1u : OOB);
            auto chunks = [&](int u) -> unsigned { return u < 8 ? 0u : (u < NT ? (ktile_of(u, rank) & 7u) + 1u : (unsigned)NCHUNK); };
            const unsigned a0 = min(chunks(t), 1u), a2 = min(chunks(t + 2), 1u), b0 = chunks(t + POLL_AHEAD), b2 = chunks(t + 2 + POLL_AHEAD);
            tabW = t < NT ? kt * (BK * 32768u) + soB + (a2 > a0 ? 1u : ((b2 > b0 && b2 > 1u) ? b2 : 0u)) : OOB;
        }
        if (half == 0) STAMP(58);
        wait_v<24>();
        if (half == 0) STAMP(59);
        BAR();
        issueW(soW0 + 2u * BK * 32768u, wstage(2), I0{}, I8{}); issueA(soOwn + 256u);
        if (mtile == 0) issueW(soW0 + 3u * BK * 32768u, wstage(3), I0{}, I6{});
        unsigned call;
        asm volatile("v_readfirstlane_b32 %0, v164" : "=s"(call) :: "memory");
        call += 1u;
        const bool own = (lane >> 3) == rank;
        bool good = true;
        unsigned prog = 0;
        auto flags_ok = [&]() -> bool {
            u32x4 f;
            asm volatile("v_mov_b32 %0, v160\n\tv_mov_b32 %1, v161\n\tv_mov_b32 %2, v162\n\tv_mov_b32 %3, v163" : "=v"(f.x), "=v"(f.y), "=v"(f.z), "=v"(f.w) :: "memory");
            return __all(own || (int)((f.x - call) | (f.y - call) | (f.z - call) | (f.w - call)) >= 0);
        };
        const unsigned pfo = (unsigned)(lane >> 3) * SLOT_BYTES + (unsigned)((idx * 2 + half) * 8 + (lane & 7)) * 128u - 65536u;
        auto arrived = [&](unsigned n) {
            prog = n;
        };
        auto await = [&](unsigned need) {
            if (__builtin_expect(prog < need, 0)) {
                if (half == 0) STAMP(40 + need - 1);
                const u64 t0 = __builtin_amdgcn_s_memrealtime();
                unsigned polls = 0;
                for (;;) {
                    POLL_ISSUE(need); wait_v<0>();
                    if (flags_ok()) break;
                    if (timed_out(tmo, call, t0, polls)) { good = false; break; }
                    __builtin_amdgcn_s_sleep(XPOLL_SLEEP);
                }
                arrived(need);
                if (half == 0) STAMP(48 + need - 1);
            }
        };
        auto pick_x = [&](int i) {
            const bool mine = i < 5;
            xb = mine ? (const char*)a.x : (const char*)wsl;
            dx = mine ? 7168u : 0u;
            const unsigned dd = mine ? 896u : 0u;
            v0 = vs[0] + (unsigned)(lane >> 3) * dd; v1 = vs[1] + (unsigned)(lane >> 3) * dd;
        };
        unsigned pc = 0;
        int i = 0;
#pragma unroll 1
        for (; i < 5 + mtile; ++i) {
            pick_x(i);
            const unsigned ta = __builtin_amdgcn_readlane(tabA, i), w3 = __builtin_amdgcn_readlane(tabW, i), w4 = __builtin_amdgcn_readlane(tabW, i + 1);
            if ((i ^ mtile) & 1) {
                wait_v<24>(); BAR();
                pc = w3 & 15u;
                if (pc != 0u) POLL_ISSUE(pc);
                issueA(ta & ~127u);
                issueW(w3 & ~255u, wstage(i + 3), I0{}, I8{});
                issueW(w4 & ~255u, wstage(i + 4), I0{}, I6{});
            } else {
                wait_v<30>(); BAR();
                issueW(w3 & ~255u, wstage(i + 3), I6{}, I8{});
                if (pc != 0u) { if (flags_ok()) arrived(pc); }
                await(ta & 15u);
                issueA(ta & ~127u);
            }
        }
        unsigned aH, aL, wH, wE, laH, laL, lbH, lbE, nH, nL, rareH, rareL;
        auto plan = [&](int h) {
            const unsigned t0 = __builtin_amdgcn_readlane(tabA, h & 63), t1 = __builtin_amdgcn_readlane(tabA, (h + 1) & 63);
            const unsigned u0 = __builtin_amdgcn_readlane(tabW, h & 63), u1 = __builtin_amdgcn_readlane(tabW, (h + 1) & 63);
            aH = t0 & ~127u; nH = t0 & 15u; aL = t1 & ~127u; nL = t1 & 15u; wH = u0 & ~255u; pc = u0 & 15u; wE = u1 & ~255u;
            laH = la; laL = (la == lA + (RA - 1) * A_STAGE) ? lA : la + A_STAGE; la = (laL == lA + (RA - 1) * A_STAGE) ? lA : laL + A_STAGE;
            lbH = wstage(h + 3); lbE = wstage(h + 4);
            rareH = pc | (t0 & 16u); rareL = pc | (t1 & 16u);
            asm volatile("" : "+s"(aH), "+s"(aL), "+s"(wH), "+s"(wE), "+s"(laH), "+s"(laL), "+s"(lbH), "+s"(lbE), "+s"(rareH), "+s"(rareL));
        };
        plan(i);
#pragma unroll 1
        for (int n = 0; n < 29; ++n) {
            wait_v<24>(); BAR();
            if (__builtin_expect(rareH != 0u, 0)) { await(nH); if (pc != 0u) POLL_ISSUE(pc); }
            issueAs(aH, laH);
            issueW(wH, lbH, I0{}, I8{});
            issueW(wE, lbE, I0{}, I6{});
            wait_v<30>(); BAR();
            issueW(wE, lbE, I6{}, I8{});
            if (__builtin_expect(rareL != 0u, 0)) { if (pc != 0u && flags_ok()) arrived(pc); await(nL); }
            issueAs(aL, laL);
            i += 2;
            plan(i);
        }
        if (mtile == 0) BAR();
        if (lane == 0) red[8 + half] = good ? 0u : 0xffffffffu;
        WAIT_L(0); BAR();
        if (half == 0) { STAMP_FLUSH(16, 24); STAMP_FLUSH(40, 64); }
        return;
    }
    if (wid == 10) {
        if (wg >= 7 * NPART) return;
        STAMP(32);
        PRIO(3);
        const int j = wg % 7 + 1, part = wg / 7, dst = (rank + j) & 7;
        char* wsd = a.ws[0];
#pragma unroll
        for (int r = 1; r < NR; ++r) wsd = (dst == r) ? a.ws[r] : wsd;
        const __amdgpu_buffer_rsrc_t xs = __builtin_amdgcn_make_buffer_rsrc((void*)a.x, 0, X_BYTES, SRD3);
        const __amdgpu_buffer_rsrc_t ds = __builtin_amdgcn_make_buffer_rsrc((void*)wsd, 0, WS_GRAN, SRD3);
        const unsigned vsrc = (unsigned)(part * 16 + (lane >> 3)) * 1024u + (unsigned)(lane & 7) * 16u;
        const unsigned vdst = (unsigned)part * 2048u + (unsigned)lane * 16u;
        __builtin_amdgcn_s_sleep(PUSH_DELAY);
        u32x4 v[NCHUNK][2];
        const unsigned call = __hip_atomic_load(seqp, __ATOMIC_RELAXED, __HIP_MEMORY_SCOPE_AGENT) + 1u;
#pragma unroll
        for (int c = 0; c < NCHUNK; ++c)
#pragma unroll
            for (int i = 0; i < 2; ++i)
                v[c][i] = __builtin_amdgcn_raw_buffer_load_b128(xs, vsrc + c * 128 + i * 8192, dst * SLOT_BYTES, 0);
        STAMP(33);
        const unsigned fdst = WS_XFLAG + (unsigned)rank * 128u + (unsigned)part * 4u;
        int nb = 0;
#pragma unroll
        for (int c = 0; c < NCHUNK; ++c) {
            const int slot = c == 0 ? PUSH0 + part % PUSH0_SPREAD : PUSH_PACE * c + (part & 2) + mtile;
#pragma unroll 1
            while (nb < slot) { BAR(); ++nb; }
            asm volatile("" ::: "memory");
#pragma unroll
            for (int i = 0; i < 2; ++i)
                __builtin_amdgcn_raw_buffer_store_b128(v[c][i], ds, vdst + i * 1024, rank * SLOT_BYTES + c * 65536, 17);
            asm volatile("" ::: "memory");
            int spin = 0;
#pragma unroll 1
            for (;;) {
                const bool last = nb >= 7 + 7 * c;
                if ((c == 0 || last || ((nb ^ mtile) & 1) == 0) && vm_outstanding() == 0u) break;
                if (!last && ++spin > (c == 0 ? PUSH_SPIN : 0)) { BAR(); ++nb; spin = 0; }
                else __builtin_amdgcn_s_sleep(1);
            }
            asm volatile("" ::: "memory");
            if (lane == 0) __builtin_amdgcn_raw_buffer_store_b32(call, ds, fdst, c * 1024, 17);
            asm volatile("" ::: "memory");
            if (c == 0) STAMP(34);
        }
        STAMP(35);
        STAMP_FLUSH(32, 40);
        return;
    }

    if (wid == 0) STAMP(0);
    const int wm = wid & 3, wn = wid >> 2;
    const int fr = lane & 15, g = lane >> 4, q = fr >> 2, p = lane & 3;
    unsigned aoff[2], boff[4];
#pragma unroll
    for (int kk = 0; kk < 2; ++kk) aoff[kk] = ldsbase + (unsigned)((wm * 64 + fr) * 128 + (((kk * 4 + g) ^ (fr >> 1)) << 4));
#pragma unroll
    for (int f = 0; f < 4; ++f)
        boff[f] = ldsbase + LDS_B0 + (unsigned)((8 * g + q) * 256 + (((wn * 8 + 2 * p + (f >> 1)) ^ (q | ((g & 1) << 3))) << 4) + ((((f & 1) ^ (p & 1))) << 3));
    asm volatile("" : "+v"(boff[0]), "+v"(boff[1]), "+v"(boff[2]), "+v"(boff[3]), "+v"(aoff[0]), "+v"(aoff[1]));
    if (wid < 2) {
        const __amdgpu_buffer_rsrc_t xs = __builtin_amdgcn_make_buffer_rsrc((void*)a.x, 0, X_BYTES, SRD3);
        const unsigned po = (unsigned)(idx * 16 + fr) * 1024u + (unsigned)(wid == 0 ? 3 + g : 7) * 128u;
        asm volatile("buffer_load_dword v161, %0, %1, %2 offen" :: "v"(po), "s"(xs), "s"((unsigned)rank * SLOT_BYTES) : "v161", "memory");
    }
    const unsigned call = __builtin_amdgcn_readfirstlane(__hip_atomic_load(seqp, __ATOMIC_RELAXED, __HIP_MEMORY_SCOPE_AGENT)) + 1u;

    f32x4 acc[4][4];
#pragma unroll
    for (int m = 0; m < 4; ++m)
#pragma unroll
        for (int f = 0; f < 4; ++f) acc[m][f] = (f32x4){0.f, 0.f, 0.f, 0.f};
    bf16x8 x0[4], w0[4], x1[4], w1[4];
    bf16x4 tl[4];
    unsigned bcur[4], acur = aoff[0];
#pragma unroll
    for (int f = 0; f < 4; ++f) bcur[f] = boff[f];
#define RD_WL(f, kk) do { tl[f] = __builtin_amdgcn_ds_read_tr16_b64_v4bf16((LAS bf16x4*)(bcur[f] + (kk) * 8192)); } while (0)
#define RD_WH(dst, f, kk) do { const bf16x4 hi_ = __builtin_amdgcn_ds_read_tr16_b64_v4bf16((LAS bf16x4*)(bcur[f] + (kk) * 8192 + 1024)); \
        dst = __builtin_shufflevector(tl[f], hi_, 0, 1, 2, 3, 4, 5, 6, 7); } while (0)
#define RD_XA(dst, m) do { dst = *(const LAS bf16x8*)(acur + (m) * 2048); } while (0)
    if (wid == 0) STAMP(1);
    asm volatile("" ::: "memory");
    BAR();
    asm volatile("" ::: "memory");
    if (wid == 0) { STAMP(2); CSTAMP(30); }
#pragma unroll
    for (int i = 0; i < 4; ++i) { RD_WL(i, 0); RD_WH(w0[i], i, 0); RD_XA(x0[i], i); }
    acur = aoff[1];
#define STEP(P0, P1, cw, cx, nw, nx, kk, X0, X1) do { \
        PRIO(P0); \
        acc[0][0] = MFMA(cw[0], cx[0], acc[0][0]); RD_WL(0, kk); SCHED(); \
        acc[0][1] = MFMA(cw[1], cx[0], acc[0][1]); RD_WH(nw[0], 0, kk); SCHED(); \
        acc[0][2] = MFMA(cw[2], cx[0], acc[0][2]); RD_XA(nx[0], 0); SCHED(); \
        acc[0][3] = MFMA(cw[3], cx[0], acc[0][3]); RD_WL(1, kk); SCHED(); \
        acc[1][3] = MFMA(cw[3], cx[1], acc[1][3]); RD_WH(nw[1], 1, kk); SCHED(); \
        acc[1][2] = MFMA(cw[2], cx[1], acc[1][2]); RD_WL(2, kk); SCHED(); \
        acc[1][1] = MFMA(cw[1], cx[1], acc[1][1]); RD_WH(nw[2], 2, kk); SCHED(); \
        acc[1][0] = MFMA(cw[0], cx[1], acc[1][0]); RD_WL(3, kk); SCHED(); \
        PRIO(P1); \
        acc[2][0] = MFMA(cw[0], cx[2], acc[2][0]); RD_WH(nw[3], 3, kk); SCHED(); \
        acc[2][1] = MFMA(cw[1], cx[2], acc[2][1]); RD_XA(nx[1], 1); SCHED(); \
        acc[2][2] = MFMA(cw[2], cx[2], acc[2][2]); RD_XA(nx[2], 2); SCHED(); \
        acc[2][3] = MFMA(cw[3], cx[2], acc[2][3]); RD_XA(nx[3], 3); SCHED(); \
        acc[3][3] = MFMA(cw[3], cx[3], acc[3][3]); X0; SCHED(); \
        acc[3][2] = MFMA(cw[2], cx[3], acc[3][2]); X1; SCHED(); \
        acc[3][1] = MFMA(cw[1], cx[3], acc[3][1]); SCHED(); \
        acc[3][0] = MFMA(cw[0], cx[3], acc[3][0]); SCHED(); \
    } while (0)
    {
        unsigned sa = 0, sb = 0;
#pragma unroll 1
        for (int t = 0; t < NT; ++t) {
            const unsigned san = (sa == RA - 1) ? 0u : sa + 1u, sbn = (sb + 1u) & (RB - 1);
            SCHED();
            STEP(1, 0, w0, x0, w1, x1, 1,
                 { bcur[0] = boff[0] + sbn * B_STAGE; bcur[1] = boff[1] + sbn * B_STAGE; acur = aoff[0] + san * A_STAGE; },
                 { bcur[2] = boff[2] + sbn * B_STAGE; bcur[3] = boff[3] + sbn * B_STAGE; });
            WAIT_L(0); BAR(); SCHED();
            STEP(3, 2, w1, x1, w0, x0, 0, { acur = aoff[1] + san * A_STAGE; }, {});
            sa = san; sb = sbn;
        }
    }
    float mx = 0.f;
#pragma unroll
    for (int m = 0; m < 4; ++m)
#pragma unroll
        for (int f = 0; f < 4; ++f) mx = fmaxf(fmaxf(mx, fmaxf(acc[m][f][0], acc[m][f][1])), fmaxf(acc[m][f][2], acc[m][f][3]));
    const unsigned wmax = wave_umax(__float_as_uint(mx));
    if (wid == 0) { STAMP(11); CSTAMP(31); }
    if (lane == 0) red[wid] = wmax;
    WAIT_L(0); BAR();
    asm volatile("" ::: "memory");
    const __amdgpu_buffer_rsrc_t wsr = __builtin_amdgcn_make_buffer_rsrc((void*)wsl, 0, WS_SEQ, SRD3);
    if (wid == 0) {
        const unsigned wmx = wave_umax(lane < 10 ? red[lane] : 0u);
        char* pw = a.ws[0];
#pragma unroll
        for (int r = 1; r < NR; ++r) pw = (lane == r) ? a.ws[r] : pw;
        if (lane < NR)
            __hip_atomic_store((u64*)(pw + WS_GRAN) + rank * NWG + wg, ((u64)call << 32) | wmx, __ATOMIC_RELAXED, __HIP_MEMORY_SCOPE_SYSTEM);
        if (lane == 0) __hip_atomic_store(seqp, call, __ATOMIC_RELAXED, __HIP_MEMORY_SCOPE_AGENT);
        STAMP(12);
    }
    asm volatile("" ::: "memory");
    BAR();
    asm volatile("" ::: "memory");
    unsigned gm = 0;
    {
        const u64 t0 = __builtin_amdgcn_s_memrealtime();
        unsigned polls = 0;
        bool need0 = true, need1 = true;
        for (;;) {
            asm volatile("" ::: "memory");
            u32x4 v0 = {0u, 0u, 0u, 0u}, v1 = {0u, 0u, 0u, 0u};
            if (need0) v0 = __builtin_amdgcn_raw_buffer_load_b128(wsr, WS_GRAN + tid * 16, 0, 17);
            if (need1) v1 = __builtin_amdgcn_raw_buffer_load_b128(wsr, WS_GRAN + 8192 + tid * 16, 0, 17);
            if (need0 && v0.y == call && v0.w == call) { need0 = false; gm = max(gm, max(v0.x, v0.z)); }
            if (need1 && v1.y == call && v1.w == call) { need1 = false; gm = max(gm, max(v1.x, v1.z)); }
            if (!__any(need0 || need1)) break;
            if (timed_out(tmo, call, t0, polls)) { gm = 0xffffffffu; break; }
            __builtin_amdgcn_s_sleep(POLL_SLEEP);
        }
    }
    gm = wave_umax(gm);
    if (lane == 0) red[16 + wid] = gm;
    WAIT_L(0); BAR();
    asm volatile("" ::: "memory");
    gm = wave_umax(lane < 8 ? red[16 + lane] : 0u);
    if (wid == 0) STAMP(13);
    const bool bad = gm == 0xffffffffu;
    const float amax = __uint_as_float(gm);
    const float inv = amax > 0.f ? 448.0f / amax : 0.f;
    char* outp = (char*)a.out + (size_t)(mtile * BM + wm * 64 + fr) * NDIM + ntile * BN + wn * 64 + g * 16;
    const bool odd = g & 1;
#pragma unroll
    for (int m = 0; m < 4; ++m) {
        unsigned d[4];
#pragma unroll
        for (int f = 0; f < 4; ++f) {
            const f32x4 y = acc[m][f];
            const float y0 = __builtin_amdgcn_fmed3f(y[0] * inv, 0.f, 448.f), y1 = __builtin_amdgcn_fmed3f(y[1] * inv, 0.f, 448.f);
            const float y2 = __builtin_amdgcn_fmed3f(y[2] * inv, 0.f, 448.f), y3 = __builtin_amdgcn_fmed3f(y[3] * inv, 0.f, 448.f);
            int w = __builtin_amdgcn_cvt_pk_fp8_f32(y0, y1, 0, false);
            w = __builtin_amdgcn_cvt_pk_fp8_f32(y2, y3, w, true);
            d[f] = bad ? 0x7f7f7f7fu : (unsigned)w;
        }
        u32x4 o;
        o.x = odd ? d[1] : d[0]; o.y = odd ? d[0] : d[1]; o.z = odd ? d[3] : d[2]; o.w = odd ? d[2] : d[3];
        *(u32x4*)(outp + (size_t)m * 16 * NDIM) = o;
    }
}

}

extern "C" void kernel_launch(int rank, int nranks, ncclComm_t comm,
                              hipStream_t stream,
                              void* const* d_in, const int* in_sizes, int n_in,
                              void* d_out, int out_size,
                              const int* in_dtypes, int out_dtype,
                              void* const* d_out_all,
                              void* const* d_ws_all, size_t ws_bytes) {
    using namespace a2ag;
    if (nranks != NR || (unsigned)rank >= (unsigned)NR || n_in < 2 || ws_bytes < WS_NEED
        || in_sizes[0] != KDIM * KSL || in_sizes[1] != KDIM * NDIM || out_size != MLOC * NDIM) {
        fprintf(stderr, "kernel_launch: unexpected problem geometry\n");
        return;
    }
    HIP_CHECK(hipFuncSetAttribute((const void*)a2a_gemm_kernel, hipFuncAttributeMaxDynamicSharedMemorySize, LDS_BYTES));
    Args a;
    a.x = d_in[0]; a.w = d_in[1]; a.out = d_out; a.rank = rank;
    for (int r = 0; r < NR; ++r) a.ws[r] = (char*)d_ws_all[r];
    hipLaunchKernelGGL(a2a_gemm_kernel, dim3(NWG), dim3(NTHREADS), LDS_BYTES, stream, a);
    HIP_CHECK(hipGetLastError());
}
```

```cpp
#include <hip/hip_runtime.h>
#include <rccl/rccl.h>
#include <cstdio>
#include <cstdint>
#include <type_traits>

#define HIP_CHECK(expr) do { hipError_t _e = (expr); if (_e != hipSuccess) { \
    fprintf(stderr, "HIP error %s at %s:%d\n", hipGetErrorString(_e), __FILE__, __LINE__); \
    return; } } while (0)

#define PUSH_PACE 2
#define W_NT 0
#define POLL_AHEAD 0
#define XPF 0
#define TOUCH_W 0
#define TLBW 0
#define SPEC_THR 512
#define SPIN_THR 128
#define FIN_AUX 0
#define OUT_AUX 17
#define PRO 1
#define XPOLL_SLEEP 4
#define POLL_SLEEP 1
#define PUSH0 1
#define PUSH0_SPREAD 2
#define PUSH_DELAY 8
#define PUSH_SPIN 3

#pragma clang diagnostic ignored "-Wint-to-pointer-cast"

namespace a2ag {

typedef __bf16 bf16x4 __attribute__((ext_vector_type(4)));
typedef __bf16 bf16x8 __attribute__((ext_vector_type(8)));
typedef float f32x4 __attribute__((ext_vector_type(4)));
typedef unsigned u32x4 __attribute__((ext_vector_type(4)));
typedef unsigned long long u64;
#define LAS __attribute__((address_space(3)))
#define DI __device__ __forceinline__

constexpr int NR = 8;
constexpr int MLOC = 512, KDIM = 4096, NDIM = 16384, KSL = 512;
constexpr int BM = 256, BN = 128, BK = 64;
constexpr int NT = KDIM / BK;
constexpr int NWG = (MLOC / BM) * (NDIM / BN);
constexpr int NTHREADS = 704;
constexpr unsigned A_STAGE = BM * BK * 2, B_STAGE = BK * BN * 2;
constexpr int RA = 3, RB = 4;
constexpr unsigned LDS_B0 = RA * A_STAGE;
constexpr unsigned LDS_BYTES = LDS_B0 + RB * B_STAGE;
constexpr unsigned SLOT_BYTES = MLOC * KSL * 2;
constexpr unsigned X_BYTES = NR * SLOT_BYTES;
constexpr unsigned W_BYTES = (unsigned)KDIM * NDIM * 2;
constexpr int NCHUNK = 8, NPART = 32;
constexpr unsigned WS_XFLAG = X_BYTES;
constexpr unsigned WS_GRAN = WS_XFLAG + NCHUNK * 1024;
constexpr unsigned WS_SEQ = WS_GRAN + NR * NWG * 8;
constexpr unsigned WS_TMO = WS_SEQ + NWG * 64;
constexpr unsigned WS_DBG = WS_TMO + 4096;
constexpr unsigned WS_NEED = WS_DBG + NWG * 64 * 8;
constexpr u64 DEADLINE_TICKS = 40000000ull;
constexpr unsigned OOB = 0x40000000u;
constexpr unsigned SRD3 = 0x00020000u;

struct Args {
    const void* x; const void* w; void* out;
    char* ws[NR];
    int rank;
};

DI void set_m0(unsigned v) { asm volatile("s_mov_b32 m0, %0\n\ts_nop 0" :: "s"(v) : "memory"); }
template <int OFF> DI void dma_x(__amdgpu_buffer_rsrc_t srd, unsigned voff, unsigned soff) {
    asm volatile("buffer_load_dwordx4 %0, %1, %2 offen offset:%3 sc1 lds" :: "v"(voff), "s"(srd), "s"(soff), "n"(OFF) : "memory");
}
template <int OFF, bool NT_> DI void dma_w(__amdgpu_buffer_rsrc_t srd, unsigned voff, unsigned soff) {
    if constexpr (NT_) asm volatile("buffer_load_dwordx4 %0, %1, %2 offen offset:%3 nt lds" :: "v"(voff), "s"(srd), "s"(soff), "n"(OFF) : "memory");
    else asm volatile("buffer_load_dwordx4 %0, %1, %2 offen offset:%3 lds" :: "v"(voff), "s"(srd), "s"(soff), "n"(OFF) : "memory");
}
template <int N> DI void wait_v() { asm volatile("s_waitcnt vmcnt(%0)" :: "n"(N) : "memory"); }
#define WAIT_L(n) asm volatile("s_waitcnt lgkmcnt(" #n ")" ::: "memory")
#define BAR() __builtin_amdgcn_s_barrier()
#define SCHED() __builtin_amdgcn_sched_barrier(0)
#define PRIO(n) __builtin_amdgcn_s_setprio(n)
#define MFMA(a, b, c) __builtin_amdgcn_mfma_f32_16x16x32_bf16(a, b, c, 0, 0, 0)

#define STAMP(i) do {} while (0)
#define CSTAMP(i) do {} while (0)
#define STAMP_DECL
#define STAMP_FLUSH(lo, hi) do {} while (0)

DI unsigned ktile_of(int t, int rank) {
    const int i = t - 8, c = (i * 37) >> 8, j = i - c * 7;
    return t < 8 ? (unsigned)(rank * 8 + t) : (unsigned)((((rank + 1 + j) & 7) << 3) + c);
}

DI bool timed_out(unsigned* tmo, unsigned call, u64 t0, unsigned& polls) {
    if ((++polls & 31u) != 0u) return false;
    if (__builtin_amdgcn_readfirstlane(__hip_atomic_load(tmo, __ATOMIC_RELAXED, __HIP_MEMORY_SCOPE_AGENT)) == call
        || __builtin_amdgcn_s_memrealtime() - t0 > DEADLINE_TICKS) {
        __hip_atomic_store(tmo, call, __ATOMIC_RELAXED, __HIP_MEMORY_SCOPE_AGENT);
        return true;
    }
    return false;
}
DI unsigned vm_outstanding() {
    const unsigned s = __builtin_amdgcn_s_getreg(7 | (0 << 6) | (31 << 11));
    return (s & 15u) | ((s >> 18) & 48u);
}
DI unsigned wave_umax(unsigned v) {
    v = max(v, (unsigned)__builtin_amdgcn_update_dpp(0, (int)v, 0x111, 0xf, 0xf, false));
    v = max(v, (unsigned)__builtin_amdgcn_update_dpp(0, (int)v, 0x112, 0xf, 0xf, false));
    v = max(v, (unsigned)__builtin_amdgcn_update_dpp(0, (int)v, 0x114, 0xf, 0xf, false));
    v = max(v, (unsigned)__builtin_amdgcn_update_dpp(0, (int)v, 0x118, 0xf, 0xf, false));
    const unsigned a = __builtin_amdgcn_readlane(v, 15), b = __builtin_amdgcn_readlane(v, 31);
    const unsigned c = __builtin_amdgcn_readlane(v, 47), d = __builtin_amdgcn_readlane(v, 63);
    return max(max(a, b), max(c, d));
}

__global__ __launch_bounds__(NTHREADS) void a2a_gemm_kernel(Args a) {
    extern __shared__ __attribute__((aligned(16))) char smem[];
    LAS char* lds = (LAS char*)smem;
    const int tid = threadIdx.x, lane = tid & 63;
    const int wid = __builtin_amdgcn_readfirstlane(tid >> 6);
    const int wg = blockIdx.x, rank = a.rank;
    const int xcd = wg & 7, idx = wg >> 3;
    const int ntile = xcd * 16 + (idx >> 1), mtile = idx & 1;
    const unsigned ldsbase = (unsigned)(size_t)lds;
    LAS unsigned* red = (LAS unsigned*)(lds + LDS_B0 + 3 * B_STAGE);
    char* wsl = a.ws[0];
#pragma unroll
    for (int r = 1; r < NR; ++r) wsl = (rank == r) ? a.ws[r] : wsl;
    unsigned* seqp = (unsigned*)(wsl + WS_SEQ + wg * 64);
    unsigned* tmo = (unsigned*)(wsl + WS_TMO);
    STAMP_DECL;

    if (wid == 8 || wid == 9) {
        PRIO(3);
        const int half = wid - 8;
        const __amdgpu_buffer_rsrc_t ss = __builtin_amdgcn_make_buffer_rsrc((void*)wsl, 0, WS_TMO, SRD3);
        const __amdgpu_buffer_rsrc_t wr = __builtin_amdgcn_make_buffer_rsrc((void*)a.w, 0, W_BYTES, SRD3);
        unsigned vs[2], vw[2];
#pragma unroll
        for (int e = 0; e < 2; ++e) {
            vs[e] = (unsigned)(lane >> 3) * 128u + (unsigned)(((lane & 7) ^ (4 * e + (lane >> 4))) << 4);
            vw[e] = (unsigned)(lane >> 4) * 32768u + (unsigned)(((lane & 15) ^ ((e << 3) | (lane >> 4))) << 4);
        }
        const unsigned row0 = (unsigned)(mtile * BM + half * 128);
        const unsigned lA = ldsbase + (unsigned)half * 16384u, lB = ldsbase + LDS_B0 + (unsigned)half * 8192u;
        const unsigned soB = (unsigned)ntile * (BN * 2u) + (unsigned)half * (32u * 32768u);
        const unsigned soOwn = (unsigned)rank * SLOT_BYTES + row0 * 1024u;
        const char* xb = (const char*)a.x;
        unsigned dx = 7168u, v0 = vs[0] + (unsigned)(lane >> 3) * 896u, v1 = vs[1] + (unsigned)(lane >> 3) * 896u;
        unsigned la = lA;
        auto issueA = [&](unsigned so) {
            const __amdgpu_buffer_rsrc_t srd = __builtin_amdgcn_make_buffer_rsrc((void*)xb, 0, X_BYTES, SRD3);
#pragma unroll
            for (int p = 0; p < 16; p += 4) {
                const unsigned s4 = so + p * (dx + 1024u);
                set_m0(la + p * 1024u);
                dma_x<0>(srd, v0, s4); dma_x<1024>(srd, v1, s4 + dx); dma_x<2048>(srd, v0, s4 + 2u * dx); dma_x<3072>(srd, v1, s4 + 3u * dx);
            }
            la = (la == lA + (RA - 1) * A_STAGE) ? lA : la + A_STAGE;
        };
        auto issueAs = [&](unsigned so, unsigned l) {
#pragma unroll
            for (int p = 0; p < 16; p += 4) {
                set_m0(l + p * 1024u);
                dma_x<0>(ss, vs[0], so + p * 1024u); dma_x<1024>(ss, vs[1], so + p * 1024u); dma_x<2048>(ss, vs[0], so + p * 1024u); dma_x<3072>(ss, vs[1], so + p * 1024u);
            }
        };
        auto wstage = [&](int t) -> unsigned { return lB + ((unsigned)t & (RB - 1)) * B_STAGE; };
        auto issueW = [&](unsigned so, unsigned lb, auto lo, auto hi) {
            constexpr int LO = decltype(lo)::value, HI = decltype(hi)::value;
            constexpr bool N = (HI - LO == 8) ? (W_NT & 2) != 0 : (W_NT & 1) != 0;
            if constexpr (LO == 0) {
                set_m0(lb);
                dma_w<0, N>(wr, vw[0], so); dma_w<1024, N>(wr, vw[0], so + 130048u); dma_w<2048, N>(wr, vw[1], so + 260096u); dma_w<3072, N>(wr, vw[1], so + 390144u);
            }
            set_m0(lb + 4096u);
            if constexpr (LO <= 4) { dma_w<0, N>(wr, vw[0], so + 524288u); dma_w<1024, N>(wr, vw[0], so + 654336u); }
            if constexpr (HI == 8) { dma_w<2048, N>(wr, vw[1], so + 784384u); dma_w<3072, N>(wr, vw[1], so + 914432u); }
        };
        using I0 = std::integral_constant<int, 0>; using I6 = std::integral_constant<int, 6>; using I8 = std::integral_constant<int, 8>;
        const unsigned fvo = WS_XFLAG - 1024u + (unsigned)lane * 16u;
#define POLL_ISSUE(n) asm volatile("buffer_load_dwordx4 v[160:163], %0, %1, %2 offen sc0 sc1" :: "v"(fvo), "s"(ss), "s"((n) * 1024u) : "v160", "v161", "v162", "v163", "memory")
        const unsigned soW0 = (unsigned)rank * (8u * BK * 32768u) + soB;
        if (half == 0) STAMP(56);
        issueA(soOwn); issueW(soW0, wstage(0), I0{}, I8{});
        if (half == 0) STAMP(57);
        asm volatile("buffer_load_dword v164, off, %0, %1 sc1" :: "s"(ss), "s"(WS_SEQ + (unsigned)wg * 64u) : "v164", "memory");
        if (PRO <= 1) issueW(soW0 + BK * 32768u, wstage(1), I0{}, I8{});
        if (PRO == 0) issueA(soOwn + 128u);
        unsigned tabA, tabW;
        {
            const int t = lane + 3;
            const unsigned kt = ktile_of(t, rank);
            const unsigned first = (int)(kt >> 3) == ((rank + 1) & 7) ? 16u : 0u;
            tabA = t < 8 ? soOwn + (unsigned)t * 128u : (t < NT ? kt * 65536u + row0 * 128u + first + (kt & 7u) + 1u : OOB);
            auto chunks = [&](int u) -> unsigned { return u < 8 ? 0u : (u < NT ? (ktile_of(u, rank) & 7u) + 1u : (unsigned)NCHUNK); };
            const unsigned a0 = min(chunks(t), 1u), a2 = min(chunks(t + 2), 1u), b0 = chunks(t + POLL_AHEAD), b2 = chunks(t + 2 + POLL_AHEAD);
            tabW = t < NT ? kt * (BK * 32768u) + soB + (a2 > a0 ? 1u : ((b2 > b0 && b2 > 1u) ? b2 : 0u)) : OOB;
        }
        if (half == 0) STAMP(58);
        wait_v<PRO == 0 ? 24 : (PRO == 1 ? 8 : 0)>();
        if (half == 0) STAMP(59);
        BAR();
        if (PRO == 2) issueW(soW0 + BK * 32768u, wstage(1), I0{}, I8{});
        if (PRO >= 1) issueA(soOwn + 128u);
        issueW(soW0 + 2u * BK * 32768u, wstage(2), I0{}, I8{}); issueA(soOwn + 256u);
        if (mtile == 0) issueW(soW0 + 3u * BK * 32768u, wstage(3), I0{}, I6{});
        unsigned call;
        asm volatile("v_readfirstlane_b32 %0, v164" : "=s"(call) :: "memory");
        call += 1u;
        const bool own = (lane >> 3) == rank;
        bool good = true;
        unsigned prog = 0;
        auto flags_ok = [&]() -> bool {
            u32x4 f;
            asm volatile("v_mov_b32 %0, v160\n\tv_mov_b32 %1, v161\n\tv_mov_b32 %2, v162\n\tv_mov_b32 %3, v163" : "=v"(f.x), "=v"(f.y), "=v"(f.z), "=v"(f.w) :: "memory");
            return __all(own || (int)((f.x - call) | (f.y - call) | (f.z - call) | (f.w - call)) >= 0);
        };
        const unsigned pfo = (unsigned)(lane >> 3) * SLOT_BYTES + (unsigned)((idx * 2 + half) * 8 + (lane & 7)) * 128u - 65536u;
        auto arrived = [&](unsigned n) {
            prog = n;
        };
        auto await = [&](unsigned need) {
            if (__builtin_expect(prog < need, 0)) {
                if (half == 0) STAMP(40 + need - 1);
                const u64 t0 = __builtin_amdgcn_s_memrealtime();
                unsigned polls = 0;
                for (;;) {
                    POLL_ISSUE(need); wait_v<0>();
                    if (flags_ok()) break;
                    if (timed_out(tmo, call, t0, polls)) { good = false; break; }
                    __builtin_amdgcn_s_sleep(XPOLL_SLEEP);
                }
                arrived(need);
                if (half == 0) STAMP(48 + need - 1);
            }
        };
        auto pick_x = [&](int i) {
            const bool mine = i < 5;
            xb = mine ? (const char*)a.x : (const char*)wsl;
            dx = mine ? 7168u : 0u;
            const unsigned dd = mine ? 896u : 0u;
            v0 = vs[0] + (unsigned)(lane >> 3) * dd; v1 = vs[1] + (unsigned)(lane >> 3) * dd;
        };
        unsigned pc = 0;
        int i = 0;
#pragma unroll 1
        for (; i < 5 + mtile; ++i) {
            pick_x(i);
            const unsigned ta = __builtin_amdgcn_readlane(tabA, i), w3 = __builtin_amdgcn_readlane(tabW, i), w4 = __builtin_amdgcn_readlane(tabW, i + 1);
            if ((i ^ mtile) & 1) {
                wait_v<24>(); BAR();
                pc = w3 & 15u;
                if (pc != 0u) POLL_ISSUE(pc);
                issueA(ta & ~127u);
                issueW(w3 & ~255u, wstage(i + 3), I0{}, I8{});
                issueW(w4 & ~255u, wstage(i + 4), I0{}, I6{});
            } else {
                wait_v<30>(); BAR();
                issueW(w3 & ~255u, wstage(i + 3), I6{}, I8{});
                if (pc != 0u) { if (flags_ok()) arrived(pc); }
                await(ta & 15u);
                issueA(ta & ~127u);
            }
        }
        unsigned aH, aL, wH, wE, laH, laL, lbH, lbE, nH, nL, rareH, rareL;
        auto plan = [&](int h) {
            const unsigned t0 = __builtin_amdgcn_readlane(tabA, h & 63), t1 = __builtin_amdgcn_readlane(tabA, (h + 1) & 63);
            const unsigned u0 = __builtin_amdgcn_readlane(tabW, h & 63), u1 = __builtin_amdgcn_readlane(tabW, (h + 1) & 63);
            aH = t0 & ~127u; nH = t0 & 15u; aL = t1 & ~127u; nL = t1 & 15u; wH = u0 & ~255u; pc = u0 & 15u; wE = u1 & ~255u;
            laH = la; laL = (la == lA + (RA - 1) * A_STAGE) ? lA : la + A_STAGE; la = (laL == lA + (RA - 1) * A_STAGE) ? lA : laL + A_STAGE;
            lbH = wstage(h + 3); lbE = wstage(h + 4);
            rareH = pc | (t0 & 16u); rareL = pc | (t1 & 16u);
            asm volatile("" : "+s"(aH), "+s"(aL), "+s"(wH), "+s"(wE), "+s"(laH), "+s"(laL), "+s"(lbH), "+s"(lbE), "+s"(rareH), "+s"(rareL));
        };
        plan(i);
#pragma unroll 1
        for (int n = 0; n < 29; ++n) {
            wait_v<24>(); BAR();
            if (__builtin_expect(rareH != 0u, 0)) { await(nH); if (pc != 0u) POLL_ISSUE(pc); }
            issueAs(aH, laH);
            issueW(wH, lbH, I0{}, I8{});
            issueW(wE, lbE, I0{}, I6{});
            wait_v<30>(); BAR();
            issueW(wE, lbE, I6{}, I8{});
            if (__builtin_expect(rareL != 0u, 0)) { if (pc != 0u && flags_ok()) arrived(pc); await(nL); }
            issueAs(aL, laL);
            i += 2;
            plan(i);
        }
        if (mtile == 0) BAR();
        if (lane == 0) red[8 + half] = good ? 0u : 0xffffffffu;
        WAIT_L(0); BAR();
        if (half == 0) { STAMP_FLUSH(16, 24); STAMP_FLUSH(40, 64); }
        return;
    }
    if (wid == 10) {
        if (wg >= 7 * NPART) return;
        STAMP(32);
        PRIO(3);
        const int j = wg % 7 + 1, part = wg / 7, dst = (rank + j) & 7;
        char* wsd = a.ws[0];
#pragma unroll
        for (int r = 1; r < NR; ++r) wsd = (dst == r) ? a.ws[r] : wsd;
        const __amdgpu_buffer_rsrc_t xs = __builtin_amdgcn_make_buffer_rsrc((void*)a.x, 0, X_BYTES, SRD3);
        const __amdgpu_buffer_rsrc_t ds = __builtin_amdgcn_make_buffer_rsrc((void*)wsd, 0, WS_GRAN, SRD3);
        const unsigned vsrc = (unsigned)(part * 16 + (lane >> 3)) * 1024u + (unsigned)(lane & 7) * 16u;
        const unsigned vdst = (unsigned)part * 2048u + (unsigned)lane * 16u;
        __builtin_amdgcn_s_sleep(PUSH_DELAY);
        u32x4 v[NCHUNK][2];
        const unsigned call = __hip_atomic_load(seqp, __ATOMIC_RELAXED, __HIP_MEMORY_SCOPE_AGENT) + 1u;
#pragma unroll
        for (int c = 0; c < NCHUNK; ++c)
#pragma unroll
            for (int i = 0; i < 2; ++i)
                v[c][i] = __builtin_amdgcn_raw_buffer_load_b128(xs, vsrc + c * 128 + i * 8192, dst * SLOT_BYTES, 0);
        const unsigned fdst = WS_XFLAG + (unsigned)rank * 128u + (unsigned)part * 4u;
        int nb = 0;
#pragma unroll
        for (int c = 0; c < NCHUNK; ++c) {
            const int slot = c == 0 ? PUSH0 + part % PUSH0_SPREAD : PUSH_PACE * c + (part & 2) + mtile;
#pragma unroll 1
            while (nb < slot) { BAR(); ++nb; }
            asm volatile("" ::: "memory");
#pragma unroll
            for (int i = 0; i < 2; ++i)
                __builtin_amdgcn_raw_buffer_store_b128(v[c][i], ds, vdst + i * 1024, rank * SLOT_BYTES + c * 65536, 17);
            asm volatile("s_nop 1" ::: "memory");
            int spin = 0;
#pragma unroll 1
            for (;;) {
                const bool last = nb >= 7 + 7 * c;
                if ((c == 0 || last || ((nb ^ mtile) & 1) == 0) && vm_outstanding() == 0u) break;
                if (!last && ++spin > (c == 0 ? PUSH_SPIN : 0)) { BAR(); ++nb; spin = 0; }
                else __builtin_amdgcn_s_sleep(1);
            }
            asm volatile("" ::: "memory");
            if (lane == 0) __builtin_amdgcn_raw_buffer_store_b32(call, ds, fdst, c * 1024, 17);
            asm volatile("" ::: "memory");
            if (c == 0) STAMP(34);
        }
        STAMP(35);
        STAMP_FLUSH(32, 40);
        return;
    }

    if (wid == 0) STAMP(0);
    const int wm = wid & 3, wn = wid >> 2;
    const int fr = lane & 15, g = lane >> 4, q = fr >> 2, p = lane & 3;
    unsigned aoff[2], boff[4];
#pragma unroll
    for (int kk = 0; kk < 2; ++kk) aoff[kk] = ldsbase + (unsigned)((wm * 64 + fr) * 128 + (((kk * 4 + g) ^ (fr >> 1)) << 4));
#pragma unroll
    for (int f = 0; f < 4; ++f)
        boff[f] = ldsbase + LDS_B0 + (unsigned)((8 * g + q) * 256 + (((wn * 8 + 2 * p + (f >> 1)) ^ (q | ((g & 1) << 3))) << 4) + ((((f & 1) ^ (p & 1))) << 3));
    asm volatile("" : "+v"(boff[0]), "+v"(boff[1]), "+v"(boff[2]), "+v"(boff[3]), "+v"(aoff[0]), "+v"(aoff[1]));
    if (wid < 2) {
        const __amdgpu_buffer_rsrc_t xs = __builtin_amdgcn_make_buffer_rsrc((void*)a.x, 0, X_BYTES, SRD3);
        const unsigned po = (unsigned)(idx * 16 + fr) * 1024u + (unsigned)(TOUCH_W ? wid * 4 + g : (wid == 0 ? 3 + g : 7)) * 128u;
        asm volatile("buffer_load_dword v161, %0, %1, %2 offen" :: "v"(po), "s"(xs), "s"((unsigned)rank * SLOT_BYTES) : "v161", "memory");
    } else if (wid < 2 + TOUCH_W) {
        const __amdgpu_buffer_rsrc_t wr = __builtin_amdgcn_make_buffer_rsrc((void*)a.w, 0, W_BYTES, SRD3);
        const unsigned po = (unsigned)(mtile * 32 + (lane >> 1)) * 32768u + (unsigned)(lane & 1) * 128u;
        asm volatile("buffer_load_dword v161, %0, %1, %2 offen" :: "v"(po), "s"(wr), "s"((unsigned)(rank * 8 + wid - 1) * (BK * 32768u) + (unsigned)ntile * (BN * 2u)) : "v161", "memory");
    }
    const unsigned call = __builtin_amdgcn_readfirstlane(__hip_atomic_load(seqp, __ATOMIC_RELAXED, __HIP_MEMORY_SCOPE_AGENT)) + 1u;

    u64 gdst;
    {
        char* pw = a.ws[0];
#pragma unroll
        for (int r = 1; r < NR; ++r) pw = (lane == r) ? a.ws[r] : pw;
        gdst = (u64)((u64*)(pw + WS_GRAN) + rank * NWG + wg);
        asm volatile("" : "+v"(gdst));
    }
    f32x4 acc[4][4];
#pragma unroll
    for (int m = 0; m < 4; ++m)
#pragma unroll
        for (int f = 0; f < 4; ++f) acc[m][f] = (f32x4){0.f, 0.f, 0.f, 0.f};
    bf16x8 x0[4], w0[4], x1[4], w1[4];
    bf16x4 tl[4];
    unsigned bcur[4], acur = aoff[0];
#pragma unroll
    for (int f = 0; f < 4; ++f) bcur[f] = boff[f];
#define RD_WL(f, kk) do { tl[f] = __builtin_amdgcn_ds_read_tr16_b64_v4bf16((LAS bf16x4*)(bcur[f] + (kk) * 8192)); } while (0)
#define RD_WH(dst, f, kk) do { const bf16x4 hi_ = __builtin_amdgcn_ds_read_tr16_b64_v4bf16((LAS bf16x4*)(bcur[f] + (kk) * 8192 + 1024)); \
        dst = __builtin_shufflevector(tl[f], hi_, 0, 1, 2, 3, 4, 5, 6, 7); } while (0)
#define RD_XA(dst, m) do { dst = *(const LAS bf16x8*)(acur + (m) * 2048); } while (0)
    if (wid == 0) STAMP(1);
    asm volatile("" ::: "memory");
    BAR();
    asm volatile("" ::: "memory");
    if (wid == 0) { STAMP(2); CSTAMP(30); }
#pragma unroll
    for (int i = 0; i < 4; ++i) { RD_WL(i, 0); RD_WH(w0[i], i, 0); RD_XA(x0[i], i); }
    acur = aoff[1];
#define STEP(P0, P1, cw, cx, nw, nx, kk, X0, X1) do { \
        PRIO(P0); \
        acc[0][0] = MFMA(cw[0], cx[0], acc[0][0]); RD_WL(0, kk); SCHED(); \
        acc[0][1] = MFMA(cw[1], cx[0], acc[0][1]); RD_WH(nw[0], 0, kk); SCHED(); \
        acc[0][2] = MFMA(cw[2], cx[0], acc[0][2]); RD_XA(nx[0], 0); SCHED(); \
        acc[0][3] = MFMA(cw[3], cx[0], acc[0][3]); RD_WL(1, kk); SCHED(); \
        acc[1][3] = MFMA(cw[3], cx[1], acc[1][3]); RD_WH(nw[1], 1, kk); SCHED(); \
        acc[1][2] = MFMA(cw[2], cx[1], acc[1][2]); RD_WL(2, kk); SCHED(); \
        acc[1][1] = MFMA(cw[1], cx[1], acc[1][1]); RD_WH(nw[2], 2, kk); SCHED(); \
        acc[1][0] = MFMA(cw[0], cx[1], acc[1][0]); RD_WL(3, kk); SCHED(); \
        PRIO(P1); \
        acc[2][0] = MFMA(cw[0], cx[2], acc[2][0]); RD_WH(nw[3], 3, kk); SCHED(); \
        acc[2][1] = MFMA(cw[1], cx[2], acc[2][1]); RD_XA(nx[1], 1); SCHED(); \
        acc[2][2] = MFMA(cw[2], cx[2], acc[2][2]); RD_XA(nx[2], 2); SCHED(); \
        acc[2][3] = MFMA(cw[3], cx[2], acc[2][3]); RD_XA(nx[3], 3); SCHED(); \
        acc[3][3] = MFMA(cw[3], cx[3], acc[3][3]); X0; SCHED(); \
        acc[3][2] = MFMA(cw[2], cx[3], acc[3][2]); X1; SCHED(); \
        acc[3][1] = MFMA(cw[1], cx[3], acc[3][1]); SCHED(); \
        acc[3][0] = MFMA(cw[0], cx[3], acc[3][0]); SCHED(); \
    } while (0)
    {
        unsigned sa = 0, sb = 0;
#pragma unroll 1
        for (int t = 0; t < NT; ++t) {
            const unsigned san = (sa == RA - 1) ? 0u : sa + 1u, sbn = (sb + 1u) & (RB - 1);
            SCHED();
            STEP(1, 0, w0, x0, w1, x1, 1,
                 { bcur[0] = boff[0] + sbn * B_STAGE; bcur[1] = boff[1] + sbn * B_STAGE; acur = aoff[0] + san * A_STAGE; },
                 { bcur[2] = boff[2] + sbn * B_STAGE; bcur[3] = boff[3] + sbn * B_STAGE; });
            WAIT_L(0); BAR(); SCHED();
            STEP(3, 2, w1, x1, w0, x0, 0, { acur = aoff[1] + san * A_STAGE; }, {});
            sa = san; sb = sbn;
        }
    }
    const __amdgpu_buffer_rsrc_t wsr = __builtin_amdgcn_make_buffer_rsrc((void*)wsl, 0, WS_SEQ, SRD3);
    u32x4 v0 = __builtin_amdgcn_raw_buffer_load_b128(wsr, WS_GRAN + tid * 16, 0, 17);
    u32x4 v1 = __builtin_amdgcn_raw_buffer_load_b128(wsr, WS_GRAN + 8192 + tid * 16, 0, 17);
    asm volatile("" ::: "memory");
    int mx0 = 0, mx1 = 0;
#pragma unroll
    for (int m = 0; m < 4; ++m)
#pragma unroll
        for (int f = 0; f < 4; ++f) {
            mx0 = max(max(mx0, __float_as_int(acc[m][f][0])), __float_as_int(acc[m][f][1]));
            mx1 = max(max(mx1, __float_as_int(acc[m][f][2])), __float_as_int(acc[m][f][3]));
        }
    const unsigned wmax = wave_umax((unsigned)max(mx0, mx1));
    if (wid == 0) { STAMP(11); CSTAMP(31); }
    if (lane == 0) red[wid] = wmax;
    WAIT_L(0); BAR();
    asm volatile("" ::: "memory");
    const unsigned wgmax = wave_umax(lane < 10 ? red[lane] : 0u);
    if (wid == 0) {
        if (lane < NR) __hip_atomic_store((__attribute__((address_space(1))) u64*)gdst, ((u64)call << 32) | wgmax, __ATOMIC_RELAXED, __HIP_MEMORY_SCOPE_SYSTEM);
        if (lane == 0) __hip_atomic_store(seqp, call, __ATOMIC_RELAXED, __HIP_MEMORY_SCOPE_AGENT);
        STAMP(12);
    }
    asm volatile("" ::: "memory");
    const bool odd = g & 1;
#pragma unroll
    for (int m = 0; m < 4; ++m)
#pragma unroll
        for (int f = 0; f < 4; f += 2)
#pragma unroll
            for (int r = 0; r < 4; ++r) {
                const float e = __int_as_float(max(__float_as_int(acc[m][f][r]), 0)), o = __int_as_float(max(__float_as_int(acc[m][f + 1][r]), 0));
                acc[m][f][r] = odd ? o : e; acc[m][f + 1][r] = odd ? e : o;
            }
    const __amdgpu_buffer_rsrc_t outr = __builtin_amdgcn_make_buffer_rsrc(a.out, 0, MLOC * NDIM, SRD3);
    const unsigned ovo = (unsigned)(mtile * BM + wm * 64 + fr) * NDIM + (unsigned)(ntile * BN + wn * 64 + g * 16);
    const int self = rank * NWG + wg - 2 * tid;
    unsigned gm = wgmax, used = 0xfffffffeu;
    {
        const u64 t0 = __builtin_amdgcn_s_memrealtime();
        unsigned polls = 0;
        bool need0 = true, need1 = true;
        bool spin = false;
        for (unsigned round = 0;; ++round) {
            unsigned miss;
            for (;;) {
                if (need0 && v0.y == call) gm = max(gm, v0.x);
                if (need0 && v0.w == call) gm = max(gm, v0.z);
                if (need1 && v1.y == call) gm = max(gm, v1.x);
                if (need1 && v1.w == call) gm = max(gm, v1.z);
                if ((v0.y == call || self == 0) && (v0.w == call || self == 1)) need0 = false;
                if ((v1.y == call || self == 1024) && (v1.w == call || self == 1025)) need1 = false;
                miss = (unsigned)__popcll(__ballot(need0)) + (unsigned)__popcll(__ballot(need1));
                if (miss != 0u && timed_out(tmo, call, t0, polls)) { gm = 0xffffffffu; break; }
                if (!spin || miss == 0u) break;
                __builtin_amdgcn_s_sleep(POLL_SLEEP);
                asm volatile("" ::: "memory");
                if (need0) v0 = __builtin_amdgcn_raw_buffer_load_b128(wsr, WS_GRAN + tid * 16, 0, 17);
                if (need1) v1 = __builtin_amdgcn_raw_buffer_load_b128(wsr, WS_GRAN + 8192 + tid * 16, 0, 17);
            }
            const unsigned wv = wave_umax(gm);
            LAS unsigned* rb = red + 16 + (round & 1u) * 16u;
            if (lane == 0) { rb[wid] = wv; rb[8 + wid] = miss; }
            WAIT_L(0); BAR();
            asm volatile("" ::: "memory");
            const unsigned rv = lane < 16 ? rb[lane] : 0u;
            gm = wave_umax(lane < 8 ? rv : 0u);
            miss = 0;
#pragma unroll
            for (int i = 0; i < 8; ++i) miss += __builtin_amdgcn_readlane(rv, 8 + i);
            const bool fin = miss == 0u || gm == 0xffffffffu;
            if (fin && wid == 0) STAMP(13);
            if (!fin) {
                if (gm == used || miss > SPEC_THR) __builtin_amdgcn_s_sleep(POLL_SLEEP);
                asm volatile("" ::: "memory");
                if (need0) v0 = __builtin_amdgcn_raw_buffer_load_b128(wsr, WS_GRAN + tid * 16, 0, 17);
                if (need1) v1 = __builtin_amdgcn_raw_buffer_load_b128(wsr, WS_GRAN + 8192 + tid * 16, 0, 17);
                asm volatile("" ::: "memory");
            }
            if (gm != used && (fin || miss <= SPEC_THR)) {
                used = gm;
                const float amax = __uint_as_float(gm);
                const float inv = amax > 0.f ? 448.0f / amax : 0.f;
                const unsigned nanm = gm == 0xffffffffu ? 0x7f7f7f7fu : 0u;
                u32x4 o[4];
#pragma unroll
                for (int m = 0; m < 4; ++m)
#pragma unroll
                    for (int f = 0; f < 4; ++f) {
                        const f32x4 y = acc[m][f] * inv;
                        int w = __builtin_amdgcn_cvt_pk_fp8_f32(y[0], y[1], 0, false);
                        o[m][f] = (unsigned)__builtin_amdgcn_cvt_pk_fp8_f32(y[2], y[3], w, true);
                    }
                if (fin) {
#pragma unroll
                    for (int m = 0; m < 4; ++m) __builtin_amdgcn_raw_buffer_store_b128(o[m] | nanm, outr, ovo + m * 16 * NDIM, 0, FIN_AUX);
                } else {
#pragma unroll
                    for (int m = 0; m < 4; ++m) __builtin_amdgcn_raw_buffer_store_b128(o[m], outr, ovo + m * 16 * NDIM, 0, OUT_AUX);
                }
            }
            if (fin) break;
            spin = miss <= SPIN_THR;
        }
    }
}

}

extern "C" void kernel_launch(int rank, int nranks, ncclComm_t comm,
                              hipStream_t stream,
                              void* const* d_in, const int* in_sizes, int n_in,
                              void* d_out, int out_size,
                              const int* in_dtypes, int out_dtype,
                              void* const* d_out_all,
                              void* const* d_ws_all, size_t ws_bytes) {
    using namespace a2ag;
    if (nranks != NR || (unsigned)rank >= (unsigned)NR || n_in < 2 || ws_bytes < WS_NEED
        || in_sizes[0] != KDIM * KSL || in_sizes[1] != KDIM * NDIM || out_size != MLOC * NDIM) {
        fprintf(stderr, "kernel_launch: unexpected problem geometry\n");
        return;
    }
    HIP_CHECK(hipFuncSetAttribute((const void*)a2a_gemm_kernel, hipFuncAttributeMaxDynamicSharedMemorySize, LDS_BYTES));
    Args a;
    a.x = d_in[0]; a.w = d_in[1]; a.out = d_out; a.rank = rank;
    for (int r = 0; r < NR; ++r) a.ws[r] = (char*)d_ws_all[r];
    hipLaunchKernelGGL(a2a_gemm_kernel, dim3(NWG), dim3(NTHREADS), LDS_BYTES, stream, a);
    HIP_CHECK(hipGetLastError());
}
```

```cpp
#include <hip/hip_runtime.h>
#include <rccl/rccl.h>
#include <cstdio>
#include <cstdint>
#include <type_traits>

#define HIP_CHECK(expr) do { hipError_t _e = (expr); if (_e != hipSuccess) { \
    fprintf(stderr, "HIP error %s at %s:%d\n", hipGetErrorString(_e), __FILE__, __LINE__); \
    return; } } while (0)

#define PUSH_PACE 4
#define PUSH0 1
#define PUSH0_SPREAD 2
#define PUSH_LEAD 0
#define BULK_AT 1
#define OWN_TOUCH_AT 0
#define PUSH_SPIN 3
#define SPEC_THR 512
#define SPIN_THR 128
#define SEQ_SCALAR 1
#define PACE_NS 810
#define PACE_T0 2800
#define PACE_TO 35
#define PACE_NS2 700
#define PACE_TO2 PACE_TO
#define XP 1
#define WP 0
#define POLSTR_0 ""
#define POLSTR_1 "sc1"
#define POLSTR_2 "nt"
#define POLSTR_3 "sc0 sc1"
#define POLSTR_4 "sc0"
#define POLCAT2(a, b) a##b
#define POLCAT(a, b) POLCAT2(a, b)
#define XPOL POLCAT(POLSTR_, XP)
#define WPOL POLCAT(POLSTR_, WP)
#define KUNR 1
#define WSPLIT 6

#pragma clang diagnostic ignored "-Wint-to-pointer-cast"

namespace a2ag {

typedef __bf16 bf16x4 __attribute__((ext_vector_type(4)));
typedef __bf16 bf16x8 __attribute__((ext_vector_type(8)));
typedef float f32x4 __attribute__((ext_vector_type(4)));
typedef unsigned u32x4 __attribute__((ext_vector_type(4)));
typedef unsigned long long u64;
#define LAS __attribute__((address_space(3)))
#define DI __device__ __forceinline__
template <int V> using IC = std::integral_constant<int, V>;

constexpr int NR = 8;
constexpr int MLOC = 512, KDIM = 4096, NDIM = 16384, KSL = 512;
constexpr int BM = 256, BN = 128, BK = 64;
constexpr int NT = KDIM / BK;
constexpr int NWG = (MLOC / BM) * (NDIM / BN);
constexpr int NTHREADS = 512;
constexpr unsigned A_STAGE = BM * BK * 2, B_STAGE = BK * BN * 2;
constexpr int RA = 3, RB = 4;
constexpr unsigned LDS_B0 = RA * A_STAGE;
constexpr unsigned LDS_BYTES = LDS_B0 + RB * B_STAGE;
constexpr unsigned SLOT_BYTES = MLOC * KSL * 2;
constexpr unsigned X_BYTES = NR * SLOT_BYTES;
constexpr unsigned W_BYTES = (unsigned)KDIM * NDIM * 2;
constexpr unsigned W_TILE = BK * NDIM * 2;
constexpr int NCHUNK = 8, NPART = 32;
constexpr unsigned WS_XFLAG = X_BYTES;
constexpr unsigned WS_GRAN = WS_XFLAG + NCHUNK * 1024;
constexpr unsigned WS_SEQ = WS_GRAN + NR * NWG * 8;
constexpr unsigned WS_TMO = WS_SEQ + NWG * 64;
constexpr unsigned WS_DBG = WS_TMO + 4096;
constexpr unsigned WS_NEED = WS_DBG + NWG * 64 * 12;
constexpr u64 DEADLINE_TICKS = 40000000ull;
constexpr unsigned OOB = 0x40000000u;
constexpr unsigned SRD3 = 0x00020000u;
constexpr int SYS = 17;

struct Args {
    const void* x; const void* w; void* out;
    char* wsl;
    char* ws[NR];
    int rank;
};

DI void set_m0(unsigned v) { asm volatile("s_mov_b32 m0, %0\n\ts_nop 0" :: "s"(v) : "memory"); }
template <int OFF> DI void dma_x(__amdgpu_buffer_rsrc_t srd, unsigned voff, unsigned soff) {
    asm volatile("buffer_load_dwordx4 %0, %1, %2 offen offset:%3 " XPOL " lds" :: "v"(voff), "s"(srd), "s"(soff), "n"(OFF) : "memory");
}
template <int OFF> DI void dma_w(__amdgpu_buffer_rsrc_t srd, unsigned voff, unsigned soff) {
    asm volatile("buffer_load_dwordx4 %0, %1, %2 offen offset:%3 " WPOL " lds" :: "v"(voff), "s"(srd), "s"(soff), "n"(OFF) : "memory");
}
#define XCLOB "v176", "v177", "v178", "v179", "v180", "v181", "v182", "v183", "v184", "v185", "v186", "v187", "v188", "v189", "v190", "v191", "v192", "v193", "v194", "v195", "v196", "v197", "v198", "v199", "v200", "v201", "v202", "v203", "v204", "v205", "v206", "v207", "v208", "v209", "v210", "v211", "v212", "v213", "v214", "v215", "v216", "v217", "v218", "v219", "v220", "v221", "v222", "v223", "v224", "v225", "v226", "v227", "v228", "v229", "v230", "v231", "v232", "v233", "v234", "v235", "v236", "v237", "v238", "v239"
template <int C> DI void x_load(unsigned v0, unsigned v1, __amdgpu_buffer_rsrc_t srd, unsigned soff) {
    asm volatile("buffer_load_dwordx4 v[%c4:%c5], %0, %2, %3 offen offset:%c8\n\tbuffer_load_dwordx4 v[%c6:%c7], %1, %2, %3 offen offset:%c8"
                 :: "v"(v0), "v"(v1), "s"(srd), "s"(soff), "n"(176 + 8 * C), "n"(179 + 8 * C), "n"(180 + 8 * C), "n"(183 + 8 * C), "n"(C * 128) : "memory", XCLOB);
}
template <int C> DI void x_store(unsigned voff, __amdgpu_buffer_rsrc_t srd, unsigned soff) {
    asm volatile("buffer_store_dwordx4 v[%c3:%c4], %0, %1, %2 offen sc0 sc1\n\tbuffer_store_dwordx4 v[%c5:%c6], %0, %1, %2 offen offset:1024 sc0 sc1"
                 :: "v"(voff), "s"(srd), "s"(soff), "n"(176 + 8 * C), "n"(179 + 8 * C), "n"(180 + 8 * C), "n"(183 + 8 * C) : "memory");
}
template <int N> DI void wait_v() { asm volatile("s_waitcnt vmcnt(%0)" :: "n"(N) : "memory"); }
#define WAIT_L0() __builtin_amdgcn_s_waitcnt(0xC07F)
#define FENCE() asm volatile("" ::: "memory")
#define BAR() __builtin_amdgcn_s_barrier()
#define SCHED() __builtin_amdgcn_sched_barrier(0)
#define PRIO(n) __builtin_amdgcn_s_setprio(n)
#define MFMA(a, b, c) __builtin_amdgcn_mfma_f32_16x16x32_bf16(a, b, c, 0, 0, 0)

#define STAMP(i) do {} while (0)
#define CSTAMP(i) do {} while (0)
#define STAMP_DECL
#define STAMP_FLUSH(lo, hi) do {} while (0)

DI unsigned ktile_of(int t, int rank) {
    const int i = t - 8, c = (i * 37) >> 8, j = i - c * 7;
    return t < 8 ? (unsigned)(rank * 8 + t) : (unsigned)((((rank + 1 + j) & 7) << 3) + c);
}

DI bool timed_out(unsigned* tmo, unsigned call, u64 t0, unsigned& polls) {
    if ((++polls & 31u) != 0u) return false;
    if (__builtin_amdgcn_readfirstlane(__hip_atomic_load(tmo, __ATOMIC_RELAXED, __HIP_MEMORY_SCOPE_AGENT)) == call
        || __builtin_amdgcn_s_memrealtime() - t0 > DEADLINE_TICKS) {
        __hip_atomic_store(tmo, call, __ATOMIC_RELAXED, __HIP_MEMORY_SCOPE_AGENT);
        return true;
    }
    return false;
}
DI unsigned load_seq(const unsigned* p) {
    unsigned v;
    asm volatile("s_load_dword %0, %1, 0x0 glc\n\ts_waitcnt lgkmcnt(0)" : "=s"(v) : "s"(p) : "memory");
    return v;
}
DI unsigned vm_outstanding() {
    const unsigned s = __builtin_amdgcn_s_getreg(7 | (0 << 6) | (31 << 11));
    return (s & 15u) | ((s >> 18) & 48u);
}
DI unsigned row_umax(unsigned v) {
    v = max(v, (unsigned)__builtin_amdgcn_update_dpp(0, (int)v, 0x111, 0xf, 0xf, false));
    v = max(v, (unsigned)__builtin_amdgcn_update_dpp(0, (int)v, 0x112, 0xf, 0xf, false));
    v = max(v, (unsigned)__builtin_amdgcn_update_dpp(0, (int)v, 0x114, 0xf, 0xf, false));
    v = max(v, (unsigned)__builtin_amdgcn_update_dpp(0, (int)v, 0x118, 0xf, 0xf, false));
    return v;
}
DI unsigned wave_umax(unsigned v) {
    v = row_umax(v);
    const unsigned a = __builtin_amdgcn_readlane(v, 15), b = __builtin_amdgcn_readlane(v, 31);
    const unsigned c = __builtin_amdgcn_readlane(v, 47), d = __builtin_amdgcn_readlane(v, 63);
    return max(max(a, b), max(c, d));
}

__global__ __launch_bounds__(NTHREADS) void a2a_gemm_kernel(Args a) {
    extern __shared__ __attribute__((aligned(16))) char smem[];
    LAS char* lds = (LAS char*)smem;
    const int tid = threadIdx.x, lane = tid & 63;
    const int wid = __builtin_amdgcn_readfirstlane(tid >> 6);
    const int wg = blockIdx.x, rank = a.rank;
    const int xcd = wg & 7, idx = wg >> 3;
    const int ntile = xcd * 16 + (idx >> 1), mtile = idx & 1;
    const unsigned ldsbase = (unsigned)(size_t)lds;
    LAS unsigned* red = (LAS unsigned*)(lds + LDS_B0 + 3 * B_STAGE);
    char* wsl = a.wsl;
    unsigned* seqp = (unsigned*)(wsl + WS_SEQ + wg * 64);
    unsigned* tmo = (unsigned*)(wsl + WS_TMO);
    STAMP_DECL;

    if (wid == 4 || wid == 5) {
        PRIO(3);
        const int half = wid - 4;
        const __amdgpu_buffer_rsrc_t ss = __builtin_amdgcn_make_buffer_rsrc((void*)wsl, 0, WS_TMO, SRD3);
        const __amdgpu_buffer_rsrc_t wr = __builtin_amdgcn_make_buffer_rsrc((void*)a.w, 0, W_BYTES, SRD3);
        unsigned vs[2], vw[2];
#pragma unroll
        for (int e = 0; e < 2; ++e) {
            vs[e] = (unsigned)(lane >> 3) * 128u + (unsigned)(((lane & 7) ^ (4 * e + (lane >> 4))) << 4);
            vw[e] = (unsigned)(lane >> 4) * 32768u + (unsigned)(((lane & 15) ^ ((e << 3) | (lane >> 4))) << 4);
        }
        const unsigned row0 = (unsigned)(mtile * BM + half * 128);
        const unsigned lA = ldsbase + (unsigned)half * 16384u, lB = ldsbase + LDS_B0 + (unsigned)half * 8192u;
        const unsigned soB = (unsigned)ntile * (BN * 2u) + (unsigned)half * (32u * 32768u);
        const unsigned soOwn = (unsigned)rank * SLOT_BYTES + row0 * 1024u;
        const char* xb = (const char*)a.x;
        unsigned dx = 7168u, v0 = vs[0] + (unsigned)(lane >> 3) * 896u, v1 = vs[1] + (unsigned)(lane >> 3) * 896u;
        unsigned la = lA;
        auto next_la = [&](unsigned l) -> unsigned { return l == lA + (RA - 1) * A_STAGE ? lA : l + A_STAGE; };
        auto issueA = [&](unsigned so) {
            const __amdgpu_buffer_rsrc_t srd = __builtin_amdgcn_make_buffer_rsrc((void*)xb, 0, X_BYTES, SRD3);
#pragma unroll
            for (int p = 0; p < 16; p += 4) {
                const unsigned s4 = so + p * (dx + 1024u);
                set_m0(la + p * 1024u);
                dma_x<0>(srd, v0, s4); dma_x<1024>(srd, v1, s4 + dx); dma_x<2048>(srd, v0, s4 + 2u * dx); dma_x<3072>(srd, v1, s4 + 3u * dx);
            }
            la = next_la(la);
        };
        auto issueAs = [&](unsigned so, unsigned l) {
#pragma unroll
            for (int p = 0; p < 16; p += 4) {
                set_m0(l + p * 1024u);
                dma_x<0>(ss, vs[0], so + p * 1024u); dma_x<1024>(ss, vs[1], so + p * 1024u); dma_x<2048>(ss, vs[0], so + p * 1024u); dma_x<3072>(ss, vs[1], so + p * 1024u);
            }
        };
        auto wstage = [&](int t) -> unsigned { return lB + ((unsigned)t & (RB - 1)) * B_STAGE; };
        auto issueW = [&](unsigned so, unsigned lb, auto lo, auto hi) {
            constexpr int LO = decltype(lo)::value, HI = decltype(hi)::value;
            if constexpr (LO == 0) {
                set_m0(lb);
                dma_w<0>(wr, vw[0], so); dma_w<1024>(wr, vw[0], so + 130048u); dma_w<2048>(wr, vw[1], so + 260096u); dma_w<3072>(wr, vw[1], so + 390144u);
            }
            if constexpr (HI > 4 && LO < 8) set_m0(lb + 4096u);
            if constexpr (LO <= 4 && HI >= 6) { dma_w<0>(wr, vw[0], so + 524288u); dma_w<1024>(wr, vw[0], so + 654336u); }
            if constexpr (LO <= 6 && HI == 8) { dma_w<2048>(wr, vw[1], so + 784384u); dma_w<3072>(wr, vw[1], so + 914432u); }
        };
        using I0 = IC<0>; using IS = IC<WSPLIT>; using I8 = IC<8>;
        constexpr int N_HEAVY = 16 + 8 + WSPLIT, N_LIGHT = 16 + 8;
        const unsigned fvo = WS_XFLAG - 1024u + (unsigned)lane * 16u;
#define POLL_ISSUE(n) asm volatile("buffer_load_dwordx4 v[240:243], %0, %1, %2 offen sc0 sc1" :: "v"(fvo), "s"(ss), "s"((n) * 1024u) : "v240", "v241", "v242", "v243", "memory")
        const unsigned soW0 = (unsigned)rank * (8u * W_TILE) + soB;
        issueA(soOwn); issueW(soW0, wstage(0), I0{}, I8{});
        issueW(soW0 + W_TILE, wstage(1), I0{}, I8{});
        unsigned tabA, tabW;
        {
            const int t = lane + 3;
            const unsigned kt = ktile_of(t, rank);
            const unsigned first = (int)(kt >> 3) == ((rank + 1) & 7) ? 16u : 0u;
            tabA = t < 8 ? soOwn + (unsigned)t * 128u : (t < NT ? kt * 65536u + row0 * 128u + first + (kt & 7u) + 1u : OOB);
            const unsigned n0 = t < 8 ? 0u : (kt & 7u) + 1u, n2 = t + 2 < 8 ? 0u : (ktile_of(t + 2, rank) & 7u) + 1u;
            tabW = t < NT ? kt * W_TILE + soB + ((t + 2 < NT && n2 > n0) ? n2 : 0u) : OOB;
        }
        const unsigned call = load_seq(seqp) + 1u;
        wait_v<8>(); BAR();
        issueA(soOwn + 128u);
        issueW(soW0 + 2u * W_TILE, wstage(2), I0{}, I8{}); issueA(soOwn + 256u);
        if (mtile == 0) issueW(soW0 + 3u * W_TILE, wstage(3), I0{}, IS{});
        const bool own = (lane >> 3) == rank;
        bool good = true;
        unsigned prog = 0;
        auto flags_ok = [&]() -> bool {
            u32x4 f;
            asm volatile("v_mov_b32 %0, v240\n\tv_mov_b32 %1, v241\n\tv_mov_b32 %2, v242\n\tv_mov_b32 %3, v243" : "=v"(f.x), "=v"(f.y), "=v"(f.z), "=v"(f.w) :: "memory");
            return __all(own || (int)((f.x - call) | (f.y - call) | (f.z - call) | (f.w - call)) >= 0);
        };
        auto await = [&](unsigned need) {
            if (__builtin_expect(prog < need, 0)) {
                if (half == 0) STAMP(40 + need - 1);
                const u64 t0 = __builtin_amdgcn_s_memrealtime();
                unsigned polls = 0;
                for (;;) {
                    POLL_ISSUE(need); wait_v<0>();
                    if (flags_ok()) break;
                    if (timed_out(tmo, call, t0, polls)) { good = false; break; }
                    __builtin_amdgcn_s_sleep(4);
                }
                prog = need;
                if (half == 0) STAMP(48 + need - 1);
            }
        };
        unsigned pc = 0;
        int i = 0;
#pragma unroll 1
        for (; i < 5 + mtile; ++i) {
            {
                const bool mine = i < 5;
                xb = mine ? (const char*)a.x : (const char*)wsl;
                dx = mine ? 7168u : 0u;
                const unsigned dd = mine ? 896u : 0u;
                v0 = vs[0] + (unsigned)(lane >> 3) * dd; v1 = vs[1] + (unsigned)(lane >> 3) * dd;
            }
            const unsigned ta = __builtin_amdgcn_readlane(tabA, i), w3 = __builtin_amdgcn_readlane(tabW, i), w4 = __builtin_amdgcn_readlane(tabW, i + 1);
            if ((i ^ mtile) & 1) {
                wait_v<N_LIGHT>(); BAR();
                pc = w3 & 15u;
                if (pc != 0u) POLL_ISSUE(pc);
                issueA(ta & ~127u);
                issueW(w3 & ~255u, wstage(i + 3), I0{}, I8{});
                issueW(w4 & ~255u, wstage(i + 4), I0{}, IS{});
            } else {
                wait_v<N_HEAVY>(); BAR();
                issueW(w3 & ~255u, wstage(i + 3), IS{}, I8{});
                if (pc != 0u) { if (flags_ok()) prog = pc; }
                await(ta & 15u);
                issueA(ta & ~127u);
            }
        }
        unsigned aH, aL, wH, wE, laH, laL, lbH, lbE, nH, nL, rareH, rareL;
        auto plan = [&](int h) {
            const unsigned t0 = __builtin_amdgcn_readlane(tabA, h & 63), t1 = __builtin_amdgcn_readlane(tabA, (h + 1) & 63);
            const unsigned u0 = __builtin_amdgcn_readlane(tabW, h & 63), u1 = __builtin_amdgcn_readlane(tabW, (h + 1) & 63);
            aH = t0 & ~127u; nH = t0 & 15u; aL = t1 & ~127u; nL = t1 & 15u; wH = u0 & ~255u; pc = u0 & 15u; wE = u1 & ~255u;
            laH = la; laL = next_la(la); la = next_la(laL);
            lbH = wstage(h + 3); lbE = wstage(h + 4);
            rareH = pc | (t0 & 16u); rareL = pc | (t1 & 16u);
            asm volatile("" : "+s"(aH), "+s"(aL), "+s"(wH), "+s"(wE), "+s"(laH), "+s"(laL), "+s"(lbH), "+s"(lbE), "+s"(rareH), "+s"(rareL));
        };
        plan(i);
#pragma unroll 1
        for (int n = 0; n < 29; ++n) {
            wait_v<N_LIGHT>(); BAR();
            if (__builtin_expect(rareH != 0u, 0)) { await(nH); if (pc != 0u) POLL_ISSUE(pc); }
            issueAs(aH, laH);
            issueW(wH, lbH, I0{}, I8{});
            issueW(wE, lbE, I0{}, IS{});
            wait_v<N_HEAVY>(); BAR();
            issueW(wE, lbE, IS{}, I8{});
            if (__builtin_expect(rareL != 0u, 0)) { if (pc != 0u && flags_ok()) prog = pc; await(nL); }
            issueAs(aL, laL);
            i += 2;
            plan(i);
        }
        if (mtile == 0) BAR();
        if (lane == 0) red[4 + half] = good ? 0u : 0xffffffffu;
        FENCE(); WAIT_L0(); BAR();
        if (half == 0) { STAMP_FLUSH(16, 24); STAMP_FLUSH(40, 64); }
        return;
    }
    if (wid == 6) {
        PRIO(3);
        STAMP(32);
        const __amdgpu_buffer_rsrc_t xs = __builtin_amdgcn_make_buffer_rsrc((void*)a.x, 0, X_BYTES, SRD3);
        const unsigned po = (unsigned)(idx * 16 + (lane & 15)) * 1024u + (unsigned)(3 + (lane >> 4)) * 128u;
#define OWN_TOUCH() asm volatile("buffer_load_dword v240, %0, %1, %2 offen\n\tbuffer_load_dword v240, %3, %1, %2 offen" \
                         :: "v"(po), "s"(xs), "s"((unsigned)rank * SLOT_BYTES), "v"(po | 896u) : "v240", "memory")
        int nb = 0;
        if (wg < 7 * NPART) {
            const int j = wg % 7 + 1, part = wg / 7, dst = (rank + j) & 7;
            char* wsd = a.ws[0];
#pragma unroll
            for (int r = 1; r < NR; ++r) wsd = (dst == r) ? a.ws[r] : wsd;
            const __amdgpu_buffer_rsrc_t ds = __builtin_amdgcn_make_buffer_rsrc((void*)wsd, 0, WS_GRAN, SRD3);
            const unsigned vsrc = (unsigned)(part * 16 + (lane >> 3)) * 1024u + (unsigned)(lane & 7) * 16u;
            const unsigned vdst = (unsigned)part * 2048u + (unsigned)lane * 16u;
            const unsigned fdst = WS_XFLAG + (unsigned)rank * 128u + (unsigned)part * 4u;
            const unsigned vsrc2 = vsrc + 8192u, ssrc = (unsigned)dst * SLOT_BYTES, sdst = (unsigned)rank * SLOT_BYTES;
            auto xload = [&](auto cc) { x_load<decltype(cc)::value>(vsrc, vsrc2, xs, ssrc); };
            auto xstore = [&](auto cc) { x_store<decltype(cc)::value>(vdst, ds, sdst + (unsigned)decltype(cc)::value * 65536u); };
            xload(IC<0>{}); xload(IC<1>{});
            if (OWN_TOUCH_AT == 0) OWN_TOUCH();
            const unsigned call = load_seq(seqp) + 1u;
            auto quiet = [&](int c, int from, int spins) {
                int spin = 0;
#pragma unroll 1
                for (;;) {
                    const bool last = nb >= 7 + 7 * c;
                    if (nb >= from && (c == 0 || last || ((nb ^ mtile) & 1) == 0) && vm_outstanding() == 0u) break;
                    if (!last && (nb < from || ++spin > spins)) { BAR(); ++nb; spin = 0; }
                    else __builtin_amdgcn_s_sleep(1);
                }
            };
#pragma unroll 1
            for (int c = 0; c < NCHUNK; ++c) {
                const int slot = c == 0 ? PUSH0 + part % PUSH0_SPREAD : (c == 1 ? 2 : PUSH_PACE * c - PUSH_LEAD) + (part & 2) + mtile;
                if (c <= 2) quiet(c, slot, 0);
                else {
#pragma unroll 1
                    while (nb < slot) { BAR(); ++nb; }
                }
                switch (c) {
                    case 0: xstore(IC<0>{}); if (BULK_AT == 0) { xload(IC<2>{}); xload(IC<3>{}); xload(IC<4>{}); xload(IC<5>{}); xload(IC<6>{}); xload(IC<7>{}); } break;
                    case 1: xstore(IC<1>{}); if (BULK_AT == 1) { xload(IC<2>{}); xload(IC<3>{}); xload(IC<4>{}); xload(IC<5>{}); xload(IC<6>{}); xload(IC<7>{}); } break;
                    case 2: xstore(IC<2>{}); break;
                    case 3: xstore(IC<3>{}); break;
                    case 4: xstore(IC<4>{}); break;
                    case 5: xstore(IC<5>{}); break;
                    case 6: xstore(IC<6>{}); break;
                    default: xstore(IC<7>{}); break;
                }
                if (c == 0 && OWN_TOUCH_AT == 1) OWN_TOUCH();
                quiet(c, 0, c == 0 ? PUSH_SPIN : 0);
                FENCE();
                if (lane == 0) __builtin_amdgcn_raw_buffer_store_b32(call, ds, fdst, c * 1024, SYS);
                FENCE();
                if (c == 0) STAMP(34);
            }
        } else OWN_TOUCH();
        STAMP(35);
        STAMP_FLUSH(32, 40);
        while (vm_outstanding() != 0u) BAR();
        return;
    }

    if (wid == 7) {
        const u64 t0 = __builtin_amdgcn_s_memrealtime();
        BAR();
        int next = PACE_T0;
#pragma unroll 1
        for (int i = 0; i < PACE_TO2; ++i) {
            next += i < PACE_TO ? PACE_NS : PACE_NS2;
            while ((int)(unsigned)(__builtin_amdgcn_s_memrealtime() - t0) * 10 < next) __builtin_amdgcn_s_sleep(1);
            BAR();
        }
        return;
    }

    if (wid == 0) STAMP(0);
    const int wm = wid & 1, wn = wid >> 1;
    const int fr = lane & 15, g = lane >> 4, q = fr >> 2, p = lane & 3;
    unsigned aoff[2], boff[4];
#pragma unroll
    for (int kk = 0; kk < 2; ++kk) aoff[kk] = ldsbase + (unsigned)((wm * 128 + fr) * 128 + (((kk * 4 + g) ^ (fr >> 1)) << 4));
#pragma unroll
    for (int f = 0; f < 4; ++f)
        boff[f] = ldsbase + LDS_B0 + (unsigned)((8 * g + q) * 256 + (((wn * 8 + 2 * p + (f >> 1)) ^ (q | ((g & 1) << 3))) << 4) + ((((f & 1) ^ (p & 1))) << 3));
    asm volatile("" : "+v"(boff[0]), "+v"(boff[1]), "+v"(boff[2]), "+v"(boff[3]), "+v"(aoff[0]), "+v"(aoff[1]));
    const unsigned call = load_seq(seqp) + 1u;
    u64 gdst;
    {
        char* pw = a.ws[0];
#pragma unroll
        for (int r = 1; r < NR; ++r) pw = (lane == r) ? a.ws[r] : pw;
        gdst = (u64)((u64*)(pw + WS_GRAN) + rank * NWG + wg);
        asm volatile("" : "+v"(gdst));
    }
    f32x4 acc[8][4];
#pragma unroll
    for (int m = 0; m < 8; ++m)
#pragma unroll
        for (int f = 0; f < 4; ++f) acc[m][f] = (f32x4){0.f, 0.f, 0.f, 0.f};
    bf16x8 x0[8], w0[4], x1[8], w1[4];
    bf16x4 tl[4];
    unsigned bcur[4], acur = aoff[0];
#pragma unroll
    for (int f = 0; f < 4; ++f) bcur[f] = boff[f];
#define RD_WL(f, kk) do { tl[f] = __builtin_amdgcn_ds_read_tr16_b64_v4bf16((LAS bf16x4*)(bcur[f] + (kk) * 8192)); } while (0)
#define RD_WH(dst, f, kk) do { const bf16x4 hi_ = __builtin_amdgcn_ds_read_tr16_b64_v4bf16((LAS bf16x4*)(bcur[f] + (kk) * 8192 + 1024)); \
        dst = __builtin_shufflevector(tl[f], hi_, 0, 1, 2, 3, 4, 5, 6, 7); } while (0)
#define RD_XA(dst, m) do { dst = *(const LAS bf16x8*)(acur + (m) * 2048); } while (0)
    FENCE(); BAR(); FENCE();
    if (wid == 0) { STAMP(2); CSTAMP(30); }
#pragma unroll
    for (int i = 0; i < 4; ++i) { RD_WL(i, 0); RD_WH(w0[i], i, 0); }
#pragma unroll
    for (int i = 0; i < 8; ++i) RD_XA(x0[i], i);
    acur = aoff[1];
    auto step = [&](bf16x8* cw, bf16x8* cx, bf16x8* nw, bf16x8* nx, auto kkc, auto side) {
        constexpr int kk = decltype(kkc)::value;
#pragma unroll
        for (int s = 0; s < 32; ++s) {
            const int m = s >> 2, f = (m & 1) ? 3 - (s & 3) : (s & 3);
            acc[m][f] = MFMA(cw[f], cx[m], acc[m][f]);
            SCHED();
            if (s < 24 && s % 3 != 1) {
                const int r = (s * 2) / 3;
                if (r < 8) { if (r & 1) RD_WH(nw[r >> 1], r >> 1, kk); else RD_WL(r >> 1, kk); }
                else RD_XA(nx[r - 8], r - 8);
            }
            if (s >= 24 && s < 31) side(s - 24);
            SCHED();
        }
    };
    unsigned oa = A_STAGE, ob = B_STAGE;
    auto stepA = [&]() {
        step(w0, x0, w1, x1, IC<1>{}, [&](int h) {
            if (h < 4) bcur[h] = boff[h] + ob;
            else if (h == 4) acur = aoff[0] + oa;
            else if (h == 5) { ob = (ob + B_STAGE) & (RB * B_STAGE - 1); asm volatile("" : "+s"(ob)); } });
        WAIT_L0(); SCHED();
    };
    auto stepB = [&]() {
        BAR(); SCHED();
        step(w1, x1, w0, x0, IC<0>{}, [&](int h) {
            if (h == 0) acur = aoff[1] + oa;
            else if (h == 1) { oa += A_STAGE; asm volatile("" : "+s"(oa)); }
            else if (h == 2) { oa = oa == RA * A_STAGE ? 0u : oa; asm volatile("" : "+s"(oa)); }
            else if (h == 6) WAIT_L0(); });
    };
    WAIT_L0();
    stepA();
#pragma unroll 1
    for (int t = 0; t < NT - 1; t += KUNR) {
#pragma unroll
        for (int u = 0; u < KUNR; ++u) { stepB(); stepA(); }
    }

    const __amdgpu_buffer_rsrc_t wsr = __builtin_amdgcn_make_buffer_rsrc((void*)wsl, 0, WS_SEQ, SRD3);
    int mx0 = 0, mx1 = 0;
    u32x4 gv[4];
    SCHED(); BAR(); SCHED();
#pragma unroll
    for (int s = 0; s < 36; ++s) {
        if (s < 32) { const int m = s >> 2, f = (m & 1) ? 3 - (s & 3) : (s & 3); acc[m][f] = MFMA(w1[f], x1[m], acc[m][f]); }
        SCHED();
        if (s >= 4) {
            const int m = (s - 4) >> 2, f = (m & 1) ? 3 - (s & 3) : (s & 3);
            mx0 = max(max(mx0, __float_as_int(acc[m][f][0])), __float_as_int(acc[m][f][1]));
            mx1 = max(max(mx1, __float_as_int(acc[m][f][2])), __float_as_int(acc[m][f][3]));
        }
        if (s >= 16 && s < 20) gv[s - 16] = __builtin_amdgcn_raw_buffer_load_b128(wsr, WS_GRAN + (s - 16) * 4096 + tid * 16, 0, SYS);
        SCHED();
    }
    const unsigned wmax = row_umax((unsigned)max(mx0, mx1));
    if (wid == 0) { STAMP(11); CSTAMP(31); }
    if (fr == 15) red[8 + wid * 4 + g] = wmax;
    FENCE(); WAIT_L0(); BAR(); FENCE();
    const unsigned wgmax = wave_umax((lane >= 4 && lane < 24 && (lane & ~1) != 6) ? red[lane] : 0u);
    if (wid == 0) {
        if (lane < NR) __hip_atomic_store((__attribute__((address_space(1))) u64*)gdst, ((u64)call << 32) | wgmax, __ATOMIC_RELAXED, __HIP_MEMORY_SCOPE_SYSTEM);
        if (lane == 0) __hip_atomic_store(seqp, call, __ATOMIC_RELAXED, __HIP_MEMORY_SCOPE_AGENT);
        STAMP(12);
    }
    FENCE();
    const bool odd = g & 1;
#pragma unroll
    for (int m = 0; m < 8; ++m)
#pragma unroll
        for (int f = 0; f < 4; f += 2)
#pragma unroll
            for (int r = 0; r < 4; ++r) {
                const float e = __int_as_float(max(__float_as_int(acc[m][f][r]), 0)), o = __int_as_float(max(__float_as_int(acc[m][f + 1][r]), 0));
                acc[m][f][r] = odd ? o : e; acc[m][f + 1][r] = odd ? e : o;
            }
    const __amdgpu_buffer_rsrc_t outr = __builtin_amdgcn_make_buffer_rsrc(a.out, 0, MLOC * NDIM, SRD3);
    const unsigned ovo = (unsigned)(mtile * BM + wm * 128 + fr) * NDIM + (unsigned)(ntile * BN + wn * 64 + g * 16);
    const int self = rank * NWG + wg - 2 * tid;
    unsigned gm = wgmax, used = 0xfffffffeu;
    {
        const u64 t0 = __builtin_amdgcn_s_memrealtime();
        unsigned polls = 0;
        bool need[4] = {true, true, true, true};
        bool spin = false;
        auto look = [&]() {
            FENCE();
#pragma unroll
            for (int k = 0; k < 4; ++k) if (need[k]) gv[k] = __builtin_amdgcn_raw_buffer_load_b128(wsr, WS_GRAN + k * 4096 + tid * 16, 0, SYS);
            FENCE();
        };
        for (unsigned round = 0;; ++round) {
            unsigned miss;
            for (;;) {
                miss = 0;
#pragma unroll
                for (int k = 0; k < 4; ++k) {
                    if (gv[k].y == call) gm = max(gm, gv[k].x);
                    if (gv[k].w == call) gm = max(gm, gv[k].z);
                    need[k] = !((gv[k].y == call || self == 512 * k) && (gv[k].w == call || self == 512 * k + 1));
                    miss += (unsigned)__popcll(__ballot(need[k]));
                }
                if (miss != 0u && timed_out(tmo, call, t0, polls)) { gm = 0xffffffffu; break; }
                if (!spin || miss == 0u) break;
                __builtin_amdgcn_s_sleep(1);
                look();
            }
            const unsigned wv = wave_umax(gm);
            LAS unsigned* rb = red + 32 + (round & 1u) * 8u;
            if (lane == 0) { rb[wid] = wv; rb[4 + wid] = miss; }
            FENCE(); WAIT_L0(); BAR(); FENCE();
            const unsigned rv = rb[lane & 7];
            gm = max(max((unsigned)__builtin_amdgcn_readlane(rv, 0), (unsigned)__builtin_amdgcn_readlane(rv, 1)), max((unsigned)__builtin_amdgcn_readlane(rv, 2), (unsigned)__builtin_amdgcn_readlane(rv, 3)));
            miss = __builtin_amdgcn_readlane(rv, 4) + __builtin_amdgcn_readlane(rv, 5) + __builtin_amdgcn_readlane(rv, 6) + __builtin_amdgcn_readlane(rv, 7);
            const bool fin = miss == 0u || gm == 0xffffffffu;
            if (fin && wid == 0) STAMP(13);
            if (!fin) look();
            if (gm != used && (fin || miss <= SPEC_THR)) {
                used = gm;
                const float amax = __uint_as_float(gm);
                const float inv = amax > 0.f ? 448.0f / amax : 0.f;
                const unsigned nanm = gm == 0xffffffffu ? 0x7f7f7f7fu : 0u;
#pragma unroll
                for (int m = 0; m < 8; ++m) {
                    u32x4 o;
#pragma unroll
                    for (int f = 0; f < 4; ++f) {
                        const f32x4 y = acc[m][f] * inv;
                        const int w = __builtin_amdgcn_cvt_pk_fp8_f32(y[0], y[1], 0, false);
                        o[f] = (unsigned)__builtin_amdgcn_cvt_pk_fp8_f32(y[2], y[3], w, true) | nanm;
                    }
                    __builtin_amdgcn_raw_buffer_store_b128(o, outr, ovo + m * 16 * NDIM, 0, SYS);
                }
            }
            if (fin) break;
            spin = miss <= SPIN_THR;
        }
    }
}

}

extern "C" void kernel_launch(int rank, int nranks, ncclComm_t comm,
                              hipStream_t stream,
                              void* const* d_in, const int* in_sizes, int n_in,
                              void* d_out, int out_size,
                              const int* in_dtypes, int out_dtype,
                              void* const* d_out_all,
                              void* const* d_ws_all, size_t ws_bytes) {
    using namespace a2ag;
    if (nranks != NR || (unsigned)rank >= (unsigned)NR || n_in < 2 || ws_bytes < WS_NEED
        || in_sizes[0] != KDIM * KSL || in_sizes[1] != KDIM * NDIM || out_size != MLOC * NDIM) {
        fprintf(stderr, "kernel_launch: unexpected problem geometry\n");
        return;
    }
    HIP_CHECK(hipFuncSetAttribute((const void*)a2a_gemm_kernel, hipFuncAttributeMaxDynamicSharedMemorySize, LDS_BYTES));
    Args a;
    a.x = d_in[0]; a.w = d_in[1]; a.out = d_out; a.rank = rank;
    a.wsl = (char*)d_ws_all[rank];
    for (int r = 0; r < NR; ++r) a.ws[r] = (char*)d_ws_all[r];
    hipLaunchKernelGGL(a2a_gemm_kernel, dim3(NWG), dim3(NTHREADS), LDS_BYTES, stream, a);
    HIP_CHECK(hipGetLastError());
}
```

```cpp
#include <hip/hip_runtime.h>
#include <rccl/rccl.h>
#include <cstdio>
#include <cstdint>
#include <type_traits>

#define HIP_CHECK(expr) do { hipError_t _e = (expr); if (_e != hipSuccess) { \
    fprintf(stderr, "HIP error %s at %s:%d\n", hipGetErrorString(_e), __FILE__, __LINE__); \
    return; } } while (0)

#define PUSH_PACE 4
#define PUSH0 1
#define PUSH0_SPREAD 2
#define PUSH_LEAD 0
#define BULK_AT 1
#define OWN_TOUCH_AT 0
#define PUSH_SPIN 3
#define SPEC_THR 4
#define SPIN_THR 128
#define SEQ_SCALAR 1
#define PACE_NS 810
#define PACE_T0 2800
#define PACE_TO 35
#define PACE_MIN 0
#define PACE_TEND 31000
#define PACE_NS2 700
#define PACE_TO2 PACE_TO
#define XP 1
#define WP 0
#define POLSTR_0 ""
#define POLSTR_1 "sc1"
#define POLSTR_2 "nt"
#define POLSTR_3 "sc0 sc1"
#define POLSTR_4 "sc0"
#define POLCAT2(a, b) a##b
#define POLCAT(a, b) POLCAT2(a, b)
#define XPOL POLCAT(POLSTR_, XP)
#define WPOL POLCAT(POLSTR_, WP)
#define OUT_AUX 2
#define QFAST 1
#define GP_AUX 17
#define FIN_SLEEP 1
#define COLL_QUIET 0
#define KUNR 1
#define WSPLIT 6

#pragma clang diagnostic ignored "-Wint-to-pointer-cast"

namespace a2ag {

typedef __bf16 bf16x4 __attribute__((ext_vector_type(4)));
typedef __bf16 bf16x8 __attribute__((ext_vector_type(8)));
typedef float f32x4 __attribute__((ext_vector_type(4)));
typedef float f32x2 __attribute__((ext_vector_type(2)));
typedef short s16x2 __attribute__((ext_vector_type(2)));
typedef unsigned u32x4 __attribute__((ext_vector_type(4)));
typedef unsigned u32x16 __attribute__((ext_vector_type(16)));
typedef unsigned long long u64;
#define LAS __attribute__((address_space(3)))
#define DI __device__ __forceinline__
template <int V> using IC = std::integral_constant<int, V>;

constexpr int NR = 8;
constexpr int MLOC = 512, KDIM = 4096, NDIM = 16384, KSL = 512;
constexpr int BM = 256, BN = 128, BK = 64;
constexpr int NT = KDIM / BK;
constexpr int NWG = (MLOC / BM) * (NDIM / BN);
constexpr int NTHREADS = 512;
constexpr unsigned A_STAGE = BM * BK * 2, B_STAGE = BK * BN * 2;
constexpr int RA = 3, RB = 4;
constexpr unsigned LDS_B0 = RA * A_STAGE;
constexpr unsigned LDS_BYTES = LDS_B0 + RB * B_STAGE;
constexpr unsigned SLOT_BYTES = MLOC * KSL * 2;
constexpr unsigned X_BYTES = NR * SLOT_BYTES;
constexpr unsigned W_BYTES = (unsigned)KDIM * NDIM * 2;
constexpr unsigned W_TILE = BK * NDIM * 2;
constexpr int NCHUNK = 8, NPART = 24;
constexpr unsigned WS_XFLAG = X_BYTES;
constexpr unsigned WS_GRAN = WS_XFLAG + NCHUNK * 1024;
constexpr unsigned WS_FIN = WS_GRAN + NWG * 8;
constexpr unsigned WS_SEQ = WS_FIN + 8 * 128;
constexpr unsigned WS_TMO = WS_SEQ + NWG * 64;
constexpr unsigned WS_DBG = WS_TMO + 4096;
constexpr unsigned WS_NEED = WS_DBG + NWG * 64 * 12;
constexpr u64 DEADLINE_TICKS = 40000000ull;
constexpr unsigned OOB = 0x40000000u;
constexpr unsigned SRD3 = 0x00020000u;
constexpr int COLLECTOR = NWG - 1;
constexpr int SYS = 17;

struct Args {
    const void* x; const void* w; void* out;
    char* wsl;
    char* ws[NR];
    int rank;
};

DI void set_m0(unsigned v) { asm volatile("s_mov_b32 m0, %0\n\ts_nop 0" :: "s"(v) : "memory"); }
template <int OFF> DI void dma_x(__amdgpu_buffer_rsrc_t srd, unsigned voff, unsigned soff) {
    asm volatile("buffer_load_dwordx4 %0, %1, %2 offen offset:%3 " XPOL " lds" :: "v"(voff), "s"(srd), "s"(soff), "n"(OFF) : "memory");
}
template <int OFF> DI void dma_w(__amdgpu_buffer_rsrc_t srd, unsigned voff, unsigned soff) {
    asm volatile("buffer_load_dwordx4 %0, %1, %2 offen offset:%3 " WPOL " lds" :: "v"(voff), "s"(srd), "s"(soff), "n"(OFF) : "memory");
}
#define XCLOB "v144", "v145", "v146", "v147", "v148", "v149", "v150", "v151", "v152", "v153", "v154", "v155", "v156", "v157", "v158", "v159", "v160", "v161", "v162", "v163", "v164", "v165", "v166", "v167", "v168", "v169", "v170", "v171", "v172", "v173", "v174", "v175", "v176", "v177", "v178", "v179", "v180", "v181", "v182", "v183", "v184", "v185", "v186", "v187", "v188", "v189", "v190", "v191", "v192", "v193", "v194", "v195", "v196", "v197", "v198", "v199", "v200", "v201", "v202", "v203", "v204", "v205", "v206", "v207", "v208", "v209", "v210", "v211", "v212", "v213", "v214", "v215", "v216", "v217", "v218", "v219", "v220", "v221", "v222", "v223", "v224", "v225", "v226", "v227", "v228", "v229", "v230", "v231", "v232", "v233", "v234", "v235", "v236", "v237", "v238", "v239"
template <int C> DI void x_load(unsigned v0, unsigned v1, unsigned v2, __amdgpu_buffer_rsrc_t srd, unsigned soff) {
    asm volatile("buffer_load_dwordx4 v[%c5:%c6], %0, %3, %4 offen offset:%c11\n\tbuffer_load_dwordx4 v[%c7:%c8], %1, %3, %4 offen offset:%c11\n\t"
                 "buffer_load_dwordx4 v[%c9:%c10], %2, %3, %4 offen offset:%c11"
                 :: "v"(v0), "v"(v1), "v"(v2), "s"(srd), "s"(soff), "n"(144 + 12 * C), "n"(147 + 12 * C), "n"(148 + 12 * C), "n"(151 + 12 * C),
                    "n"(152 + 12 * C), "n"(155 + 12 * C), "n"(C * 128) : "memory", XCLOB);
}
template <int C> DI void x_store(unsigned voff, unsigned voff2, __amdgpu_buffer_rsrc_t srd, unsigned soff) {
    asm volatile("buffer_store_dwordx4 v[%c4:%c5], %0, %2, %3 offen sc0 sc1\n\tbuffer_store_dwordx4 v[%c6:%c7], %0, %2, %3 offen offset:1024 sc0 sc1\n\t"
                 "buffer_store_dwordx4 v[%c8:%c9], %1, %2, %3 offen sc0 sc1"
                 :: "v"(voff), "v"(voff2), "s"(srd), "s"(soff), "n"(144 + 12 * C), "n"(147 + 12 * C), "n"(148 + 12 * C), "n"(151 + 12 * C),
                    "n"(152 + 12 * C), "n"(155 + 12 * C) : "memory");
}
template <int N> DI void wait_v() { asm volatile("s_waitcnt vmcnt(%0)" :: "n"(N) : "memory"); }
#define WAIT_L0() __builtin_amdgcn_s_waitcnt(0xC07F)
#define FENCE() asm volatile("" ::: "memory")
#define BAR() __builtin_amdgcn_s_barrier()
#define SCHED() __builtin_amdgcn_sched_barrier(0)
#define PRIO(n) __builtin_amdgcn_s_setprio(n)
#define MFMA(a, b, c) __builtin_amdgcn_mfma_f32_16x16x32_bf16(a, b, c, 0, 0, 0)

#define STAMP(i) do {} while (0)
#define CSTAMP(i) do {} while (0)
#define STAMP_DECL
#define STAMP_FLUSH(lo, hi) do {} while (0)

DI unsigned ktile_of(int t, int rank) {
    const int i = t - 8, c = (i * 37) >> 8, j = i - c * 7;
    return t < 8 ? (unsigned)(rank * 8 + t) : (unsigned)((((rank + 1 + j) & 7) << 3) + c);
}

DI bool timed_out(unsigned* tmo, unsigned call, u64 t0, unsigned& polls) {
    if ((++polls & 31u) != 0u) return false;
    if (__builtin_amdgcn_readfirstlane(__hip_atomic_load(tmo, __ATOMIC_RELAXED, __HIP_MEMORY_SCOPE_AGENT)) == call
        || __builtin_amdgcn_s_memrealtime() - t0 > DEADLINE_TICKS) {
        __hip_atomic_store(tmo, call, __ATOMIC_RELAXED, __HIP_MEMORY_SCOPE_AGENT);
        return true;
    }
    return false;
}
DI unsigned load_seq(const unsigned* p) {
    unsigned v;
    asm volatile("s_load_dword %0, %1, 0x0 glc\n\ts_waitcnt lgkmcnt(0)" : "=s"(v) : "s"(p) : "memory");
    return v;
}
DI unsigned vm_outstanding() {
    const unsigned s = __builtin_amdgcn_s_getreg(7 | (0 << 6) | (31 << 11));
    return (s & 15u) | ((s >> 18) & 48u);
}
DI unsigned row_umax(unsigned v) {
    v = max(v, (unsigned)__builtin_amdgcn_update_dpp(0, (int)v, 0x111, 0xf, 0xf, false));
    v = max(v, (unsigned)__builtin_amdgcn_update_dpp(0, (int)v, 0x112, 0xf, 0xf, false));
    v = max(v, (unsigned)__builtin_amdgcn_update_dpp(0, (int)v, 0x114, 0xf, 0xf, false));
    v = max(v, (unsigned)__builtin_amdgcn_update_dpp(0, (int)v, 0x118, 0xf, 0xf, false));
    return v;
}
DI unsigned wave_umax(unsigned v) {
    v = row_umax(v);
    const unsigned a = __builtin_amdgcn_readlane(v, 15), b = __builtin_amdgcn_readlane(v, 31);
    const unsigned c = __builtin_amdgcn_readlane(v, 47), d = __builtin_amdgcn_readlane(v, 63);
    return max(max(a, b), max(c, d));
}

__global__ __launch_bounds__(NTHREADS) void a2a_gemm_kernel(Args a) {
    extern __shared__ __attribute__((aligned(16))) char smem[];
    LAS char* lds = (LAS char*)smem;
    const int tid = threadIdx.x, lane = tid & 63;
    const int wid = __builtin_amdgcn_readfirstlane(tid >> 6);
    const int wg = blockIdx.x, rank = a.rank;
    const int xcd = wg & 7, idx = wg >> 3;
    const int ntile = xcd * 16 + (idx >> 1), mtile = idx & 1;
    const unsigned ldsbase = (unsigned)(size_t)lds;
    LAS unsigned* red = (LAS unsigned*)(lds + LDS_B0 + 3 * B_STAGE);
    char* wsl = a.wsl;
    unsigned* seqp = (unsigned*)(wsl + WS_SEQ + wg * 64);
    unsigned* tmo = (unsigned*)(wsl + WS_TMO);
    STAMP_DECL;

    if (wid == 4 || wid == 5) {
        PRIO(3);
        const int half = wid - 4;
        const __amdgpu_buffer_rsrc_t ss = __builtin_amdgcn_make_buffer_rsrc((void*)wsl, 0, WS_TMO, SRD3);
        const __amdgpu_buffer_rsrc_t wr = __builtin_amdgcn_make_buffer_rsrc((void*)a.w, 0, W_BYTES, SRD3);
        unsigned vs[2], vw[2];
#pragma unroll
        for (int e = 0; e < 2; ++e) {
            vs[e] = (unsigned)(lane >> 3) * 128u + (unsigned)(((lane & 7) ^ (4 * e + (lane >> 4))) << 4);
            vw[e] = (unsigned)(lane >> 4) * 32768u + (unsigned)(((lane & 15) ^ ((e << 3) | (lane >> 4))) << 4);
        }
        const unsigned row0 = (unsigned)(mtile * BM + half * 128);
        const unsigned lA = ldsbase + (unsigned)half * 16384u, lB = ldsbase + LDS_B0 + (unsigned)half * 8192u;
        const unsigned soB = (unsigned)ntile * (BN * 2u) + (unsigned)half * (32u * 32768u);
        const unsigned soOwn = (unsigned)rank * SLOT_BYTES + row0 * 1024u;
        const char* xb = (const char*)a.x;
        unsigned dx = 7168u, v0 = vs[0] + (unsigned)(lane >> 3) * 896u, v1 = vs[1] + (unsigned)(lane >> 3) * 896u;
        unsigned la = lA;
        auto next_la = [&](unsigned l) -> unsigned { return l == lA + (RA - 1) * A_STAGE ? lA : l + A_STAGE; };
        auto issueA = [&](unsigned so) {
            const __amdgpu_buffer_rsrc_t srd = __builtin_amdgcn_make_buffer_rsrc((void*)xb, 0, X_BYTES, SRD3);
#pragma unroll
            for (int p = 0; p < 16; p += 4) {
                const unsigned s4 = so + p * (dx + 1024u);
                set_m0(la + p * 1024u);
                dma_x<0>(srd, v0, s4); dma_x<1024>(srd, v1, s4 + dx); dma_x<2048>(srd, v0, s4 + 2u * dx); dma_x<3072>(srd, v1, s4 + 3u * dx);
            }
            la = next_la(la);
        };
        auto issueAs = [&](unsigned so, unsigned l) {
#pragma unroll
            for (int p = 0; p < 16; p += 4) {
                set_m0(l + p * 1024u);
                dma_x<0>(ss, vs[0], so + p * 1024u); dma_x<1024>(ss, vs[1], so + p * 1024u); dma_x<2048>(ss, vs[0], so + p * 1024u); dma_x<3072>(ss, vs[1], so + p * 1024u);
            }
        };
        auto wstage = [&](int t) -> unsigned { return lB + ((unsigned)t & (RB - 1)) * B_STAGE; };
        auto issueW = [&](unsigned so, unsigned lb, auto lo, auto hi) {
            constexpr int LO = decltype(lo)::value, HI = decltype(hi)::value;
            if constexpr (LO == 0) {
                set_m0(lb);
                dma_w<0>(wr, vw[0], so); dma_w<1024>(wr, vw[0], so + 130048u); dma_w<2048>(wr, vw[1], so + 260096u); dma_w<3072>(wr, vw[1], so + 390144u);
            }
            if constexpr (HI > 4 && LO < 8) set_m0(lb + 4096u);
            if constexpr (LO <= 4 && HI >= 6) { dma_w<0>(wr, vw[0], so + 524288u); dma_w<1024>(wr, vw[0], so + 654336u); }
            if constexpr (LO <= 6 && HI == 8) { dma_w<2048>(wr, vw[1], so + 784384u); dma_w<3072>(wr, vw[1], so + 914432u); }
        };
        using I0 = IC<0>; using IS = IC<WSPLIT>; using I8 = IC<8>;
        constexpr int N_HEAVY = 16 + 8 + WSPLIT, N_LIGHT = 16 + 8;
        const unsigned fvo = WS_XFLAG - 1024u + (unsigned)lane * 16u;
#define POLL_ISSUE(n) asm volatile("buffer_load_dwordx4 v[240:243], %0, %1, %2 offen sc0 sc1" :: "v"(fvo), "s"(ss), "s"((n) * 1024u) : "v240", "v241", "v242", "v243", "memory")
        const unsigned soW0 = (unsigned)rank * (8u * W_TILE) + soB;
        issueA(soOwn); issueW(soW0, wstage(0), I0{}, I8{});
        issueW(soW0 + W_TILE, wstage(1), I0{}, I8{});
        unsigned tabA, tabW;
        {
            const int t = lane + 3;
            const unsigned kt = ktile_of(t, rank);
            const unsigned first = (int)(kt >> 3) == ((rank + 1) & 7) ? 16u : 0u;
            tabA = t < 8 ? soOwn + (unsigned)t * 128u : (t < NT ? kt * 65536u + row0 * 128u + first + (kt & 7u) + 1u : OOB);
            const unsigned n0 = t < 8 ? 0u : (kt & 7u) + 1u, n2 = t + 2 < 8 ? 0u : (ktile_of(t + 2, rank) & 7u) + 1u;
            tabW = t < NT ? kt * W_TILE + soB + ((t + 2 < NT && n2 > n0) ? n2 : 0u) : OOB;
        }
        const unsigned call = load_seq(seqp) + 1u;
        wait_v<8>(); BAR();
        issueA(soOwn + 128u);
        issueW(soW0 + 2u * W_TILE, wstage(2), I0{}, I8{}); issueA(soOwn + 256u);
        if (mtile == 0) issueW(soW0 + 3u * W_TILE, wstage(3), I0{}, IS{});
        const bool own = (lane >> 3) == rank || (lane & 7) >= NPART / 4;
        bool good = true;
        unsigned prog = 0;
        auto flags_ok = [&]() -> bool {
            u32x4 f;
            asm volatile("v_mov_b32 %0, v240\n\tv_mov_b32 %1, v241\n\tv_mov_b32 %2, v242\n\tv_mov_b32 %3, v243" : "=v"(f.x), "=v"(f.y), "=v"(f.z), "=v"(f.w) :: "memory");
            return __all(own || (int)((f.x - call) | (f.y - call) | (f.z - call) | (f.w - call)) >= 0);
        };
        auto await = [&](unsigned need) {
            if (__builtin_expect(prog < need, 0)) {
                if (half == 0) STAMP(40 + need - 1);
                const u64 t0 = __builtin_amdgcn_s_memrealtime();
                unsigned polls = 0;
                for (;;) {
                    POLL_ISSUE(need); wait_v<0>();
                    if (flags_ok()) break;
                    if (timed_out(tmo, call, t0, polls)) { good = false; break; }
                    __builtin_amdgcn_s_sleep(4);
                }
                prog = need;
                if (half == 0) STAMP(48 + need - 1);
            }
        };
        unsigned pc = 0;
        int i = 0;
#pragma unroll 1
        for (; i < 5 + mtile; ++i) {
            {
                const bool mine = i < 5;
                xb = mine ? (const char*)a.x : (const char*)wsl;
                dx = mine ? 7168u : 0u;
                const unsigned dd = mine ? 896u : 0u;
                v0 = vs[0] + (unsigned)(lane >> 3) * dd; v1 = vs[1] + (unsigned)(lane >> 3) * dd;
            }
            const unsigned ta = __builtin_amdgcn_readlane(tabA, i), w3 = __builtin_amdgcn_readlane(tabW, i), w4 = __builtin_amdgcn_readlane(tabW, i + 1);
            if ((i ^ mtile) & 1) {
                wait_v<N_LIGHT>(); BAR();
                pc = w3 & 15u;
                if (pc != 0u) POLL_ISSUE(pc);
                issueA(ta & ~127u);
                issueW(w3 & ~255u, wstage(i + 3), I0{}, I8{});
                issueW(w4 & ~255u, wstage(i + 4), I0{}, IS{});
            } else {
                wait_v<N_HEAVY>(); BAR();
                issueW(w3 & ~255u, wstage(i + 3), IS{}, I8{});
                if (pc != 0u) { if (flags_ok()) prog = pc; }
                await(ta & 15u);
                issueA(ta & ~127u);
            }
        }
        unsigned aH, aL, wH, wE, laH, laL, lbH, lbE, nH, nL, rareH, rareL;
        auto plan = [&](int h) {
            const unsigned t0 = __builtin_amdgcn_readlane(tabA, h & 63), t1 = __builtin_amdgcn_readlane(tabA, (h + 1) & 63);
            const unsigned u0 = __builtin_amdgcn_readlane(tabW, h & 63), u1 = __builtin_amdgcn_readlane(tabW, (h + 1) & 63);
            aH = t0 & ~127u; nH = t0 & 15u; aL = t1 & ~127u; nL = t1 & 15u; wH = u0 & ~255u; pc = u0 & 15u; wE = u1 & ~255u;
            laH = la; laL = next_la(la); la = next_la(laL);
            lbH = wstage(h + 3); lbE = wstage(h + 4);
            rareH = pc | (t0 & 16u); rareL = pc | (t1 & 16u);
            asm volatile("" : "+s"(aH), "+s"(aL), "+s"(wH), "+s"(wE), "+s"(laH), "+s"(laL), "+s"(lbH), "+s"(lbE), "+s"(rareH), "+s"(rareL));
        };
        plan(i);
#pragma unroll 1
        for (int n = 0; n < 29; ++n) {
            wait_v<N_LIGHT>(); BAR();
            if (__builtin_expect(rareH != 0u, 0)) { await(nH); if (pc != 0u) POLL_ISSUE(pc); }
            issueAs(aH, laH);
            issueW(wH, lbH, I0{}, I8{});
            issueW(wE, lbE, I0{}, IS{});
            wait_v<N_HEAVY>(); BAR();
            issueW(wE, lbE, IS{}, I8{});
            if (__builtin_expect(rareL != 0u, 0)) { if (pc != 0u && flags_ok()) prog = pc; await(nL); }
            issueAs(aL, laL);
            i += 2;
            plan(i);
        }
        if (mtile == 0) BAR();
        if (lane == 0) red[4 + half] = good ? 0u : 0xffffffffu;
        u64 gdst = 0;
        if (half == 1 && wg == COLLECTOR) {
            char* pw = a.ws[0];
#pragma unroll
            for (int r = 1; r < NR; ++r) pw = ((lane >> 3) == r) ? a.ws[r] : pw;
            gdst = (u64)(pw + WS_FIN + (lane & 7) * 128 + rank * 8);
            asm volatile("" : "+v"(gdst));
        }
        FENCE(); WAIT_L0(); BAR(); FENCE();
        if (half == 0) {
            const unsigned wgmax = wave_umax((lane >= 4 && lane < 24 && (lane & ~1) != 6) ? red[lane] : 0u);
            if (lane == 0) {
                __hip_atomic_store((u64*)(wsl + WS_GRAN) + wg, ((u64)call << 32) | wgmax, __ATOMIC_RELAXED, __HIP_MEMORY_SCOPE_AGENT);
                __hip_atomic_store(seqp, call, __ATOMIC_RELAXED, __HIP_MEMORY_SCOPE_AGENT);
            }
            STAMP(60);
        } else if (wg == COLLECTOR) {
            const unsigned wgmax = wave_umax((lane >= 4 && lane < 24 && (lane & ~1) != 6) ? red[lane] : 0u);
            const u64 t0 = __builtin_amdgcn_s_memrealtime();
            unsigned polls = 0, gm;
            for (;;) {
                FENCE();
                const u32x4 g0 = __builtin_amdgcn_raw_buffer_load_b128(ss, WS_GRAN + lane * 16, 0, 16);
                const u32x4 g1 = __builtin_amdgcn_raw_buffer_load_b128(ss, WS_GRAN + 1024 + lane * 16, 0, 16);
                const bool self = lane == 63;
                gm = max(max(g0.x, g0.z), max(g1.x, self ? wgmax : g1.z));
                if (__all(g0.y == call && g0.w == call && g1.y == call && (self || g1.w == call))) break;
                if (timed_out(tmo, call, t0, polls)) { gm = 0xffffffffu; break; }
            }
            gm = wave_umax(gm);
            __hip_atomic_store((__attribute__((address_space(1))) u64*)gdst, ((u64)call << 32) | gm, __ATOMIC_RELAXED, __HIP_MEMORY_SCOPE_SYSTEM);
        }
        if (half == 0) { STAMP_FLUSH(16, 24); STAMP_FLUSH(40, 64); }
        return;
    }
    if (wid == 6) {
        PRIO(3);
        STAMP(32);
        const __amdgpu_buffer_rsrc_t xs = __builtin_amdgcn_make_buffer_rsrc((void*)a.x, 0, X_BYTES, SRD3);
        const unsigned po = (unsigned)(idx * 16 + (lane & 15)) * 1024u + (unsigned)(3 + (lane >> 4)) * 128u;
#define OWN_TOUCH() asm volatile("buffer_load_dword v240, %0, %1, %2 offen\n\tbuffer_load_dword v240, %3, %1, %2 offen" \
                         :: "v"(po), "s"(xs), "s"((unsigned)rank * SLOT_BYTES), "v"(po | 896u) : "v240", "memory")
        int nb = 0;
        const int s4 = (int)(((unsigned)(size_t)a.w >> 8) + 3u) & 3, u4 = s4 + (idx >> 1);
        const int fid = ((idx >> 1) - (((u4 + 3) >> 2) - ((s4 + 3) >> 2))) * 2 + mtile;
        if ((u4 & 3) != 0 && fid < 7 * NPART / 8) {
            const int id = fid * 8 + xcd, j = id % 7 + 1, part = id / 7, dst = (rank + j) & 7;
            char* wsd = a.ws[0];
#pragma unroll
            for (int r = 1; r < NR; ++r) wsd = (dst == r) ? a.ws[r] : wsd;
            const __amdgpu_buffer_rsrc_t ds = __builtin_amdgcn_make_buffer_rsrc((void*)wsd, 0, WS_GRAN, SRD3);
            const unsigned piece = part < 16 ? 3u * part : 2u * part + 16u, none = part < 16 ? 0u : OOB;
            const unsigned vsrc = (piece * 8u + (unsigned)(lane >> 3)) * 1024u + (unsigned)(lane & 7) * 16u;
            const unsigned vdst = piece * 1024u + (unsigned)lane * 16u;
            const unsigned fdst = WS_XFLAG + (unsigned)rank * 128u + (unsigned)part * 4u;
            const unsigned vsrc2 = vsrc + 8192u, vsrc3 = vsrc + 16384u + none, vdst3 = vdst + 2048u + none;
            const unsigned ssrc = (unsigned)dst * SLOT_BYTES, sdst = (unsigned)rank * SLOT_BYTES;
            auto xload = [&](auto cc) { x_load<decltype(cc)::value>(vsrc, vsrc2, vsrc3, xs, ssrc); };
            auto xstore = [&](auto cc) { x_store<decltype(cc)::value>(vdst, vdst3, ds, sdst + (unsigned)decltype(cc)::value * 65536u); };
            xload(IC<0>{}); xload(IC<1>{});
            if (OWN_TOUCH_AT == 0) OWN_TOUCH();
            const unsigned call = load_seq(seqp) + 1u;
            auto quiet = [&](int c, int from, int spins) {
                int spin = 0;
#pragma unroll 1
                for (;;) {
                    const bool last = nb >= 7 + 7 * c;
                    if (nb >= from && (c == 0 || last || ((nb ^ mtile) & 1) == 0) && vm_outstanding() == 0u) break;
                    if (!last && (nb < from || ++spin > spins)) { BAR(); ++nb; spin = 0; }
                    else __builtin_amdgcn_s_sleep(1);
                }
            };
#pragma unroll 1
            for (int c = 0; c < NCHUNK; ++c) {
                const int slot = c == 0 ? PUSH0 + part % PUSH0_SPREAD : (c == 1 ? 2 : PUSH_PACE * c - PUSH_LEAD) + (part & 2) + mtile;
                if (c <= 2) quiet(c, slot, 0);
                else {
#pragma unroll 1
                    while (nb < slot) { BAR(); ++nb; }
                }
                switch (c) {
                    case 0: xstore(IC<0>{}); if (BULK_AT == 0) { xload(IC<2>{}); xload(IC<3>{}); xload(IC<4>{}); xload(IC<5>{}); xload(IC<6>{}); xload(IC<7>{}); } break;
                    case 1: xstore(IC<1>{}); if (BULK_AT == 1) { xload(IC<2>{}); xload(IC<3>{}); xload(IC<4>{}); xload(IC<5>{}); xload(IC<6>{}); xload(IC<7>{}); } break;
                    case 2: xstore(IC<2>{}); break;
                    case 3: xstore(IC<3>{}); break;
                    case 4: xstore(IC<4>{}); break;
                    case 5: xstore(IC<5>{}); break;
                    case 6: xstore(IC<6>{}); break;
                    default: xstore(IC<7>{}); break;
                }
                if (c == 0 && OWN_TOUCH_AT == 1) OWN_TOUCH();
                quiet(c, 0, c == 0 ? PUSH_SPIN : 0);
                FENCE();
                if (lane == 0) __builtin_amdgcn_raw_buffer_store_b32(call, ds, fdst, c * 1024, SYS);
                FENCE();
                if (c == 0) STAMP(34);
            }
        } else OWN_TOUCH();
        STAMP(35);
        STAMP_FLUSH(32, 40);
        while (vm_outstanding() != 0u) BAR();
        return;
    }

    if (wid == 7) {
        const u64 t0 = __builtin_amdgcn_s_memrealtime();
        BAR();
        int next = PACE_T0, cap = 0;
#pragma unroll 1
        for (int i = 0; i < PACE_TO2; ++i) {
            next += i < PACE_TO ? PACE_NS : PACE_NS2;
            int now = (int)(unsigned)(__builtin_amdgcn_s_memrealtime() - t0) * 10;
            const int lim = next;
            while (now < lim) { __builtin_amdgcn_s_sleep(1); now = (int)(unsigned)(__builtin_amdgcn_s_memrealtime() - t0) * 10; }
            BAR();
        }
        return;
    }

    if (wid == 0) STAMP(0);
    const int wm = wid & 1, wn = wid >> 1;
    const int fr = lane & 15, g = lane >> 4, q = fr >> 2, p = lane & 3;
    unsigned aoff[2], boff[4];
#pragma unroll
    for (int kk = 0; kk < 2; ++kk) aoff[kk] = ldsbase + (unsigned)((wm * 128 + fr) * 128 + (((kk * 4 + g) ^ (fr >> 1)) << 4));
#pragma unroll
    for (int f = 0; f < 4; ++f)
        boff[f] = ldsbase + LDS_B0 + (unsigned)((8 * g + q) * 256 + (((wn * 8 + 2 * p + (f >> 1)) ^ (q | ((g & 1) << 3))) << 4) + ((((f & 1) ^ (p & 1))) << 3));
    asm volatile("" : "+v"(boff[0]), "+v"(boff[1]), "+v"(boff[2]), "+v"(boff[3]), "+v"(aoff[0]), "+v"(aoff[1]));
    const unsigned call = load_seq(seqp) + 1u;
    f32x4 acc[8][4];
#pragma unroll
    for (int m = 0; m < 8; ++m)
#pragma unroll
        for (int f = 0; f < 4; ++f) acc[m][f] = (f32x4){0.f, 0.f, 0.f, 0.f};
    bf16x8 x0[8], w0[4], x1[8], w1[4];
    bf16x4 tl[4];
    unsigned bcur[4], acur = aoff[0];
#pragma unroll
    for (int f = 0; f < 4; ++f) bcur[f] = boff[f];
#define RD_WL(f, kk) do { tl[f] = __builtin_amdgcn_ds_read_tr16_b64_v4bf16((LAS bf16x4*)(bcur[f] + (kk) * 8192)); } while (0)
#define RD_WH(dst, f, kk) do { const bf16x4 hi_ = __builtin_amdgcn_ds_read_tr16_b64_v4bf16((LAS bf16x4*)(bcur[f] + (kk) * 8192 + 1024)); \
        dst = __builtin_shufflevector(tl[f], hi_, 0, 1, 2, 3, 4, 5, 6, 7); } while (0)
#define RD_XA(dst, m) do { dst = *(const LAS bf16x8*)(acur + (m) * 2048); } while (0)
    FENCE(); BAR(); FENCE();
    if (wid == 0) { STAMP(2); CSTAMP(30); }
#pragma unroll
    for (int i = 0; i < 4; ++i) { RD_WL(i, 0); RD_WH(w0[i], i, 0); }
#pragma unroll
    for (int i = 0; i < 8; ++i) RD_XA(x0[i], i);
    acur = aoff[1];
    auto step = [&](bf16x8* cw, bf16x8* cx, bf16x8* nw, bf16x8* nx, auto kkc, auto side) {
        constexpr int kk = decltype(kkc)::value;
#pragma unroll
        for (int s = 0; s < 32; ++s) {
            const int m = s >> 2, f = (m & 1) ? 3 - (s & 3) : (s & 3);
            acc[m][f] = MFMA(cw[f], cx[m], acc[m][f]);
            SCHED();
            if (s < 24 && s % 3 != 1) {
                const int r = (s * 2) / 3;
                if (r < 8) { if (r & 1) RD_WH(nw[r >> 1], r >> 1, kk); else RD_WL(r >> 1, kk); }
                else RD_XA(nx[r - 8], r - 8);
            }
            if (s >= 24 && s < 31) side(s - 24);
            SCHED();
        }
    };
    unsigned oa = A_STAGE, ob = B_STAGE;
    auto stepA = [&]() {
        step(w0, x0, w1, x1, IC<1>{}, [&](int h) {
            if (h < 4) bcur[h] = boff[h] + ob;
            else if (h == 4) acur = aoff[0] + oa;
            else if (h == 5) { ob = (ob + B_STAGE) & (RB * B_STAGE - 1); asm volatile("" : "+s"(ob)); } });
        WAIT_L0(); SCHED();
    };
    auto stepB = [&]() {
        BAR(); SCHED();
        step(w1, x1, w0, x0, IC<0>{}, [&](int h) {
            if (h == 0) acur = aoff[1] + oa;
            else if (h == 1) { oa += A_STAGE; asm volatile("" : "+s"(oa)); }
            else if (h == 2) { oa = oa == RA * A_STAGE ? 0u : oa; asm volatile("" : "+s"(oa)); }
            else if (h == 6) WAIT_L0(); });
    };
    WAIT_L0();
    stepA();
#pragma unroll 1
    for (int t = 0; t < NT - 1; t += KUNR) {
#pragma unroll
        for (int u = 0; u < KUNR; ++u) { stepB(); stepA(); }
    }

    int mx0 = 0, mx1 = 0;
    SCHED(); BAR(); SCHED();
#pragma unroll
    for (int s = 0; s < 36; ++s) {
        if (s < 32) { const int m = s >> 2, f = (m & 1) ? 3 - (s & 3) : (s & 3); acc[m][f] = MFMA(w1[f], x1[m], acc[m][f]); }
        SCHED();
        if (s >= 4) {
            const int m = (s - 4) >> 2, f = (m & 1) ? 3 - (s & 3) : (s & 3);
            mx0 = max(max(mx0, __float_as_int(acc[m][f][0])), __float_as_int(acc[m][f][1]));
            mx1 = max(max(mx1, __float_as_int(acc[m][f][2])), __float_as_int(acc[m][f][3]));
        }
        SCHED();
    }
    const unsigned wmax = row_umax((unsigned)max(mx0, mx1));
    if (wid == 0) { STAMP(11); CSTAMP(31); }
    if (fr == 15) red[8 + wid * 4 + g] = wmax;
    FENCE(); WAIT_L0(); BAR(); FENCE();
    const unsigned wgmax = wave_umax((lane >= 4 && lane < 24 && (lane & ~1) != 6) ? red[lane] : 0u);
    const bool odd = g & 1;
    const __amdgpu_buffer_rsrc_t outr = __builtin_amdgcn_make_buffer_rsrc(a.out, 0, MLOC * NDIM, SRD3);
    const unsigned ovo = (unsigned)(mtile * BM + wm * 128 + fr) * NDIM + (unsigned)(ntile * BN + wn * 64 + g * 16);
    unsigned used = 0xfffffffeu;
    const char* finp = wsl + WS_FIN + xcd * 128;
    const u64 t0 = __builtin_amdgcn_s_memrealtime();
    unsigned polls = 0;
    for (;;) {
        unsigned gm = wgmax, miss = 0;
        {
            const u64 fv = __hip_atomic_load((const u64*)finp + (lane & 7), __ATOMIC_RELAXED, __HIP_MEMORY_SCOPE_SYSTEM);
            const bool ok = (unsigned)(fv >> 32) == call;
            gm = max(gm, (unsigned)__builtin_amdgcn_readlane(row_umax(ok ? (unsigned)fv : 0u), 15));
            miss = (unsigned)__popcll(__ballot(!ok)) >> 3;
        }
        if (miss != 0u && timed_out(tmo, call, t0, polls)) gm = 0xffffffffu;
        if (gm == 0xffffffffu) {
            const u32x4 o = {0x7f7f7f7fu, 0x7f7f7f7fu, 0x7f7f7f7fu, 0x7f7f7f7fu};
#pragma unroll
            for (int m = 0; m < 8; ++m) __builtin_amdgcn_raw_buffer_store_b128(o, outr, ovo + m * 16 * NDIM, 0, OUT_AUX);
            break;
        }
        if (miss == 0u && wid == 0) STAMP(13);
        if (gm != used && miss <= (wg == COLLECTOR && COLL_QUIET ? 0u : SPEC_THR)) {
            used = gm;
            const float amax = __uint_as_float(gm);
            const float inv = amax > 0.f ? 448.0f / amax : 0.f;
            const f32x2 sc = {inv * (1.0f / 512.0f), inv * (1.0f / 512.0f)};
#pragma unroll
            for (int m = 0; m < 8; ++m) {
                unsigned d[4];
#pragma unroll
                for (int f = 0; f < 4; ++f) {
                    f32x2 lo = {acc[m][f][0], acc[m][f][1]}, hi = {acc[m][f][2], acc[m][f][3]};
                    asm("v_pk_mul_f32 %0, %1, %2 clamp" : "=v"(lo) : "v"(lo), "v"(sc));
                    asm("v_pk_mul_f32 %0, %1, %2 clamp" : "=v"(hi) : "v"(hi), "v"(sc));
                    s16x2 w = __builtin_bit_cast(s16x2, lo.x);
                    w = __builtin_amdgcn_cvt_scalef32_pk_fp8_f32(w, lo.x, lo.y, 1.0f / 512.0f, false);
                    w = __builtin_amdgcn_cvt_scalef32_pk_fp8_f32(w, hi.x, hi.y, 1.0f / 512.0f, true);
                    d[f] = __builtin_bit_cast(unsigned, w);
                }
                u32x4 o;
                o.x = odd ? d[1] : d[0]; o.y = odd ? d[0] : d[1]; o.z = odd ? d[3] : d[2]; o.w = odd ? d[2] : d[3];
                __builtin_amdgcn_raw_buffer_store_b128(o, outr, ovo + m * 16 * NDIM, 0, OUT_AUX);
            }
        }
        if (miss == 0u) break;
        __builtin_amdgcn_s_sleep(FIN_SLEEP);
    }
}

}

extern "C" void kernel_launch(int rank, int nranks, ncclComm_t comm,
                              hipStream_t stream,
                              void* const* d_in, const int* in_sizes, int n_in,
                              void* d_out, int out_size,
                              const int* in_dtypes, int out_dtype,
                              void* const* d_out_all,
                              void* const* d_ws_all, size_t ws_bytes) {
    using namespace a2ag;
    if (nranks != NR || (unsigned)rank >= (unsigned)NR || n_in < 2 || ws_bytes < WS_NEED
        || in_sizes[0] != KDIM * KSL || in_sizes[1] != KDIM * NDIM || out_size != MLOC * NDIM) {
        fprintf(stderr, "kernel_launch: unexpected problem geometry\n");
        return;
    }
    HIP_CHECK(hipFuncSetAttribute((const void*)a2a_gemm_kernel, hipFuncAttributeMaxDynamicSharedMemorySize, LDS_BYTES));
    Args a;
    a.x = d_in[0]; a.w = d_in[1]; a.out = d_out; a.rank = rank;
    a.wsl = (char*)d_ws_all[rank];
    for (int r = 0; r < NR; ++r) a.ws[r] = (char*)d_ws_all[r];
    hipLaunchKernelGGL(a2a_gemm_kernel, dim3(NWG), dim3(NTHREADS), LDS_BYTES, stream, a);
    HIP_CHECK(hipGetLastError());
}
```

```cpp
#include <hip/hip_runtime.h>
#include <rccl/rccl.h>
#include <cstdio>
#include <cstdint>
#include <type_traits>

#define HIP_CHECK(expr) do { hipError_t _e = (expr); if (_e != hipSuccess) { \
    fprintf(stderr, "HIP error %s at %s:%d\n", hipGetErrorString(_e), __FILE__, __LINE__); \
    return; } } while (0)

#define PUSH_PACE 4
#define SPEC_THR 4
#define OUT_AUX 17
#define WSPLIT 6
#define PER_PEER 1
#define PACE_NS 825
#define PACE_T0 1700
#define PACE_TO 36

#pragma clang diagnostic ignored "-Wint-to-pointer-cast"

namespace a2ag {

typedef __bf16 bf16x4 __attribute__((ext_vector_type(4)));
typedef __bf16 bf16x8 __attribute__((ext_vector_type(8)));
typedef float f32x4 __attribute__((ext_vector_type(4)));
typedef float f32x2 __attribute__((ext_vector_type(2)));
typedef short s16x2 __attribute__((ext_vector_type(2)));
typedef unsigned u32x4 __attribute__((ext_vector_type(4)));
typedef unsigned long long u64;
#define LAS __attribute__((address_space(3)))
#define DI __device__ __forceinline__
template <int V> using IC = std::integral_constant<int, V>;

constexpr int NR = 8;
constexpr int MLOC = 512, KDIM = 4096, NDIM = 16384, KSL = 512;
constexpr int BM = 256, BN = 128, BK = 64;
constexpr int NT = KDIM / BK;
constexpr int NWG = (MLOC / BM) * (NDIM / BN);
constexpr int NTHREADS = 512;
constexpr unsigned A_STAGE = BM * BK * 2, B_STAGE = BK * BN * 2;
constexpr int RA = 3, RB = 4;
constexpr unsigned LDS_B0 = RA * A_STAGE;
constexpr unsigned LDS_BYTES = LDS_B0 + RB * B_STAGE;
constexpr unsigned SLOT_BYTES = MLOC * KSL * 2;
constexpr unsigned X_BYTES = NR * SLOT_BYTES;
constexpr unsigned W_BYTES = (unsigned)KDIM * NDIM * 2;
constexpr unsigned W_TILE = BK * NDIM * 2;
constexpr int NCHUNK = 8, NPART = 24;
constexpr unsigned WS_XFLAG = X_BYTES;
constexpr unsigned WS_GRAN = WS_XFLAG + NCHUNK * 1024;
constexpr unsigned WS_FIN = WS_GRAN + NWG * 8;
constexpr unsigned WS_SEQ = WS_FIN + 8 * 128;
constexpr unsigned WS_TMO = WS_SEQ + NWG * 64;
constexpr unsigned WS_DBG = WS_TMO + 4096;
constexpr unsigned WS_NEED = WS_DBG + NWG * 64 * 12;
constexpr u64 DEADLINE_TICKS = 40000000ull;
constexpr unsigned OOB = 0x40000000u;
constexpr unsigned SRD3 = 0x00020000u;
constexpr int COLLECTOR = NWG - 1;
constexpr int SYS = 17;

struct Args {
    const void* x; const void* w;
    char* wsl;
    int rank, pad;
    void* out;
    char* ws[NR];
};

DI void set_m0(unsigned v) { asm volatile("s_mov_b32 m0, %0\n\ts_nop 0" :: "s"(v) : "memory"); }
template <int OFF> DI void dma_x(__amdgpu_buffer_rsrc_t srd, unsigned voff, unsigned soff) {
    asm volatile("buffer_load_dwordx4 %0, %1, %2 offen offset:%3 sc1 lds" :: "v"(voff), "s"(srd), "s"(soff), "n"(OFF) : "memory");
}
template <int OFF> DI void dma_w(__amdgpu_buffer_rsrc_t srd, unsigned voff, unsigned soff) {
    asm volatile("buffer_load_dwordx4 %0, %1, %2 offen offset:%3 lds" :: "v"(voff), "s"(srd), "s"(soff), "n"(OFF) : "memory");
}
#define XCLOB "v144", "v145", "v146", "v147", "v148", "v149", "v150", "v151", "v152", "v153", "v154", "v155", "v156", "v157", "v158", "v159", "v160", "v161", "v162", "v163", "v164", "v165", "v166", "v167", "v168", "v169", "v170", "v171", "v172", "v173", "v174", "v175", "v176", "v177", "v178", "v179", "v180", "v181", "v182", "v183", "v184", "v185", "v186", "v187", "v188", "v189", "v190", "v191", "v192", "v193", "v194", "v195", "v196", "v197", "v198", "v199", "v200", "v201", "v202", "v203", "v204", "v205", "v206", "v207", "v208", "v209", "v210", "v211", "v212", "v213", "v214", "v215", "v216", "v217", "v218", "v219", "v220", "v221", "v222", "v223", "v224", "v225", "v226", "v227", "v228", "v229", "v230", "v231", "v232", "v233", "v234", "v235", "v236", "v237", "v238", "v239"
template <int C> DI void x_load(unsigned v0, unsigned v1, unsigned v2, __amdgpu_buffer_rsrc_t srd, unsigned soff) {
    asm volatile("buffer_load_dwordx4 v[%c5:%c6], %0, %3, %4 offen offset:%c11\n\tbuffer_load_dwordx4 v[%c7:%c8], %1, %3, %4 offen offset:%c11\n\t"
                 "buffer_load_dwordx4 v[%c9:%c10], %2, %3, %4 offen offset:%c11"
                 :: "v"(v0), "v"(v1), "v"(v2), "s"(srd), "s"(soff), "n"(144 + 12 * C), "n"(147 + 12 * C), "n"(148 + 12 * C), "n"(151 + 12 * C),
                    "n"(152 + 12 * C), "n"(155 + 12 * C), "n"(C * 128) : "memory", XCLOB);
}
template <int C> DI void x_store(unsigned voff, unsigned voff2, __amdgpu_buffer_rsrc_t srd, unsigned soff) {
    asm volatile("buffer_store_dwordx4 v[%c4:%c5], %0, %2, %3 offen sc0 sc1\n\tbuffer_store_dwordx4 v[%c6:%c7], %0, %2, %3 offen offset:1024 sc0 sc1\n\t"
                 "buffer_store_dwordx4 v[%c8:%c9], %1, %2, %3 offen sc0 sc1"
                 :: "v"(voff), "v"(voff2), "s"(srd), "s"(soff), "n"(144 + 12 * C), "n"(147 + 12 * C), "n"(148 + 12 * C), "n"(151 + 12 * C),
                    "n"(152 + 12 * C), "n"(155 + 12 * C) : "memory");
}
template <int N> DI void wait_v() { asm volatile("s_waitcnt vmcnt(%0)" :: "n"(N) : "memory"); }
#define WAIT_L0() __builtin_amdgcn_s_waitcnt(0xC07F)
#define FENCE() asm volatile("" ::: "memory")
#define BAR() __builtin_amdgcn_s_barrier()
#define SCHED() __builtin_amdgcn_sched_barrier(0)
#define PRIO(n) __builtin_amdgcn_s_setprio(n)
#define MFMA(a, b, c) __builtin_amdgcn_mfma_f32_16x16x32_bf16(a, b, c, 0, 0, 0)

#define STAMP(i) do {} while (0)
#define CSTAMP(i) do {} while (0)
#define STAMP_DECL
#define STAMP_FLUSH(lo, hi) do {} while (0)

DI unsigned ktile_of(int t, int rank) {
    const int i = t - 8, c = (i * 37) >> 8, j = i - c * 7;
    return t < 8 ? (unsigned)(rank * 8 + t) : (unsigned)((((rank + 1 + j) & 7) << 3) + c);
}

DI bool timed_out(unsigned* tmo, unsigned call, u64 t0, unsigned& polls) {
    if ((++polls & 31u) != 0u) return false;
    if (__builtin_amdgcn_readfirstlane(__hip_atomic_load(tmo, __ATOMIC_RELAXED, __HIP_MEMORY_SCOPE_AGENT)) == call
        || __builtin_amdgcn_s_memrealtime() - t0 > DEADLINE_TICKS) {
        __hip_atomic_store(tmo, call, __ATOMIC_RELAXED, __HIP_MEMORY_SCOPE_AGENT);
        return true;
    }
    return false;
}
DI unsigned load_seq(const unsigned* p) {
    unsigned v;
    asm volatile("s_load_dword %0, %1, 0x0 glc\n\ts_waitcnt lgkmcnt(0)" : "=s"(v) : "s"(p) : "memory");
    return v;
}
DI unsigned vm_outstanding() {
    const unsigned s = __builtin_amdgcn_s_getreg(7 | (0 << 6) | (31 << 11));
    return (s & 15u) | ((s >> 18) & 48u);
}
DI unsigned row_umax(unsigned v) {
    v = max(v, (unsigned)__builtin_amdgcn_update_dpp(0, (int)v, 0x111, 0xf, 0xf, false));
    v = max(v, (unsigned)__builtin_amdgcn_update_dpp(0, (int)v, 0x112, 0xf, 0xf, false));
    v = max(v, (unsigned)__builtin_amdgcn_update_dpp(0, (int)v, 0x114, 0xf, 0xf, false));
    v = max(v, (unsigned)__builtin_amdgcn_update_dpp(0, (int)v, 0x118, 0xf, 0xf, false));
    return v;
}
DI unsigned wave_umax(unsigned v) {
    v = row_umax(v);
    const unsigned a = __builtin_amdgcn_readlane(v, 15), b = __builtin_amdgcn_readlane(v, 31);
    const unsigned c = __builtin_amdgcn_readlane(v, 47), d = __builtin_amdgcn_readlane(v, 63);
    return max(max(a, b), max(c, d));
}

__global__ __launch_bounds__(NTHREADS) void a2a_gemm_kernel(Args a) {
    extern __shared__ __attribute__((aligned(16))) char smem[];
    LAS char* lds = (LAS char*)smem;
    const int tid = threadIdx.x, lane = tid & 63;
    const int wid = __builtin_amdgcn_readfirstlane(tid >> 6);
    const int wg = blockIdx.x, rank = a.rank;
    const int xcd = wg & 7, idx = wg >> 3;
    const int ntile = xcd * 16 + (idx >> 1), mtile = idx & 1;
    const unsigned ldsbase = (unsigned)(size_t)lds;
    LAS unsigned* red = (LAS unsigned*)(lds + LDS_B0 + 3 * B_STAGE);
    char* wsl = a.wsl;
    asm volatile("" :: "s"(a.x), "s"(a.w), "s"(wsl), "s"(rank));
    unsigned* seqp = (unsigned*)(wsl + WS_SEQ + wg * 64);
    unsigned* tmo = (unsigned*)(wsl + WS_TMO);
    STAMP_DECL;

    if (wid == 4 || wid == 5) {
        PRIO(3);
        const int half = wid - 4;
        const __amdgpu_buffer_rsrc_t ss = __builtin_amdgcn_make_buffer_rsrc((void*)wsl, 0, WS_TMO, SRD3);
        const __amdgpu_buffer_rsrc_t wr = __builtin_amdgcn_make_buffer_rsrc((void*)a.w, 0, W_BYTES, SRD3);
        unsigned vs[2], vw[2];
#pragma unroll
        for (int e = 0; e < 2; ++e) {
            vs[e] = (unsigned)(lane >> 3) * 128u + (unsigned)(((lane & 7) ^ (4 * e + (lane >> 4))) << 4);
            vw[e] = (unsigned)(lane >> 4) * 32768u + (unsigned)(((lane & 15) ^ ((e << 3) | (lane >> 4))) << 4);
        }
        const unsigned row0 = (unsigned)(mtile * BM + half * 128);
        const unsigned lA = ldsbase + (unsigned)half * 16384u, lB = ldsbase + LDS_B0 + (unsigned)half * 8192u;
        const unsigned soB = (unsigned)ntile * (BN * 2u) + (unsigned)half * (32u * 32768u);
        const unsigned soOwn = (unsigned)rank * SLOT_BYTES + row0 * 1024u;
        const char* xb = (const char*)a.x;
        unsigned dx = 7168u, v0 = vs[0] + (unsigned)(lane >> 3) * 896u, v1 = vs[1] + (unsigned)(lane >> 3) * 896u;
        unsigned la = lA;
        auto next_la = [&](unsigned l) -> unsigned { return l == lA + (RA - 1) * A_STAGE ? lA : l + A_STAGE; };
        auto issueA = [&](unsigned so) {
            const __amdgpu_buffer_rsrc_t srd = __builtin_amdgcn_make_buffer_rsrc((void*)xb, 0, X_BYTES, SRD3);
#pragma unroll
            for (int p = 0; p < 16; p += 4) {
                const unsigned s4 = so + p * (dx + 1024u);
                set_m0(la + p * 1024u);
                dma_x<0>(srd, v0, s4); dma_x<1024>(srd, v1, s4 + dx); dma_x<2048>(srd, v0, s4 + 2u * dx); dma_x<3072>(srd, v1, s4 + 3u * dx);
            }
            la = next_la(la);
        };
        auto issueAs = [&](unsigned so, unsigned l) {
#pragma unroll
            for (int p = 0; p < 16; p += 4) {
                set_m0(l + p * 1024u);
                dma_x<0>(ss, vs[0], so + p * 1024u); dma_x<1024>(ss, vs[1], so + p * 1024u); dma_x<2048>(ss, vs[0], so + p * 1024u); dma_x<3072>(ss, vs[1], so + p * 1024u);
            }
        };
        auto wstage = [&](int t) -> unsigned { return lB + ((unsigned)t & (RB - 1)) * B_STAGE; };
        auto issueW = [&](unsigned so, unsigned lb, auto lo, auto hi) {
            constexpr int LO = decltype(lo)::value, HI = decltype(hi)::value;
            if constexpr (LO == 0) {
                set_m0(lb);
                dma_w<0>(wr, vw[0], so); dma_w<1024>(wr, vw[0], so + 130048u); dma_w<2048>(wr, vw[1], so + 260096u); dma_w<3072>(wr, vw[1], so + 390144u);
            }
            if constexpr (HI > 4 && LO < 8) set_m0(lb + 4096u);
            if constexpr (LO <= 4 && HI >= 6) { dma_w<0>(wr, vw[0], so + 524288u); dma_w<1024>(wr, vw[0], so + 654336u); }
            if constexpr (LO <= 6 && HI == 8) { dma_w<2048>(wr, vw[1], so + 784384u); dma_w<3072>(wr, vw[1], so + 914432u); }
        };
        using I0 = IC<0>; using IS = IC<WSPLIT>; using I8 = IC<8>;
        constexpr int N_HEAVY = 16 + 8 + WSPLIT, N_LIGHT = 16 + 8;
        const unsigned fvo = WS_XFLAG - 1024u + (unsigned)lane * 16u;
#define POLL_ISSUE(n) asm volatile("buffer_load_dwordx4 v[240:243], %0, %1, %2 offen sc0 sc1" :: "v"(fvo), "s"(ss), "s"((n) * 1024u) : "v240", "v241", "v242", "v243", "memory")
        const unsigned soW0 = (unsigned)rank * (8u * W_TILE) + soB;
        issueA(soOwn); issueW(soW0, wstage(0), I0{}, I8{});
        asm volatile("s_load_dword s96, %0, 0x0 glc" :: "s"(seqp) : "s96", "memory");
        issueW(soW0 + W_TILE, wstage(1), I0{}, I8{});
        issueA(soOwn + 128u);
        unsigned tabA, tabW;
        {
            const int t = lane + 3;
            const unsigned kt = ktile_of(t, rank);
            const unsigned first = (int)(kt >> 3) == ((rank + 1) & 7) ? 16u : 0u;
            tabA = t < 8 ? soOwn + (unsigned)t * 128u : (t < NT ? kt * 65536u + row0 * 128u + first + (kt & 7u) + 1u : OOB);
            const unsigned n0 = t < 8 ? 0u : (kt & 7u) + 1u, n2 = t + 2 < 8 ? 0u : (ktile_of(t + 2, rank) & 7u) + 1u;
            tabW = t < NT ? kt * W_TILE + soB + ((t + 2 < NT && n2 > n0) ? n2 : 0u) : OOB;
        }
        unsigned call;
        asm volatile("s_waitcnt lgkmcnt(0)\n\ts_add_u32 %0, s96, 1" : "=s"(call) :: "scc", "memory");
        wait_v<24>(); BAR();
        issueW(soW0 + 2u * W_TILE, wstage(2), I0{}, I8{}); issueA(soOwn + 256u);
        if (mtile == 0) issueW(soW0 + 3u * W_TILE, wstage(3), I0{}, IS{});
        const bool own = (lane >> 3) == rank || (lane & 7) >= NPART / 4;
        bool good = true;
        unsigned prog = 0;
        unsigned pend = 0, okc = 0;
        u64 okm = 0;
        auto read_flags = [&](unsigned n) {
            u32x4 f;
            asm volatile("v_mov_b32 %0, v240\n\tv_mov_b32 %1, v241\n\tv_mov_b32 %2, v242\n\tv_mov_b32 %3, v243" : "=v"(f.x), "=v"(f.y), "=v"(f.z), "=v"(f.w) :: "memory");
            okm = __ballot(own || (int)((f.x - call) | (f.y - call) | (f.z - call) | (f.w - call)) >= 0);
            okc = n;
            if (okm == ~0ull) prog = n;
        };
        auto await = [&](unsigned need, unsigned so) {
            if (__builtin_expect(prog < need, 0)) {
                const unsigned sh = (so >> 19) * 8u;
                if (!(PER_PEER && okc >= need && ((unsigned)(okm >> sh) & 255u) == 255u)) {
                    if (half == 0) STAMP(40 + need - 1);
                    const u64 t0 = __builtin_amdgcn_s_memrealtime();
                    unsigned polls = 0;
                    for (;;) {
                        POLL_ISSUE(need); wait_v<0>();
                        read_flags(need);
                        if (PER_PEER ? ((unsigned)(okm >> sh) & 255u) == 255u : prog >= need) break;
                        if (timed_out(tmo, call, t0, polls)) { good = false; prog = NCHUNK; break; }
                        __builtin_amdgcn_s_sleep(4);
                    }
                    if (half == 0) STAMP(48 + need - 1);
                }
            }
            pend = need - min(prog, need);
        };
        unsigned pc = 0;
        int i = 0;
#pragma unroll 1
        for (; i < 5 + mtile; ++i) {
            {
                const bool mine = i < 5;
                xb = mine ? (const char*)a.x : (const char*)wsl;
                dx = mine ? 7168u : 0u;
                const unsigned dd = mine ? 896u : 0u;
                v0 = vs[0] + (unsigned)(lane >> 3) * dd; v1 = vs[1] + (unsigned)(lane >> 3) * dd;
            }
            const unsigned ta = __builtin_amdgcn_readlane(tabA, i), w3 = __builtin_amdgcn_readlane(tabW, i), w4 = __builtin_amdgcn_readlane(tabW, i + 1);
            if ((i ^ mtile) & 1) {
                wait_v<N_LIGHT>(); BAR();
                pc = w3 & 15u;
                if (pc != 0u) POLL_ISSUE(pc);
                issueA(ta & ~127u);
                issueW(w3 & ~255u, wstage(i + 3), I0{}, I8{});
                issueW(w4 & ~255u, wstage(i + 4), I0{}, IS{});
            } else {
                wait_v<N_HEAVY>(); BAR();
                issueW(w3 & ~255u, wstage(i + 3), IS{}, I8{});
                if (pc != 0u) read_flags(pc);
                await(ta & 15u, ta);
                issueA(ta & ~127u);
            }
        }
        unsigned aH, aL, wH, wE, laH, laL, lbH, lbE, nH, nL, rareH, rareL;
        auto plan = [&](int h) {
            const unsigned t0 = __builtin_amdgcn_readlane(tabA, h & 63), t1 = __builtin_amdgcn_readlane(tabA, (h + 1) & 63);
            const unsigned u0 = __builtin_amdgcn_readlane(tabW, h & 63), u1 = __builtin_amdgcn_readlane(tabW, (h + 1) & 63);
            aH = t0 & ~127u; nH = t0 & 15u; aL = t1 & ~127u; nL = t1 & 15u; wH = u0 & ~255u; pc = u0 & 15u; wE = u1 & ~255u;
            laH = la; laL = next_la(la); la = next_la(laL);
            lbH = wstage(h + 3); lbE = wstage(h + 4);
            rareH = pc | (t0 & 16u); rareL = pc | (t1 & 16u);
            asm volatile("" : "+s"(aH), "+s"(aL), "+s"(wH), "+s"(wE), "+s"(laH), "+s"(laL), "+s"(lbH), "+s"(lbE), "+s"(rareH), "+s"(rareL));
        };
        plan(i);
#pragma unroll 1
        for (int n = 0; n < 29; ++n) {
            wait_v<N_LIGHT>(); BAR();
            if (__builtin_expect((rareH | pend) != 0u, 0)) { await(nH, aH); if (pc != 0u) POLL_ISSUE(pc); }
            issueAs(aH, laH);
            issueW(wH, lbH, I0{}, I8{});
            issueW(wE, lbE, I0{}, IS{});
            wait_v<N_HEAVY>(); BAR();
            issueW(wE, lbE, IS{}, I8{});
            if (__builtin_expect((rareL | pend) != 0u, 0)) { if (pc != 0u) read_flags(pc); await(nL, aL); }
            issueAs(aL, laL);
            i += 2;
            plan(i);
        }
        if (mtile == 0) BAR();
        if (lane == 0) red[4 + half] = good ? 0u : 0xffffffffu;
        u64 gdst = 0;
        if (half == 1 && wg == COLLECTOR) {
            char* pw = a.ws[0];
#pragma unroll
            for (int r = 1; r < NR; ++r) pw = ((lane >> 3) == r) ? a.ws[r] : pw;
            gdst = (u64)(pw + WS_FIN + (lane & 7) * 128 + rank * 8);
            asm volatile("" : "+v"(gdst));
        }
        FENCE(); WAIT_L0(); BAR(); FENCE();
        if (half == 0) {
            const unsigned wgmax = wave_umax((lane >= 4 && lane < 24 && (lane & ~1) != 6) ? red[lane] : 0u);
            if (lane == 0) {
                __hip_atomic_store((u64*)(wsl + WS_GRAN) + wg, ((u64)call << 32) | wgmax, __ATOMIC_RELAXED, __HIP_MEMORY_SCOPE_AGENT);
                __hip_atomic_store(seqp, call, __ATOMIC_RELAXED, __HIP_MEMORY_SCOPE_AGENT);
            }
            STAMP(60);
        } else if (wg == COLLECTOR) {
            const unsigned wgmax = wave_umax((lane >= 4 && lane < 24 && (lane & ~1) != 6) ? red[lane] : 0u);
            const u64 t0 = __builtin_amdgcn_s_memrealtime();
            unsigned polls = 0, gm;
            for (;;) {
                FENCE();
                const u32x4 g0 = __builtin_amdgcn_raw_buffer_load_b128(ss, WS_GRAN + lane * 16, 0, 16);
                const u32x4 g1 = __builtin_amdgcn_raw_buffer_load_b128(ss, WS_GRAN + 1024 + lane * 16, 0, 16);
                const bool self = lane == 63;
                gm = max(max(g0.x, g0.z), max(g1.x, self ? wgmax : g1.z));
                if (__all(g0.y == call && g0.w == call && g1.y == call && (self || g1.w == call))) break;
                if (timed_out(tmo, call, t0, polls)) { gm = 0xffffffffu; break; }
            }
            gm = wave_umax(gm);
            __hip_atomic_store((__attribute__((address_space(1))) u64*)gdst, ((u64)call << 32) | gm, __ATOMIC_RELAXED, __HIP_MEMORY_SCOPE_SYSTEM);
        }
        if (half == 0) { STAMP_FLUSH(16, 24); STAMP_FLUSH(40, 64); }
        return;
    }
    if (wid == 6) {
        PRIO(3);
        STAMP(32);
        const __amdgpu_buffer_rsrc_t xs = __builtin_amdgcn_make_buffer_rsrc((void*)a.x, 0, X_BYTES, SRD3);
        const unsigned po = (unsigned)(idx * 16 + (lane & 15)) * 1024u + (unsigned)(3 + (lane >> 4)) * 128u;
#define OWN_TOUCH() asm volatile("buffer_load_dword v240, %0, %1, %2 offen\n\tbuffer_load_dword v240, %3, %1, %2 offen" \
                         :: "v"(po), "s"(xs), "s"((unsigned)rank * SLOT_BYTES), "v"(po | 896u) : "v240", "memory")
        int nb = 0;
        const int s4 = (int)(((unsigned)(size_t)a.w >> 8) + 3u) & 3, u4 = s4 + (idx >> 1);
        const int fid = ((idx >> 1) - (((u4 + 3) >> 2) - ((s4 + 3) >> 2))) * 2 + mtile;
        if ((u4 & 3) != 0 && fid < 7 * NPART / 8) {
            const int id = fid * 8 + xcd, j = id % 7 + 1, part = id / 7, dst = (rank + j) & 7;
            char* wsd = a.ws[0];
#pragma unroll
            for (int r = 1; r < NR; ++r) wsd = (dst == r) ? a.ws[r] : wsd;
            const __amdgpu_buffer_rsrc_t ds = __builtin_amdgcn_make_buffer_rsrc((void*)wsd, 0, WS_GRAN, SRD3);
            const unsigned piece = part < 16 ? 3u * part : 2u * part + 16u, none = part < 16 ? 0u : OOB;
            const unsigned vsrc = (piece * 8u + (unsigned)(lane >> 3)) * 1024u + (unsigned)(lane & 7) * 16u;
            const unsigned vdst = piece * 1024u + (unsigned)lane * 16u;
            const unsigned fdst = WS_XFLAG + (unsigned)rank * 128u + (unsigned)part * 4u;
            const unsigned vsrc2 = vsrc + 8192u, vsrc3 = vsrc + 16384u + none, vdst3 = vdst + 2048u + none;
            const unsigned ssrc = (unsigned)dst * SLOT_BYTES, sdst = (unsigned)rank * SLOT_BYTES;
            auto xload = [&](auto cc) { x_load<decltype(cc)::value>(vsrc, vsrc2, vsrc3, xs, ssrc); };
            auto xstore = [&](auto cc) { x_store<decltype(cc)::value>(vdst, vdst3, ds, sdst + (unsigned)decltype(cc)::value * 65536u); };
            xload(IC<0>{}); xload(IC<1>{});
            OWN_TOUCH();
            const unsigned call = load_seq(seqp) + 1u;
            auto quiet = [&](int c, int from, int spins) {
                int spin = 0;
#pragma unroll 1
                for (;;) {
                    const bool last = nb >= 7 + 7 * c;
                    if (nb >= from && (c == 0 || last || ((nb ^ mtile) & 1) == 0) && vm_outstanding() == 0u) break;
                    if (!last && (nb < from || ++spin > spins)) { BAR(); ++nb; spin = 0; }
                    else __builtin_amdgcn_s_sleep(1);
                }
                wait_v<0>();
            };
#pragma unroll 1
            for (int c = 0; c < NCHUNK; ++c) {
                const int slot = c == 0 ? 1 + (part & 1) : (c == 1 ? 2 : PUSH_PACE * c) + (part & 2) + mtile;
                if (c <= 2) quiet(c, slot, 0);
                else {
#pragma unroll 1
                    while (nb < slot) { BAR(); ++nb; }
                }
                switch (c) {
                    case 0: xstore(IC<0>{}); break;
                    case 1: xstore(IC<1>{}); xload(IC<2>{}); xload(IC<3>{}); xload(IC<4>{}); xload(IC<5>{}); xload(IC<6>{}); xload(IC<7>{}); break;
                    case 2: xstore(IC<2>{}); break;
                    case 3: xstore(IC<3>{}); break;
                    case 4: xstore(IC<4>{}); break;
                    case 5: xstore(IC<5>{}); break;
                    case 6: xstore(IC<6>{}); break;
                    default: xstore(IC<7>{}); break;
                }
                quiet(c, 0, c == 0 ? 3 : 0);
                FENCE();
                if (lane == 0) __builtin_amdgcn_raw_buffer_store_b32(call, ds, fdst, c * 1024, SYS);
                FENCE();
                if (c == 0) STAMP(34);
            }
        } else OWN_TOUCH();
        STAMP(35);
        STAMP_FLUSH(32, 40);
        while (vm_outstanding() != 0u) BAR();
        return;
    }

    if (wid == 7) {
        const u64 t0 = __builtin_amdgcn_s_memrealtime();
        BAR();
#pragma unroll 1
        for (int i = 0; i < PACE_TO; ++i) {
            const unsigned lim = (unsigned)(PACE_T0 + (i + 1) * PACE_NS);
            while ((unsigned)(__builtin_amdgcn_s_memrealtime() - t0) * 10u < lim) __builtin_amdgcn_s_sleep(1);
            BAR();
        }
        return;
    }

    if (wid == 0) STAMP(0);
    const int wm = wid & 1, wn = wid >> 1;
    const int fr = lane & 15, g = lane >> 4, q = fr >> 2, p = lane & 3;
    unsigned aoff[2], boff[4];
#pragma unroll
    for (int kk = 0; kk < 2; ++kk) aoff[kk] = ldsbase + (unsigned)((wm * 128 + fr) * 128 + (((kk * 4 + g) ^ (fr >> 1)) << 4));
#pragma unroll
    for (int f = 0; f < 4; ++f)
        boff[f] = ldsbase + LDS_B0 + (unsigned)((8 * g + q) * 256 + (((wn * 8 + 2 * p + (f >> 1)) ^ (q | ((g & 1) << 3))) << 4) + ((((f & 1) ^ (p & 1))) << 3));
    asm volatile("" : "+v"(boff[0]), "+v"(boff[1]), "+v"(boff[2]), "+v"(boff[3]), "+v"(aoff[0]), "+v"(aoff[1]));
    const unsigned call = load_seq(seqp) + 1u;
    f32x4 acc[8][4];
#pragma unroll
    for (int m = 0; m < 8; ++m)
#pragma unroll
        for (int f = 0; f < 4; ++f) acc[m][f] = (f32x4){0.f, 0.f, 0.f, 0.f};
    bf16x8 x0[8], w0[4], x1[8], w1[4];
    bf16x4 tl[4];
    unsigned bcur[4], acur = aoff[0];
#pragma unroll
    for (int f = 0; f < 4; ++f) bcur[f] = boff[f];
#define RD_WL(f, kk) do { tl[f] = __builtin_amdgcn_ds_read_tr16_b64_v4bf16((LAS bf16x4*)(bcur[f] + (kk) * 8192)); } while (0)
#define RD_WH(dst, f, kk) do { const bf16x4 hi_ = __builtin_amdgcn_ds_read_tr16_b64_v4bf16((LAS bf16x4*)(bcur[f] + (kk) * 8192 + 1024)); \
        dst = __builtin_shufflevector(tl[f], hi_, 0, 1, 2, 3, 4, 5, 6, 7); } while (0)
#define RD_XA(dst, m) do { dst = *(const LAS bf16x8*)(acur + (m) * 2048); } while (0)
    FENCE(); BAR(); FENCE();
    if (wid == 0) { STAMP(2); CSTAMP(30); }
#pragma unroll
    for (int i = 0; i < 4; ++i) { RD_WL(i, 0); RD_WH(w0[i], i, 0); }
#pragma unroll
    for (int i = 0; i < 8; ++i) RD_XA(x0[i], i);
    acur = aoff[1];
    auto step = [&](bf16x8* cw, bf16x8* cx, bf16x8* nw, bf16x8* nx, auto kkc, auto side) {
        constexpr int kk = decltype(kkc)::value;
#pragma unroll
        for (int s = 0; s < 32; ++s) {
            const int m = s >> 2, f = (m & 1) ? 3 - (s & 3) : (s & 3);
            acc[m][f] = MFMA(cw[f], cx[m], acc[m][f]);
            SCHED();
            if (s < 24 && s % 3 != 1) {
                const int r = (s * 2) / 3;
                if (r < 8) { if (r & 1) RD_WH(nw[r >> 1], r >> 1, kk); else RD_WL(r >> 1, kk); }
                else RD_XA(nx[r - 8], r - 8);
            }
            if (s >= 24 && s < 31) side(s - 24);
            SCHED();
        }
    };
    unsigned oa = A_STAGE, ob = B_STAGE;
    auto stepA = [&]() {
        step(w0, x0, w1, x1, IC<1>{}, [&](int h) {
            if (h < 4) bcur[h] = boff[h] + ob;
            else if (h == 4) acur = aoff[0] + oa;
            else if (h == 5) { ob = (ob + B_STAGE) & (RB * B_STAGE - 1); asm volatile("" : "+s"(ob)); } });
        WAIT_L0(); SCHED();
    };
    auto stepB = [&]() {
        BAR(); SCHED();
        step(w1, x1, w0, x0, IC<0>{}, [&](int h) {
            if (h == 0) acur = aoff[1] + oa;
            else if (h == 1) { oa += A_STAGE; asm volatile("" : "+s"(oa)); }
            else if (h == 2) { oa = oa == RA * A_STAGE ? 0u : oa; asm volatile("" : "+s"(oa)); }
            else if (h == 6) WAIT_L0(); });
    };
    WAIT_L0();
    stepA();
#pragma unroll 1
    for (int t = 0; t < NT - 1; ++t) { stepB(); stepA(); }

    int mx0 = 0, mx1 = 0;
    SCHED(); BAR(); SCHED();
#pragma unroll
    for (int s = 0; s < 36; ++s) {
        if (s < 32) { const int m = s >> 2, f = (m & 1) ? 3 - (s & 3) : (s & 3); acc[m][f] = MFMA(w1[f], x1[m], acc[m][f]); }
        SCHED();
        if (s >= 4) {
            const int m = (s - 4) >> 2, f = (m & 1) ? 3 - (s & 3) : (s & 3);
            mx0 = max(max(mx0, __float_as_int(acc[m][f][0])), __float_as_int(acc[m][f][1]));
            mx1 = max(max(mx1, __float_as_int(acc[m][f][2])), __float_as_int(acc[m][f][3]));
        }
        SCHED();
    }
    const unsigned wmax = row_umax((unsigned)max(mx0, mx1));
    if (wid == 0) { STAMP(11); CSTAMP(31); }
    if (fr == 15) red[8 + wid * 4 + g] = wmax;
    FENCE(); WAIT_L0(); BAR(); FENCE();
    const unsigned wgmax = wave_umax((lane >= 4 && lane < 24 && (lane & ~1) != 6) ? red[lane] : 0u);
    const bool odd = g & 1;
    const __amdgpu_buffer_rsrc_t outr = __builtin_amdgcn_make_buffer_rsrc(a.out, 0, MLOC * NDIM, SRD3);
    const unsigned ovo = (unsigned)(mtile * BM + wm * 128 + fr) * NDIM + (unsigned)(ntile * BN + wn * 64 + g * 16);
    unsigned used = 0xfffffffeu;
    const char* finp = wsl + WS_FIN + xcd * 128;
    const u64 t0 = __builtin_amdgcn_s_memrealtime();
    unsigned polls = 0;
    for (;;) {
        unsigned gm = wgmax, miss = 0;
        {
            const u64 fv = __hip_atomic_load((const u64*)finp + (lane & 7), __ATOMIC_RELAXED, __HIP_MEMORY_SCOPE_SYSTEM);
            const bool ok = (unsigned)(fv >> 32) == call;
            gm = max(gm, (unsigned)__builtin_amdgcn_readlane(row_umax(ok ? (unsigned)fv : 0u), 15));
            miss = (unsigned)__popcll(__ballot(!ok)) >> 3;
        }
        if (miss != 0u && timed_out(tmo, call, t0, polls)) gm = 0xffffffffu;
        if (gm == 0xffffffffu) {
            const u32x4 o = {0x7f7f7f7fu, 0x7f7f7f7fu, 0x7f7f7f7fu, 0x7f7f7f7fu};
#pragma unroll
            for (int m = 0; m < 8; ++m) __builtin_amdgcn_raw_buffer_store_b128(o, outr, ovo + m * 16 * NDIM, 0, OUT_AUX);
            break;
        }
        if (miss == 0u && wid == 0) STAMP(13);
        if (gm != used && miss <= SPEC_THR) {
            used = gm;
            const float amax = __uint_as_float(gm);
            const float inv = amax > 0.f ? 448.0f / amax : 0.f;
            const f32x2 sc = {inv * (1.0f / 512.0f), inv * (1.0f / 512.0f)};
#pragma unroll
            for (int m = 0; m < 8; ++m) {
                unsigned d[4];
#pragma unroll
                for (int f = 0; f < 4; ++f) {
                    f32x2 lo = {acc[m][f][0], acc[m][f][1]}, hi = {acc[m][f][2], acc[m][f][3]};
                    asm("v_pk_mul_f32 %0, %1, %2 clamp" : "=v"(lo) : "v"(lo), "v"(sc));
                    asm("v_pk_mul_f32 %0, %1, %2 clamp" : "=v"(hi) : "v"(hi), "v"(sc));
                    s16x2 w = __builtin_bit_cast(s16x2, lo.x);
                    w = __builtin_amdgcn_cvt_scalef32_pk_fp8_f32(w, lo.x, lo.y, 1.0f / 512.0f, false);
                    w = __builtin_amdgcn_cvt_scalef32_pk_fp8_f32(w, hi.x, hi.y, 1.0f / 512.0f, true);
                    d[f] = __builtin_bit_cast(unsigned, w);
                }
                u32x4 o;
                o.x = odd ? d[1] : d[0]; o.y = odd ? d[0] : d[1]; o.z = odd ? d[3] : d[2]; o.w = odd ? d[2] : d[3];
                __builtin_amdgcn_raw_buffer_store_b128(o, outr, ovo + m * 16 * NDIM, 0, OUT_AUX);
            }
        }
        if (miss == 0u) break;
        __builtin_amdgcn_s_sleep(1);
    }
}

}

extern "C" void kernel_launch(int rank, int nranks, ncclComm_t comm,
                              hipStream_t stream,
                              void* const* d_in, const int* in_sizes, int n_in,
                              void* d_out, int out_size,
                              const int* in_dtypes, int out_dtype,
                              void* const* d_out_all,
                              void* const* d_ws_all, size_t ws_bytes) {
    using namespace a2ag;
    if (nranks != NR || (unsigned)rank >= (unsigned)NR || n_in < 2 || ws_bytes < WS_NEED
        || in_sizes[0] != KDIM * KSL || in_sizes[1] != KDIM * NDIM || out_size != MLOC * NDIM) {
        fprintf(stderr, "kernel_launch: unexpected problem geometry\n");
        return;
    }
    HIP_CHECK(hipFuncSetAttribute((const void*)a2a_gemm_kernel, hipFuncAttributeMaxDynamicSharedMemorySize, LDS_BYTES));
    Args a;
    a.x = d_in[0]; a.w = d_in[1]; a.out = d_out; a.rank = rank; a.pad = 0;
    a.wsl = (char*)d_ws_all[rank];
    for (int r = 0; r < NR; ++r) a.ws[r] = (char*)d_ws_all[r];
    hipLaunchKernelGGL(a2a_gemm_kernel, dim3(NWG), dim3(NTHREADS), LDS_BYTES, stream, a);
    HIP_CHECK(hipGetLastError());
}
```

```cpp
#include <hip/hip_runtime.h>
#include <rccl/rccl.h>
#include <cstdio>
#include <cstdint>
#include <type_traits>

#define HIP_CHECK(expr) do { hipError_t _e = (expr); if (_e != hipSuccess) { \
    fprintf(stderr, "HIP error %s at %s:%d\n", hipGetErrorString(_e), __FILE__, __LINE__); \
    return; } } while (0)

#define PUSH_PACE 4
#define SPEC_THR 2
#define OUT_AUX 17
#define WSPLIT 6
#define PER_PEER 1
#define PACE_NS 820
#define PACE_T0 1700
#define PACE_TO 36

#pragma clang diagnostic ignored "-Wint-to-pointer-cast"

namespace a2ag {

typedef __bf16 bf16x4 __attribute__((ext_vector_type(4)));
typedef __bf16 bf16x8 __attribute__((ext_vector_type(8)));
typedef float f32x4 __attribute__((ext_vector_type(4)));
typedef float f32x2 __attribute__((ext_vector_type(2)));
typedef short s16x2 __attribute__((ext_vector_type(2)));
typedef unsigned u32x4 __attribute__((ext_vector_type(4)));
typedef unsigned long long u64;
#define LAS __attribute__((address_space(3)))
#define DI __device__ __forceinline__
template <int V> using IC = std::integral_constant<int, V>;

constexpr int NR = 8;
constexpr int MLOC = 512, KDIM = 4096, NDIM = 16384, KSL = 512;
constexpr int BM = 256, BN = 128, BK = 64;
constexpr int NT = KDIM / BK;
constexpr int NWG = (MLOC / BM) * (NDIM / BN);
constexpr int NTHREADS = 512;
constexpr unsigned A_STAGE = BM * BK * 2, B_STAGE = BK * BN * 2;
constexpr int RA = 3, RB = 4;
constexpr unsigned LDS_B0 = RA * A_STAGE;
constexpr unsigned LDS_BYTES = LDS_B0 + RB * B_STAGE;
constexpr unsigned SLOT_BYTES = MLOC * KSL * 2;
constexpr unsigned X_BYTES = NR * SLOT_BYTES;
constexpr unsigned W_BYTES = (unsigned)KDIM * NDIM * 2;
constexpr unsigned W_TILE = BK * NDIM * 2;
constexpr int NCHUNK = 8, NPART = 24;
constexpr unsigned WS_XFLAG = X_BYTES;
constexpr unsigned WS_GRAN = WS_XFLAG + NCHUNK * 1024;
constexpr unsigned WS_FIN = WS_GRAN + NWG * 8;
constexpr unsigned WS_SEQ = WS_FIN + 8 * 128;
constexpr unsigned WS_TMO = WS_SEQ + NWG * 64;
constexpr unsigned WS_DBG = WS_TMO + 4096;
constexpr unsigned WS_NEED = WS_DBG + NWG * 64 * 12;
constexpr u64 DEADLINE_TICKS = 40000000ull;
constexpr unsigned OOB = 0x40000000u;
constexpr unsigned SRD3 = 0x00020000u;
constexpr int COLLECTOR = NWG - 1;
constexpr int SYS = 17;

struct Args {
    const void* x; const void* w;
    char* wsl;
    int rank, pad;
    void* out;
    char* ws[NR];
};

DI void set_m0(unsigned v) { asm volatile("s_mov_b32 m0, %0\n\ts_nop 0" :: "s"(v) : "memory"); }
template <int OFF> DI void dma_x(__amdgpu_buffer_rsrc_t srd, unsigned voff, unsigned soff) {
    asm volatile("buffer_load_dwordx4 %0, %1, %2 offen offset:%3 sc1 lds" :: "v"(voff), "s"(srd), "s"(soff), "n"(OFF) : "memory");
}
template <int OFF> DI void dma_w(__amdgpu_buffer_rsrc_t srd, unsigned voff, unsigned soff) {
    asm volatile("buffer_load_dwordx4 %0, %1, %2 offen offset:%3 lds" :: "v"(voff), "s"(srd), "s"(soff), "n"(OFF) : "memory");
}
#define XCLOB "v144", "v145", "v146", "v147", "v148", "v149", "v150", "v151", "v152", "v153", "v154", "v155", "v156", "v157", "v158", "v159", "v160", "v161", "v162", "v163", "v164", "v165", "v166", "v167", "v168", "v169", "v170", "v171", "v172", "v173", "v174", "v175", "v176", "v177", "v178", "v179", "v180", "v181", "v182", "v183", "v184", "v185", "v186", "v187", "v188", "v189", "v190", "v191", "v192", "v193", "v194", "v195", "v196", "v197", "v198", "v199", "v200", "v201", "v202", "v203", "v204", "v205", "v206", "v207", "v208", "v209", "v210", "v211", "v212", "v213", "v214", "v215", "v216", "v217", "v218", "v219", "v220", "v221", "v222", "v223", "v224", "v225", "v226", "v227", "v228", "v229", "v230", "v231", "v232", "v233", "v234", "v235", "v236", "v237", "v238", "v239"
template <int C> DI void x_load(unsigned v0, unsigned v1, unsigned v2, __amdgpu_buffer_rsrc_t srd, unsigned soff) {
    asm volatile("buffer_load_dwordx4 v[%c5:%c6], %0, %3, %4 offen offset:%c11\n\tbuffer_load_dwordx4 v[%c7:%c8], %1, %3, %4 offen offset:%c11\n\t"
                 "buffer_load_dwordx4 v[%c9:%c10], %2, %3, %4 offen offset:%c11"
                 :: "v"(v0), "v"(v1), "v"(v2), "s"(srd), "s"(soff), "n"(144 + 12 * C), "n"(147 + 12 * C), "n"(148 + 12 * C), "n"(151 + 12 * C),
                    "n"(152 + 12 * C), "n"(155 + 12 * C), "n"(C * 128) : "memory", XCLOB);
}
template <int C> DI void x_store(unsigned voff, unsigned voff2, __amdgpu_buffer_rsrc_t srd, unsigned soff) {
    asm volatile("buffer_store_dwordx4 v[%c4:%c5], %0, %2, %3 offen sc0 sc1\n\tbuffer_store_dwordx4 v[%c6:%c7], %0, %2, %3 offen offset:1024 sc0 sc1\n\t"
                 "buffer_store_dwordx4 v[%c8:%c9], %1, %2, %3 offen sc0 sc1"
                 :: "v"(voff), "v"(voff2), "s"(srd), "s"(soff), "n"(144 + 12 * C), "n"(147 + 12 * C), "n"(148 + 12 * C), "n"(151 + 12 * C),
                    "n"(152 + 12 * C), "n"(155 + 12 * C) : "memory");
}
template <int N> DI void wait_v() { asm volatile("s_waitcnt vmcnt(%0)" :: "n"(N) : "memory"); }
#define WAIT_L0() __builtin_amdgcn_s_waitcnt(0xC07F)
#define FENCE() asm volatile("" ::: "memory")
#define BAR() __builtin_amdgcn_s_barrier()
#define SCHED() __builtin_amdgcn_sched_barrier(0)
#define PRIO(n) __builtin_amdgcn_s_setprio(n)
#define MFMA(a, b, c) __builtin_amdgcn_mfma_f32_16x16x32_bf16(a, b, c, 0, 0, 0)

#define STAMP(i) do {} while (0)
#define CSTAMP(i) do {} while (0)
#define STAMP_DECL
#define STAMP_FLUSH(lo, hi) do {} while (0)

DI unsigned ktile_of(int t, int rank) {
    const int i = t - 8, c = (i * 37) >> 8, j = i - c * 7;
    return t < 8 ? (unsigned)(rank * 8 + t) : (unsigned)((((rank + 1 + j) & 7) << 3) + c);
}

DI bool timed_out(unsigned* tmo, unsigned call, u64 t0, unsigned& polls) {
    if ((++polls & 31u) != 0u) return false;
    if (__builtin_amdgcn_readfirstlane(__hip_atomic_load(tmo, __ATOMIC_RELAXED, __HIP_MEMORY_SCOPE_AGENT)) == call
        || __builtin_amdgcn_s_memrealtime() - t0 > DEADLINE_TICKS) {
        __hip_atomic_store(tmo, call, __ATOMIC_RELAXED, __HIP_MEMORY_SCOPE_AGENT);
        return true;
    }
    return false;
}
DI unsigned load_seq(const unsigned* p) {
    unsigned v;
    asm volatile("s_load_dword %0, %1, 0x0 glc\n\ts_waitcnt lgkmcnt(0)" : "=s"(v) : "s"(p) : "memory");
    return v;
}
DI unsigned vm_outstanding() {
    const unsigned s = __builtin_amdgcn_s_getreg(7 | (0 << 6) | (31 << 11));
    return (s & 15u) | ((s >> 18) & 48u);
}
DI unsigned row_umax(unsigned v) {
    v = max(v, (unsigned)__builtin_amdgcn_update_dpp(0, (int)v, 0x111, 0xf, 0xf, false));
    v = max(v, (unsigned)__builtin_amdgcn_update_dpp(0, (int)v, 0x112, 0xf, 0xf, false));
    v = max(v, (unsigned)__builtin_amdgcn_update_dpp(0, (int)v, 0x114, 0xf, 0xf, false));
    v = max(v, (unsigned)__builtin_amdgcn_update_dpp(0, (int)v, 0x118, 0xf, 0xf, false));
    return v;
}
DI unsigned wave_umax(unsigned v) {
    v = row_umax(v);
    const unsigned a = __builtin_amdgcn_readlane(v, 15), b = __builtin_amdgcn_readlane(v, 31);
    const unsigned c = __builtin_amdgcn_readlane(v, 47), d = __builtin_amdgcn_readlane(v, 63);
    return max(max(a, b), max(c, d));
}

__global__ __launch_bounds__(NTHREADS) void a2a_gemm_kernel(Args a) {
    extern __shared__ __attribute__((aligned(16))) char smem[];
    LAS char* lds = (LAS char*)smem;
    const int tid = threadIdx.x, lane = tid & 63;
    const int wid = __builtin_amdgcn_readfirstlane(tid >> 6);
    const int wg = blockIdx.x, rank = a.rank;
    const int xcd = wg & 7, idx = wg >> 3;
    const int ntile = xcd * 16 + (idx >> 1), mtile = idx & 1;
    const unsigned ldsbase = (unsigned)(size_t)lds;
    LAS unsigned* red = (LAS unsigned*)(lds + LDS_B0 + 3 * B_STAGE);
    char* wsl = a.wsl;
    asm volatile("" :: "s"(a.x), "s"(a.w), "s"(wsl), "s"(rank));
    unsigned* seqp = (unsigned*)(wsl + WS_SEQ + wg * 64);
    unsigned* tmo = (unsigned*)(wsl + WS_TMO);
    STAMP_DECL;

    if (wid == 4 || wid == 5) {
        PRIO(3);
        const int half = wid - 4;
        const __amdgpu_buffer_rsrc_t ss = __builtin_amdgcn_make_buffer_rsrc((void*)wsl, 0, WS_TMO, SRD3);
        const __amdgpu_buffer_rsrc_t wr = __builtin_amdgcn_make_buffer_rsrc((void*)a.w, 0, W_BYTES, SRD3);
        unsigned vs[2], vw[2];
#pragma unroll
        for (int e = 0; e < 2; ++e) {
            vs[e] = (unsigned)(lane >> 3) * 128u + (unsigned)(((lane & 7) ^ (4 * e + (lane >> 4))) << 4);
            vw[e] = (unsigned)(lane >> 4) * 32768u + (unsigned)(((lane & 15) ^ ((e << 3) | (lane >> 4))) << 4);
        }
        const unsigned row0 = (unsigned)(mtile * BM + half * 128);
        const unsigned lA = ldsbase + (unsigned)half * 16384u, lB = ldsbase + LDS_B0 + (unsigned)half * 8192u;
        const unsigned soB = (unsigned)ntile * (BN * 2u) + (unsigned)half * (32u * 32768u);
        const unsigned soOwn = (unsigned)rank * SLOT_BYTES + row0 * 1024u;
        const char* xb = (const char*)a.x;
        unsigned dx = 7168u, v0 = vs[0] + (unsigned)(lane >> 3) * 896u, v1 = vs[1] + (unsigned)(lane >> 3) * 896u;
        unsigned la = lA;
        auto next_la = [&](unsigned l) -> unsigned { return l == lA + (RA - 1) * A_STAGE ? lA : l + A_STAGE; };
        auto issueA = [&](unsigned so) {
            const __amdgpu_buffer_rsrc_t srd = __builtin_amdgcn_make_buffer_rsrc((void*)xb, 0, X_BYTES, SRD3);
#pragma unroll
            for (int p = 0; p < 16; p += 4) {
                const unsigned s4 = so + p * (dx + 1024u);
                set_m0(la + p * 1024u);
                dma_x<0>(srd, v0, s4); dma_x<1024>(srd, v1, s4 + dx); dma_x<2048>(srd, v0, s4 + 2u * dx); dma_x<3072>(srd, v1, s4 + 3u * dx);
            }
            la = next_la(la);
        };
        auto issueAs = [&](unsigned so, unsigned l) {
#pragma unroll
            for (int p = 0; p < 16; p += 4) {
                set_m0(l + p * 1024u);
                dma_x<0>(ss, vs[0], so + p * 1024u); dma_x<1024>(ss, vs[1], so + p * 1024u); dma_x<2048>(ss, vs[0], so + p * 1024u); dma_x<3072>(ss, vs[1], so + p * 1024u);
            }
        };
        auto wstage = [&](int t) -> unsigned { return lB + ((unsigned)t & (RB - 1)) * B_STAGE; };
        auto issueW = [&](unsigned so, unsigned lb, auto lo, auto hi) {
            constexpr int LO = decltype(lo)::value, HI = decltype(hi)::value;
            if constexpr (LO == 0) {
                set_m0(lb);
                dma_w<0>(wr, vw[0], so); dma_w<1024>(wr, vw[0], so + 130048u); dma_w<2048>(wr, vw[1], so + 260096u); dma_w<3072>(wr, vw[1], so + 390144u);
            }
            if constexpr (HI > 4 && LO < 8) set_m0(lb + 4096u);
            if constexpr (LO <= 4 && HI >= 6) { dma_w<0>(wr, vw[0], so + 524288u); dma_w<1024>(wr, vw[0], so + 654336u); }
            if constexpr (LO <= 6 && HI == 8) { dma_w<2048>(wr, vw[1], so + 784384u); dma_w<3072>(wr, vw[1], so + 914432u); }
        };
        using I0 = IC<0>; using IS = IC<WSPLIT>; using I8 = IC<8>;
        constexpr int N_HEAVY = 16 + 8 + WSPLIT, N_LIGHT = 16 + 8;
        const unsigned fvo = WS_XFLAG - 1024u + (unsigned)lane * 16u;
#define POLL_ISSUE(n) asm volatile("buffer_load_dwordx4 v[240:243], %0, %1, %2 offen sc0 sc1" :: "v"(fvo), "s"(ss), "s"((n) * 1024u) : "v240", "v241", "v242", "v243", "memory")
        const unsigned soW0 = (unsigned)rank * (8u * W_TILE) + soB;
        issueA(soOwn); issueW(soW0, wstage(0), I0{}, I8{});
        asm volatile("s_load_dword s96, %0, 0x0 glc" :: "s"(seqp) : "s96", "memory");
        issueW(soW0 + W_TILE, wstage(1), I0{}, I8{});
        issueA(soOwn + 128u);
        unsigned tabA, tabW;
        {
            const int t = lane + 3;
            const unsigned kt = ktile_of(t, rank);
            const unsigned first = (int)(kt >> 3) == ((rank + 1) & 7) ? 16u : 0u;
            tabA = t < 8 ? soOwn + (unsigned)t * 128u : (t < NT ? kt * 65536u + row0 * 128u + first + (kt & 7u) + 1u : OOB);
            const unsigned n0 = t < 8 ? 0u : (kt & 7u) + 1u, n2 = t + 2 < 8 ? 0u : (ktile_of(t + 2, rank) & 7u) + 1u;
            tabW = t < NT ? kt * W_TILE + soB + ((t + 2 < NT && n2 > n0) ? n2 : 0u) : OOB;
        }
        unsigned call;
        asm volatile("s_waitcnt lgkmcnt(0)\n\ts_add_u32 %0, s96, 1" : "=s"(call) :: "scc", "memory");
        wait_v<24>(); BAR();
        issueW(soW0 + 2u * W_TILE, wstage(2), I0{}, I8{}); issueA(soOwn + 256u);
        if (mtile == 0) issueW(soW0 + 3u * W_TILE, wstage(3), I0{}, IS{});
        const bool own = (lane >> 3) == rank || (lane & 7) >= NPART / 4;
        bool good = true;
        unsigned prog = 0;
        unsigned pend = 0, okc = 0;
        u64 okm = 0;
        auto read_flags = [&](unsigned n) {
            u32x4 f;
            asm volatile("v_mov_b32 %0, v240\n\tv_mov_b32 %1, v241\n\tv_mov_b32 %2, v242\n\tv_mov_b32 %3, v243" : "=v"(f.x), "=v"(f.y), "=v"(f.z), "=v"(f.w) :: "memory");
            okm = __ballot(own || (int)((f.x - call) | (f.y - call) | (f.z - call) | (f.w - call)) >= 0);
            okc = n;
            if (okm == ~0ull) prog = n;
        };
        auto await = [&](unsigned need, unsigned so) {
            if (__builtin_expect(prog < need, 0)) {
                const unsigned sh = (so >> 19) * 8u;
                if (!(PER_PEER && okc >= need && ((unsigned)(okm >> sh) & 255u) == 255u)) {
                    if (half == 0) STAMP(40 + need - 1);
                    const u64 t0 = __builtin_amdgcn_s_memrealtime();
                    unsigned polls = 0;
                    for (;;) {
                        POLL_ISSUE(need); wait_v<0>();
                        read_flags(need);
                        if (PER_PEER ? ((unsigned)(okm >> sh) & 255u) == 255u : prog >= need) break;
                        if (timed_out(tmo, call, t0, polls)) { good = false; prog = NCHUNK; break; }
                        __builtin_amdgcn_s_sleep(4);
                    }
                    if (half == 0) STAMP(48 + need - 1);
                }
            }
            pend = need - min(prog, need);
        };
        unsigned pc = 0;
        int i = 0;
#pragma unroll 1
        for (; i < 5 + mtile; ++i) {
            {
                const bool mine = i < 5;
                xb = mine ? (const char*)a.x : (const char*)wsl;
                dx = mine ? 7168u : 0u;
                const unsigned dd = mine ? 896u : 0u;
                v0 = vs[0] + (unsigned)(lane >> 3) * dd; v1 = vs[1] + (unsigned)(lane >> 3) * dd;
            }
            const unsigned ta = __builtin_amdgcn_readlane(tabA, i), w3 = __builtin_amdgcn_readlane(tabW, i), w4 = __builtin_amdgcn_readlane(tabW, i + 1);
            if ((i ^ mtile) & 1) {
                wait_v<N_LIGHT>(); BAR();
                pc = w3 & 15u;
                if (pc != 0u) POLL_ISSUE(pc);
                issueA(ta & ~127u);
                issueW(w3 & ~255u, wstage(i + 3), I0{}, I8{});
                issueW(w4 & ~255u, wstage(i + 4), I0{}, IS{});
            } else {
                wait_v<N_HEAVY>(); BAR();
                issueW(w3 & ~255u, wstage(i + 3), IS{}, I8{});
                if (pc != 0u) read_flags(pc);
                await(ta & 15u, ta);
                issueA(ta & ~127u);
            }
        }
        unsigned aH, aL, wH, wE, laH, laL, lbH, lbE, nH, nL, rareH, rareL;
        auto plan = [&](int h) {
            const unsigned t0 = __builtin_amdgcn_readlane(tabA, h & 63), t1 = __builtin_amdgcn_readlane(tabA, (h + 1) & 63);
            const unsigned u0 = __builtin_amdgcn_readlane(tabW, h & 63), u1 = __builtin_amdgcn_readlane(tabW, (h + 1) & 63);
            aH = t0 & ~127u; nH = t0 & 15u; aL = t1 & ~127u; nL = t1 & 15u; wH = u0 & ~255u; pc = u0 & 15u; wE = u1 & ~255u;
            laH = la; laL = next_la(la); la = next_la(laL);
            lbH = wstage(h + 3); lbE = wstage(h + 4);
            rareH = pc | (t0 & 16u); rareL = pc | (t1 & 16u);
            asm volatile("" : "+s"(aH), "+s"(aL), "+s"(wH), "+s"(wE), "+s"(laH), "+s"(laL), "+s"(lbH), "+s"(lbE), "+s"(rareH), "+s"(rareL));
        };
        plan(i);
#pragma unroll 1
        for (int n = 0; n < 29; ++n) {
            wait_v<N_LIGHT>(); BAR();
            if (__builtin_expect((rareH | pend) != 0u, 0)) { await(nH, aH); if (pc != 0u) POLL_ISSUE(pc); }
            issueAs(aH, laH);
            issueW(wH, lbH, I0{}, I8{});
            issueW(wE, lbE, I0{}, IS{});
            wait_v<N_HEAVY>(); BAR();
            issueW(wE, lbE, IS{}, I8{});
            if (__builtin_expect((rareL | pend) != 0u, 0)) { if (pc != 0u) read_flags(pc); await(nL, aL); }
            issueAs(aL, laL);
            i += 2;
            plan(i);
        }
        if (mtile == 0) BAR();
        if (lane == 0) red[4 + half] = good ? 0u : 0xffffffffu;
        u64 gdst = 0;
        if (half == 1 && wg == COLLECTOR) {
            char* pw = a.ws[0];
#pragma unroll
            for (int r = 1; r < NR; ++r) pw = ((lane >> 3) == r) ? a.ws[r] : pw;
            gdst = (u64)(pw + WS_FIN + (lane & 7) * 128 + rank * 8);
            asm volatile("" : "+v"(gdst));
        }
        FENCE(); WAIT_L0(); BAR(); FENCE();
        if (half == 0) {
            const unsigned wgmax = wave_umax((lane >= 4 && lane < 24 && (lane & ~1) != 6) ? red[lane] : 0u);
            if (lane == 0) {
                __hip_atomic_store((u64*)(wsl + WS_GRAN) + wg, ((u64)call << 32) | wgmax, __ATOMIC_RELAXED, __HIP_MEMORY_SCOPE_AGENT);
                __hip_atomic_store(seqp, call, __ATOMIC_RELAXED, __HIP_MEMORY_SCOPE_AGENT);
            }
            STAMP(60);
        } else if (wg == COLLECTOR) {
            const unsigned wgmax = wave_umax((lane >= 4 && lane < 24 && (lane & ~1) != 6) ? red[lane] : 0u);
            const u64 t0 = __builtin_amdgcn_s_memrealtime();
            unsigned polls = 0, gm;
            for (;;) {
                FENCE();
                const u32x4 g0 = __builtin_amdgcn_raw_buffer_load_b128(ss, WS_GRAN + lane * 16, 0, 16);
                const u32x4 g1 = __builtin_amdgcn_raw_buffer_load_b128(ss, WS_GRAN + 1024 + lane * 16, 0, 16);
                const bool self = lane == 63;
                gm = max(max(g0.x, g0.z), max(g1.x, self ? wgmax : g1.z));
                if (__all(g0.y == call && g0.w == call && g1.y == call && (self || g1.w == call))) break;
                if (timed_out(tmo, call, t0, polls)) { gm = 0xffffffffu; break; }
            }
            gm = wave_umax(gm);
            __hip_atomic_store((__attribute__((address_space(1))) u64*)gdst, ((u64)call << 32) | gm, __ATOMIC_RELAXED, __HIP_MEMORY_SCOPE_SYSTEM);
        }
        if (half == 0) { STAMP_FLUSH(16, 24); STAMP_FLUSH(40, 64); }
        return;
    }
    if (wid == 6) {
        PRIO(3);
        STAMP(32);
        const __amdgpu_buffer_rsrc_t xs = __builtin_amdgcn_make_buffer_rsrc((void*)a.x, 0, X_BYTES, SRD3);
        const unsigned po = (unsigned)(idx * 16 + (lane & 15)) * 1024u + (unsigned)(3 + (lane >> 4)) * 128u;
#define OWN_TOUCH() asm volatile("buffer_load_dword v240, %0, %1, %2 offen\n\tbuffer_load_dword v240, %3, %1, %2 offen" \
                         :: "v"(po), "s"(xs), "s"((unsigned)rank * SLOT_BYTES), "v"(po | 896u) : "v240", "memory")
        int nb = 0;
        const int s4 = (int)(((unsigned)(size_t)a.w >> 8) + 3u) & 3, u4 = s4 + (idx >> 1);
        const int fid = ((idx >> 1) - (((u4 + 3) >> 2) - ((s4 + 3) >> 2))) * 2 + mtile;
        if ((u4 & 3) != 0 && fid < 7 * NPART / 8) {
            const int id = fid * 8 + xcd, j = id % 7 + 1, part = id / 7, dst = (rank + j) & 7;
            char* wsd = a.ws[0];
#pragma unroll
            for (int r = 1; r < NR; ++r) wsd = (dst == r) ? a.ws[r] : wsd;
            const __amdgpu_buffer_rsrc_t ds = __builtin_amdgcn_make_buffer_rsrc((void*)wsd, 0, WS_GRAN, SRD3);
            const unsigned piece = part < 16 ? 3u * part : 2u * part + 16u, none = part < 16 ? 0u : OOB;
            const unsigned vsrc = (piece * 8u + (unsigned)(lane >> 3)) * 1024u + (unsigned)(lane & 7) * 16u;
            const unsigned vdst = piece * 1024u + (unsigned)lane * 16u;
            const unsigned fdst = WS_XFLAG + (unsigned)rank * 128u + (unsigned)part * 4u;
            const unsigned vsrc2 = vsrc + 8192u, vsrc3 = vsrc + 16384u + none, vdst3 = vdst + 2048u + none;
            const unsigned ssrc = (unsigned)dst * SLOT_BYTES, sdst = (unsigned)rank * SLOT_BYTES;
            auto xload = [&](auto cc) { x_load<decltype(cc)::value>(vsrc, vsrc2, vsrc3, xs, ssrc); };
            auto xstore = [&](auto cc) { x_store<decltype(cc)::value>(vdst, vdst3, ds, sdst + (unsigned)decltype(cc)::value * 65536u); };
            xload(IC<0>{}); xload(IC<1>{});
            OWN_TOUCH();
            const unsigned call = load_seq(seqp) + 1u;
            auto quiet = [&](int c, int from, int spins) {
                int spin = 0;
#pragma unroll 1
                for (;;) {
                    const bool last = nb >= 7 + 7 * c;
                    if (nb >= from && (c == 0 || last || ((nb ^ mtile) & 1) == 0) && vm_outstanding() == 0u) break;
                    if (!last && (nb < from || ++spin > spins)) { BAR(); ++nb; spin = 0; }
                    else __builtin_amdgcn_s_sleep(1);
                }
                wait_v<0>();
            };
#pragma unroll 1
            for (int c = 0; c < NCHUNK; ++c) {
                const int slot = c == 0 ? 1 + (part & 1) : (c == 1 ? 2 : PUSH_PACE * c) + (part & 2) + mtile;
                if (c <= 2) quiet(c, slot, 0);
                else {
#pragma unroll 1
                    while (nb < slot) { BAR(); ++nb; }
                }
                switch (c) {
                    case 0: xstore(IC<0>{}); break;
                    case 1: xstore(IC<1>{}); xload(IC<2>{}); xload(IC<3>{}); xload(IC<4>{}); xload(IC<5>{}); xload(IC<6>{}); xload(IC<7>{}); break;
                    case 2: xstore(IC<2>{}); break;
                    case 3: xstore(IC<3>{}); break;
                    case 4: xstore(IC<4>{}); break;
                    case 5: xstore(IC<5>{}); break;
                    case 6: xstore(IC<6>{}); break;
                    default: xstore(IC<7>{}); break;
                }
                quiet(c, 0, c == 0 ? 3 : 0);
                FENCE();
                if (lane == 0) __builtin_amdgcn_raw_buffer_store_b32(call, ds, fdst, c * 1024, SYS);
                FENCE();
                if (c == 0) STAMP(34);
            }
        } else OWN_TOUCH();
        STAMP(35);
        STAMP_FLUSH(32, 40);
        while (vm_outstanding() != 0u) BAR();
        return;
    }

    if (wid == 7) {
        const u64 t0 = __builtin_amdgcn_s_memrealtime();
        BAR();
#pragma unroll 1
        for (int i = 0; i < PACE_TO; ++i) {
            const unsigned lim = (unsigned)(PACE_T0 + (i + 1) * PACE_NS);
            while ((unsigned)(__builtin_amdgcn_s_memrealtime() - t0) * 10u < lim) __builtin_amdgcn_s_sleep(1);
            BAR();
        }
        return;
    }

    if (wid == 0) STAMP(0);
    const int wm = wid & 1, wn = wid >> 1;
    const int fr = lane & 15, g = lane >> 4, q = fr >> 2, p = lane & 3;
    unsigned aoff[2], boff[4];
#pragma unroll
    for (int kk = 0; kk < 2; ++kk) aoff[kk] = ldsbase + (unsigned)((wm * 128 + fr) * 128 + (((kk * 4 + g) ^ (fr >> 1)) << 4));
#pragma unroll
    for (int f = 0; f < 4; ++f)
        boff[f] = ldsbase + LDS_B0 + (unsigned)((8 * g + q) * 256 + (((wn * 8 + 2 * p + (f >> 1)) ^ (q | ((g & 1) << 3))) << 4) + ((((f & 1) ^ (p & 1))) << 3));
    asm volatile("" : "+v"(boff[0]), "+v"(boff[1]), "+v"(boff[2]), "+v"(boff[3]), "+v"(aoff[0]), "+v"(aoff[1]));
    const unsigned call = load_seq(seqp) + 1u;
    f32x4 acc[8][4];
#pragma unroll
    for (int m = 0; m < 8; ++m)
#pragma unroll
        for (int f = 0; f < 4; ++f) acc[m][f] = (f32x4){0.f, 0.f, 0.f, 0.f};
    bf16x8 xa[4][2], xb[4][2], wf[4][2];
    bf16x4 tl;
    unsigned bcur[4], acur[2];
#pragma unroll
    for (int f = 0; f < 4; ++f) bcur[f] = boff[f];
    acur[0] = aoff[0]; acur[1] = aoff[1];
#define RD_W(f, r) do { const bf16x4 v_ = __builtin_amdgcn_ds_read_tr16_b64_v4bf16((LAS bf16x4*)(bcur[f] + ((r) >> 1) * 8192 + ((r) & 1) * 1024)); \
        if ((r) & 1) wf[f][(r) >> 1] = __builtin_shufflevector(tl, v_, 0, 1, 2, 3, 4, 5, 6, 7); else tl = v_; } while (0)
#define RE_W(f, r) do { if (!((r) & 1)) asm volatile("" :: "v"(wf[f][(r) >> 1])); RD_W(f, r); } while (0)
#define RD_X(dst, h, r) do { dst[(r) >> 1][(r) & 1] = *(const LAS bf16x8*)(acur[(r) & 1] + ((h) * 4 + ((r) >> 1)) * 2048); } while (0)
#define SLOT(s) const int f = (s) >> 3, j = (f & 1) ? 3 - (((s) >> 1) & 3) : (((s) >> 1) & 3), kk = (s) & 1
    FENCE(); BAR(); FENCE();
    if (wid == 0) { STAMP(2); CSTAMP(30); }
#pragma unroll
    for (int r = 0; r < 4; ++r) RD_W(0, r);
#pragma unroll
    for (int r = 0; r < 8; ++r) RD_X(xa, 0, r);
#pragma unroll
    for (int r = 0; r < 8; ++r) RD_W(1 + (r >> 2), r & 3);
    wf[3][0] = wf[0][0]; wf[3][1] = wf[0][1];
    unsigned oa = A_STAGE, ob = B_STAGE;
    auto stepA = [&]() {
#pragma unroll
        for (int s = 0; s < 32; ++s) {
            SLOT(s);
            acc[j][f] = MFMA(wf[f][kk], xa[j][kk], acc[j][f]);
            SCHED();
            if (s < 4) RE_W(3, s);
            else if (s < 12) RD_X(xb, 1, s - 4);
            else if (s == 23) WAIT_L0();
            else if (s >= 24 && s < 28) bcur[s - 24] = boff[s - 24] + ob;
            else if (s >= 28 && s < 30) acur[s - 28] = aoff[s - 28] + oa;
            SCHED();
        }
    };
    auto stepB = [&]() {
        BAR(); SCHED();
#pragma unroll
        for (int s = 0; s < 32; ++s) {
            SLOT(s);
            acc[4 + j][f] = MFMA(wf[f][kk], xb[j][kk], acc[4 + j][f]);
            SCHED();
            if (s < 8) RD_X(xa, 0, s);
            else if ((s & 7) < 4) RE_W((s >> 3) - 1, s & 3);
            else if (s == 12) { ob = (ob + B_STAGE) & (RB * B_STAGE - 1); asm volatile("" : "+s"(ob)); }
            else if (s == 13) { oa += A_STAGE; asm volatile("" : "+s"(oa)); }
            else if (s == 14) { oa = oa == RA * A_STAGE ? 0u : oa; asm volatile("" : "+s"(oa)); }
            else if (s == 15) WAIT_L0();
            SCHED();
        }
    };
    WAIT_L0();
    stepA();
#pragma unroll 1
    for (int t = 0; t < NT - 1; ++t) { stepB(); stepA(); }

    int mx0 = 0, mx1 = 0;
#define TAKE_MAX(v) do { mx0 = max(max(mx0, __float_as_int((v)[0])), __float_as_int((v)[1])); mx1 = max(max(mx1, __float_as_int((v)[2])), __float_as_int((v)[3])); } while (0)
    SCHED(); BAR(); SCHED();
#pragma unroll
    for (int s = 0; s < 32; ++s) {
        { SLOT(s); acc[4 + j][f] = MFMA(wf[f][kk], xb[j][kk], acc[4 + j][f]); }
        SCHED();
        if (s < 16) TAKE_MAX(acc[s & 3][s >> 2]);
        else if (s < 28) { SLOT(2 * (s - 16)); (void)kk; TAKE_MAX(acc[4 + j][f]); }
        SCHED();
    }
#pragma unroll
    for (int j = 0; j < 4; ++j) TAKE_MAX(acc[7 - j][3]);
    const unsigned wmax = row_umax((unsigned)max(mx0, mx1));
    if (wid == 0) { STAMP(11); CSTAMP(31); }
    if (fr == 15) red[8 + wid * 4 + g] = wmax;
    FENCE(); WAIT_L0(); BAR(); FENCE();
    const unsigned wgmax = wave_umax((lane >= 4 && lane < 24 && (lane & ~1) != 6) ? red[lane] : 0u);
    const bool odd = g & 1;
    const __amdgpu_buffer_rsrc_t outr = __builtin_amdgcn_make_buffer_rsrc(a.out, 0, MLOC * NDIM, SRD3);
    const unsigned ovo = (unsigned)(mtile * BM + wm * 128 + fr) * NDIM + (unsigned)(ntile * BN + wn * 64 + g * 16);
    unsigned used = 0xfffffffeu;
    const char* finp = wsl + WS_FIN + xcd * 128;
    const u64 t0 = __builtin_amdgcn_s_memrealtime();
    unsigned polls = 0;
    for (;;) {
        unsigned gm = wgmax, miss = 0;
        {
            const u64 fv = __hip_atomic_load((const u64*)finp + (lane & 7), __ATOMIC_RELAXED, __HIP_MEMORY_SCOPE_SYSTEM);
            const bool ok = (unsigned)(fv >> 32) == call;
            gm = max(gm, (unsigned)__builtin_amdgcn_readlane(row_umax(ok ? (unsigned)fv : 0u), 15));
            miss = (unsigned)__popcll(__ballot(!ok)) >> 3;
        }
        if (miss != 0u && timed_out(tmo, call, t0, polls)) gm = 0xffffffffu;
        if (gm == 0xffffffffu) {
            const u32x4 o = {0x7f7f7f7fu, 0x7f7f7f7fu, 0x7f7f7f7fu, 0x7f7f7f7fu};
#pragma unroll
            for (int m = 0; m < 8; ++m) __builtin_amdgcn_raw_buffer_store_b128(o, outr, ovo + m * 16 * NDIM, 0, OUT_AUX);
            break;
        }
        if (miss == 0u && wid == 0) STAMP(13);
        if (gm != used && miss <= SPEC_THR) {
            used = gm;
            const float amax = __uint_as_float(gm);
            const float inv = amax > 0.f ? 448.0f / amax : 0.f;
            const f32x2 sc = {inv * (1.0f / 512.0f), inv * (1.0f / 512.0f)};
#pragma unroll
            for (int m = 0; m < 8; ++m) {
                unsigned d[4];
#pragma unroll
                for (int f = 0; f < 4; ++f) {
                    f32x2 lo = {acc[m][f][0], acc[m][f][1]}, hi = {acc[m][f][2], acc[m][f][3]};
                    asm("v_pk_mul_f32 %0, %1, %2 clamp" : "=v"(lo) : "v"(lo), "v"(sc));
                    asm("v_pk_mul_f32 %0, %1, %2 clamp" : "=v"(hi) : "v"(hi), "v"(sc));
                    s16x2 w = __builtin_bit_cast(s16x2, lo.x);
                    w = __builtin_amdgcn_cvt_scalef32_pk_fp8_f32(w, lo.x, lo.y, 1.0f / 512.0f, false);
                    w = __builtin_amdgcn_cvt_scalef32_pk_fp8_f32(w, hi.x, hi.y, 1.0f / 512.0f, true);
                    d[f] = __builtin_bit_cast(unsigned, w);
                }
                u32x4 o;
                o.x = odd ? d[1] : d[0]; o.y = odd ? d[0] : d[1]; o.z = odd ? d[3] : d[2]; o.w = odd ? d[2] : d[3];
                __builtin_amdgcn_raw_buffer_store_b128(o, outr, ovo + m * 16 * NDIM, 0, OUT_AUX);
            }
        }
        if (miss == 0u) break;
        __builtin_amdgcn_s_sleep(1);
    }
}

}

extern "C" void kernel_launch(int rank, int nranks, ncclComm_t comm,
                              hipStream_t stream,
                              void* const* d_in, const int* in_sizes, int n_in,
                              void* d_out, int out_size,
                              const int* in_dtypes, int out_dtype,
                              void* const* d_out_all,
                              void* const* d_ws_all, size_t ws_bytes) {
    using namespace a2ag;
    if (nranks != NR || (unsigned)rank >= (unsigned)NR || n_in < 2 || ws_bytes < WS_NEED
        || in_sizes[0] != KDIM * KSL || in_sizes[1] != KDIM * NDIM || out_size != MLOC * NDIM) {
        fprintf(stderr, "kernel_launch: unexpected problem geometry\n");
        return;
    }
    HIP_CHECK(hipFuncSetAttribute((const void*)a2a_gemm_kernel, hipFuncAttributeMaxDynamicSharedMemorySize, LDS_BYTES));
    Args a;
    a.x = d_in[0]; a.w = d_in[1]; a.out = d_out; a.rank = rank; a.pad = 0;
    a.wsl = (char*)d_ws_all[rank];
    for (int r = 0; r < NR; ++r) a.ws[r] = (char*)d_ws_all[r];
    hipLaunchKernelGGL(a2a_gemm_kernel, dim3(NWG), dim3(NTHREADS), LDS_BYTES, stream, a);
    HIP_CHECK(hipGetLastError());
}
```

```cpp
#include <hip/hip_runtime.h>
#include <rccl/rccl.h>
#include <cstdio>
#include <cstdint>
#include <type_traits>

#define HIP_CHECK(expr) do { hipError_t _e = (expr); if (_e != hipSuccess) { \
    fprintf(stderr, "HIP error %s at %s:%d\n", hipGetErrorString(_e), __FILE__, __LINE__); \
    return; } } while (0)

#define PUSH_PACE 4
#define SPEC_THR 8
#define OUT_AUX 17
#define WSPLIT 6
#define PACE_NS 820
#define PACE_T0 1700
#define PACE_TO 36

#pragma clang diagnostic ignored "-Wint-to-pointer-cast"

namespace a2ag {

typedef __bf16 bf16x4 __attribute__((ext_vector_type(4)));
typedef __bf16 bf16x8 __attribute__((ext_vector_type(8)));
typedef float f32x4 __attribute__((ext_vector_type(4)));
typedef float f32x2 __attribute__((ext_vector_type(2)));
typedef short s16x2 __attribute__((ext_vector_type(2)));
typedef unsigned u32x4 __attribute__((ext_vector_type(4)));
typedef unsigned long long u64;
#define LAS __attribute__((address_space(3)))
#define DI __device__ __forceinline__
template <int V> using IC = std::integral_constant<int, V>;

constexpr int NR = 8;
constexpr int MLOC = 512, KDIM = 4096, NDIM = 16384, KSL = 512;
constexpr int BM = 256, BN = 128, BK = 64;
constexpr int NT = KDIM / BK;
constexpr int NWG = (MLOC / BM) * (NDIM / BN);
constexpr int NTHREADS = 512;
constexpr unsigned A_STAGE = BM * BK * 2, B_STAGE = BK * BN * 2;
constexpr int RA = 3, RB = 4;
constexpr unsigned LDS_B0 = RA * A_STAGE;
constexpr unsigned LDS_BYTES = LDS_B0 + RB * B_STAGE;
constexpr unsigned SLOT_BYTES = MLOC * KSL * 2;
constexpr unsigned X_BYTES = NR * SLOT_BYTES;
constexpr unsigned W_BYTES = (unsigned)KDIM * NDIM * 2;
constexpr unsigned W_TILE = BK * NDIM * 2;
constexpr int NCHUNK = 8, NPART = 24;
constexpr unsigned WS_XFLAG = X_BYTES;
constexpr unsigned WS_GRAN = WS_XFLAG + NCHUNK * 1024;
constexpr unsigned WS_FIN = WS_GRAN + NWG * 8;
constexpr unsigned WS_SEQ = WS_FIN + 8 * 512;
constexpr unsigned WS_TMO = WS_SEQ + NWG * 64;
constexpr unsigned WS_DBG = WS_TMO + 4096;
constexpr unsigned WS_NEED = WS_DBG + NWG * 64 * 12;
constexpr u64 DEADLINE_TICKS = 40000000ull;
constexpr unsigned OOB = 0x40000000u;
constexpr unsigned SRD3 = 0x00020000u;
constexpr int SYS = 17;

struct Args {
    const void* x; const void* w;
    char* wsl;
    int rank, pad;
    void* out;
    char* ws[NR];
};

DI void set_m0(unsigned v) { asm volatile("s_mov_b32 m0, %0\n\ts_nop 0" :: "s"(v) : "memory"); }
template <int OFF> DI void dma_x(__amdgpu_buffer_rsrc_t srd, unsigned voff, unsigned soff) {
    asm volatile("buffer_load_dwordx4 %0, %1, %2 offen offset:%3 sc1 lds" :: "v"(voff), "s"(srd), "s"(soff), "n"(OFF) : "memory");
}
template <int OFF> DI void dma_w(__amdgpu_buffer_rsrc_t srd, unsigned voff, unsigned soff) {
    asm volatile("buffer_load_dwordx4 %0, %1, %2 offen offset:%3 lds" :: "v"(voff), "s"(srd), "s"(soff), "n"(OFF) : "memory");
}
#define XCLOB "v144", "v145", "v146", "v147", "v148", "v149", "v150", "v151", "v152", "v153", "v154", "v155", "v156", "v157", "v158", "v159", "v160", "v161", "v162", "v163", "v164", "v165", "v166", "v167", "v168", "v169", "v170", "v171", "v172", "v173", "v174", "v175", "v176", "v177", "v178", "v179", "v180", "v181", "v182", "v183", "v184", "v185", "v186", "v187", "v188", "v189", "v190", "v191", "v192", "v193", "v194", "v195", "v196", "v197", "v198", "v199", "v200", "v201", "v202", "v203", "v204", "v205", "v206", "v207", "v208", "v209", "v210", "v211", "v212", "v213", "v214", "v215", "v216", "v217", "v218", "v219", "v220", "v221", "v222", "v223", "v224", "v225", "v226", "v227", "v228", "v229", "v230", "v231", "v232", "v233", "v234", "v235", "v236", "v237", "v238", "v239"
template <int C> DI void x_load(unsigned v0, unsigned v1, unsigned v2, __amdgpu_buffer_rsrc_t srd, unsigned soff) {
    asm volatile("buffer_load_dwordx4 v[%c5:%c6], %0, %3, %4 offen offset:%c11\n\tbuffer_load_dwordx4 v[%c7:%c8], %1, %3, %4 offen offset:%c11\n\t"
                 "buffer_load_dwordx4 v[%c9:%c10], %2, %3, %4 offen offset:%c11"
                 :: "v"(v0), "v"(v1), "v"(v2), "s"(srd), "s"(soff), "n"(144 + 12 * C), "n"(147 + 12 * C), "n"(148 + 12 * C), "n"(151 + 12 * C),
                    "n"(152 + 12 * C), "n"(155 + 12 * C), "n"(C * 128) : "memory", XCLOB);
}
template <int C> DI void x_store(unsigned voff, unsigned voff2, __amdgpu_buffer_rsrc_t srd, unsigned soff) {
    asm volatile("buffer_store_dwordx4 v[%c4:%c5], %0, %2, %3 offen sc0 sc1\n\tbuffer_store_dwordx4 v[%c6:%c7], %0, %2, %3 offen offset:1024 sc0 sc1\n\t"
                 "buffer_store_dwordx4 v[%c8:%c9], %1, %2, %3 offen sc0 sc1"
                 :: "v"(voff), "v"(voff2), "s"(srd), "s"(soff), "n"(144 + 12 * C), "n"(147 + 12 * C), "n"(148 + 12 * C), "n"(151 + 12 * C),
                    "n"(152 + 12 * C), "n"(155 + 12 * C) : "memory");
}
template <int N> DI void wait_v() { asm volatile("s_waitcnt vmcnt(%0)" :: "n"(N) : "memory"); }
#define WAIT_L0() __builtin_amdgcn_s_waitcnt(0xC07F)
#define FENCE() asm volatile("" ::: "memory")
#define BAR() __builtin_amdgcn_s_barrier()
#define SCHED() __builtin_amdgcn_sched_barrier(0)
#define PRIO(n) __builtin_amdgcn_s_setprio(n)
#define MFMA(a, b, c) __builtin_amdgcn_mfma_f32_16x16x32_bf16(a, b, c, 0, 0, 0)

#define STAMP(i) do {} while (0)
#define CSTAMP(i) do {} while (0)
#define STAMP_DECL
#define STAMP_FLUSH(lo, hi) do {} while (0)

DI unsigned ktile_of(int t, int rank) {
    const int i = t - 8, c = (i * 37) >> 8, j = i - c * 7;
    return t < 8 ? (unsigned)(rank * 8 + t) : (unsigned)((((rank + 1 + j) & 7) << 3) + c);
}

DI bool timed_out(unsigned* tmo, unsigned call, u64 t0, unsigned& polls) {
    if ((++polls & 31u) != 0u) return false;
    if (__builtin_amdgcn_readfirstlane(__hip_atomic_load(tmo, __ATOMIC_RELAXED, __HIP_MEMORY_SCOPE_AGENT)) == call
        || __builtin_amdgcn_s_memrealtime() - t0 > DEADLINE_TICKS) {
        __hip_atomic_store(tmo, call, __ATOMIC_RELAXED, __HIP_MEMORY_SCOPE_AGENT);
        return true;
    }
    return false;
}
DI unsigned load_seq(const unsigned* p) {
    unsigned v;
    asm volatile("s_load_dword %0, %1, 0x0 glc\n\ts_waitcnt lgkmcnt(0)" : "=s"(v) : "s"(p) : "memory");
    return v;
}
DI unsigned vm_outstanding() {
    const unsigned s = __builtin_amdgcn_s_getreg(7 | (0 << 6) | (31 << 11));
    return (s & 15u) | ((s >> 18) & 48u);
}
DI unsigned row_umax(unsigned v) {
    v = max(v, (unsigned)__builtin_amdgcn_update_dpp(0, (int)v, 0x111, 0xf, 0xf, false));
    v = max(v, (unsigned)__builtin_amdgcn_update_dpp(0, (int)v, 0x112, 0xf, 0xf, false));
    v = max(v, (unsigned)__builtin_amdgcn_update_dpp(0, (int)v, 0x114, 0xf, 0xf, false));
    v = max(v, (unsigned)__builtin_amdgcn_update_dpp(0, (int)v, 0x118, 0xf, 0xf, false));
    return v;
}
DI unsigned wave_umax(unsigned v) {
    v = row_umax(v);
    const unsigned a = __builtin_amdgcn_readlane(v, 15), b = __builtin_amdgcn_readlane(v, 31);
    const unsigned c = __builtin_amdgcn_readlane(v, 47), d = __builtin_amdgcn_readlane(v, 63);
    return max(max(a, b), max(c, d));
}

__global__ __launch_bounds__(NTHREADS) void a2a_gemm_kernel(Args a) {
    extern __shared__ __attribute__((aligned(16))) char smem[];
    LAS char* lds = (LAS char*)smem;
    const int tid = threadIdx.x, lane = tid & 63;
    const int wid = __builtin_amdgcn_readfirstlane(tid >> 6);
    const int wg = blockIdx.x, rank = a.rank;
    const int xcd = wg & 7, idx = wg >> 3;
    const int ntile = xcd * 16 + (idx >> 1), mtile = idx & 1;
    const unsigned ldsbase = (unsigned)(size_t)lds;
    LAS unsigned* red = (LAS unsigned*)(lds + LDS_B0 + 3 * B_STAGE);
    char* wsl = a.wsl;
    asm volatile("" :: "s"(a.x), "s"(a.w), "s"(wsl), "s"(rank));
    unsigned* seqp = (unsigned*)(wsl + WS_SEQ + wg * 64);
    unsigned* tmo = (unsigned*)(wsl + WS_TMO);
    STAMP_DECL;

    if (wid == 4 || wid == 5) {
        PRIO(3);
        const int half = wid - 4;
        const __amdgpu_buffer_rsrc_t ss = __builtin_amdgcn_make_buffer_rsrc((void*)wsl, 0, WS_TMO, SRD3);
        const __amdgpu_buffer_rsrc_t wr = __builtin_amdgcn_make_buffer_rsrc((void*)a.w, 0, W_BYTES, SRD3);
        unsigned vs[2], vw[2];
#pragma unroll
        for (int e = 0; e < 2; ++e) {
            vs[e] = (unsigned)(lane >> 3) * 128u + (unsigned)(((lane & 7) ^ (4 * e + (lane >> 4))) << 4);
            vw[e] = (unsigned)(lane >> 4) * 32768u + (unsigned)(((lane & 15) ^ ((e << 3) | (lane >> 4))) << 4);
        }
        const unsigned row0 = (unsigned)(mtile * BM + half * 128);
        const unsigned lA = ldsbase + (unsigned)half * 16384u, lB = ldsbase + LDS_B0 + (unsigned)half * 8192u;
        const unsigned soB = (unsigned)ntile * (BN * 2u) + (unsigned)half * (32u * 32768u);
        const unsigned soOwn = (unsigned)rank * SLOT_BYTES + row0 * 1024u;
        const char* xb = (const char*)a.x;
        unsigned dx = 7168u, v0 = vs[0] + (unsigned)(lane >> 3) * 896u, v1 = vs[1] + (unsigned)(lane >> 3) * 896u;
        unsigned la = lA;
        auto next_la = [&](unsigned l) -> unsigned { return l == lA + (RA - 1) * A_STAGE ? lA : l + A_STAGE; };
        auto issueA = [&](unsigned so) {
            const __amdgpu_buffer_rsrc_t srd = __builtin_amdgcn_make_buffer_rsrc((void*)xb, 0, X_BYTES, SRD3);
#pragma unroll
            for (int p = 0; p < 16; p += 4) {
                const unsigned s4 = so + p * (dx + 1024u);
                set_m0(la + p * 1024u);
                dma_x<0>(srd, v0, s4); dma_x<1024>(srd, v1, s4 + dx); dma_x<2048>(srd, v0, s4 + 2u * dx); dma_x<3072>(srd, v1, s4 + 3u * dx);
            }
            la = next_la(la);
        };
        auto issueAs = [&](unsigned so, unsigned l) {
#pragma unroll
            for (int p = 0; p < 16; p += 4) {
                set_m0(l + p * 1024u);
                dma_x<0>(ss, vs[0], so + p * 1024u); dma_x<1024>(ss, vs[1], so + p * 1024u); dma_x<2048>(ss, vs[0], so + p * 1024u); dma_x<3072>(ss, vs[1], so + p * 1024u);
            }
        };
        auto wstage = [&](int t) -> unsigned { return lB + ((unsigned)t & (RB - 1)) * B_STAGE; };
        auto issueW = [&](unsigned so, unsigned lb, auto lo, auto hi) {
            constexpr int LO = decltype(lo)::value, HI = decltype(hi)::value;
            if constexpr (LO == 0) {
                set_m0(lb);
                dma_w<0>(wr, vw[0], so); dma_w<1024>(wr, vw[0], so + 130048u); dma_w<2048>(wr, vw[1], so + 260096u); dma_w<3072>(wr, vw[1], so + 390144u);
            }
            if constexpr (HI > 4 && LO < 8) set_m0(lb + 4096u);
            if constexpr (LO <= 4 && HI >= 6) { dma_w<0>(wr, vw[0], so + 524288u); dma_w<1024>(wr, vw[0], so + 654336u); }
            if constexpr (LO <= 6 && HI == 8) { dma_w<2048>(wr, vw[1], so + 784384u); dma_w<3072>(wr, vw[1], so + 914432u); }
        };
        using I0 = IC<0>; using IS = IC<WSPLIT>; using I8 = IC<8>;
        constexpr int N_HEAVY = 16 + 8 + WSPLIT, N_LIGHT = 16 + 8;
        const unsigned fvo = WS_XFLAG - 1024u + (unsigned)lane * 16u;
#define POLL_ISSUE(n) asm volatile("buffer_load_dwordx4 v[240:243], %0, %1, %2 offen sc0 sc1" :: "v"(fvo), "s"(ss), "s"((n) * 1024u) : "v240", "v241", "v242", "v243", "memory")
        const unsigned soW0 = (unsigned)rank * (8u * W_TILE) + soB;
        issueA(soOwn); issueW(soW0, wstage(0), I0{}, I8{});
        asm volatile("s_load_dword s96, %0, 0x0 glc" :: "s"(seqp) : "s96", "memory");
        issueW(soW0 + W_TILE, wstage(1), I0{}, I8{});
        issueA(soOwn + 128u);
        unsigned tabA, tabW;
        {
            const int t = lane + 3;
            const unsigned kt = ktile_of(t, rank);
            const unsigned first = (int)(kt >> 3) == ((rank + 1) & 7) ? 16u : 0u;
            tabA = t < 8 ? soOwn + (unsigned)t * 128u : (t < NT ? kt * 65536u + row0 * 128u + first + (kt & 7u) + 1u : OOB);
            const unsigned n0 = t < 8 ? 0u : (kt & 7u) + 1u, n2 = t + 2 < 8 ? 0u : (ktile_of(t + 2, rank) & 7u) + 1u;
            tabW = t < NT ? kt * W_TILE + soB + ((t + 2 < NT && n2 > n0) ? n2 : 0u) : OOB;
        }
        unsigned call;
        asm volatile("s_waitcnt lgkmcnt(0)\n\ts_add_u32 %0, s96, 1" : "=s"(call) :: "scc", "memory");
        wait_v<24>(); BAR();
        issueW(soW0 + 2u * W_TILE, wstage(2), I0{}, I8{}); issueA(soOwn + 256u);
        if (mtile == 0) issueW(soW0 + 3u * W_TILE, wstage(3), I0{}, IS{});
        const bool own = (lane >> 3) == rank || (lane & 7) >= NPART / 4;
        bool good = true;
        unsigned prog = 0;
        unsigned pend = 0, okc = 0;
        u64 okm = 0;
        auto read_flags = [&](unsigned n) {
            u32x4 f;
            asm volatile("v_mov_b32 %0, v240\n\tv_mov_b32 %1, v241\n\tv_mov_b32 %2, v242\n\tv_mov_b32 %3, v243" : "=v"(f.x), "=v"(f.y), "=v"(f.z), "=v"(f.w) :: "memory");
            okm = __ballot(own || (int)((f.x - call) | (f.y - call) | (f.z - call) | (f.w - call)) >= 0);
            okc = n;
            if (okm == ~0ull) prog = n;
        };
        auto await = [&](unsigned need, unsigned so) {
            if (__builtin_expect(prog < need, 0)) {
                const unsigned sh = (so >> 19) * 8u;
                if (!(okc >= need && ((unsigned)(okm >> sh) & 255u) == 255u)) {
                    if (half == 0) STAMP(40 + need - 1);
                    const u64 t0 = __builtin_amdgcn_s_memrealtime();
                    unsigned polls = 0;
                    for (;;) {
                        POLL_ISSUE(need); wait_v<0>();
                        read_flags(need);
                        if (((unsigned)(okm >> sh) & 255u) == 255u) break;
                        if (timed_out(tmo, call, t0, polls)) { good = false; prog = NCHUNK; break; }
                        __builtin_amdgcn_s_sleep(4);
                    }
                    if (half == 0) STAMP(48 + need - 1);
                }
            }
            pend = need - min(prog, need);
        };
        unsigned pc = 0;
        int i = 0;
#pragma unroll 1
        for (; i < 5 + mtile; ++i) {
            {
                const bool mine = i < 5;
                xb = mine ? (const char*)a.x : (const char*)wsl;
                dx = mine ? 7168u : 0u;
                const unsigned dd = mine ? 896u : 0u;
                v0 = vs[0] + (unsigned)(lane >> 3) * dd; v1 = vs[1] + (unsigned)(lane >> 3) * dd;
            }
            const unsigned ta = __builtin_amdgcn_readlane(tabA, i), w3 = __builtin_amdgcn_readlane(tabW, i), w4 = __builtin_amdgcn_readlane(tabW, i + 1);
            if ((i ^ mtile) & 1) {
                wait_v<N_LIGHT>(); BAR();
                pc = w3 & 15u;
                if (pc != 0u) POLL_ISSUE(pc);
                issueA(ta & ~127u);
                issueW(w3 & ~255u, wstage(i + 3), I0{}, I8{});
                issueW(w4 & ~255u, wstage(i + 4), I0{}, IS{});
            } else {
                wait_v<N_HEAVY>(); BAR();
                issueW(w3 & ~255u, wstage(i + 3), IS{}, I8{});
                if (pc != 0u) read_flags(pc);
                await(ta & 15u, ta);
                issueA(ta & ~127u);
            }
        }
        unsigned aH, aL, wH, wE, laH, laL, lbH, lbE, nH, nL, rareH, rareL;
        auto plan = [&](int h) {
            const unsigned t0 = __builtin_amdgcn_readlane(tabA, h & 63), t1 = __builtin_amdgcn_readlane(tabA, (h + 1) & 63);
            const unsigned u0 = __builtin_amdgcn_readlane(tabW, h & 63), u1 = __builtin_amdgcn_readlane(tabW, (h + 1) & 63);
            aH = t0 & ~127u; nH = t0 & 15u; aL = t1 & ~127u; nL = t1 & 15u; wH = u0 & ~255u; pc = u0 & 15u; wE = u1 & ~255u;
            laH = la; laL = next_la(la); la = next_la(laL);
            lbH = wstage(h + 3); lbE = wstage(h + 4);
            rareH = pc | (t0 & 16u); rareL = pc | (t1 & 16u);
            asm volatile("" : "+s"(aH), "+s"(aL), "+s"(wH), "+s"(wE), "+s"(laH), "+s"(laL), "+s"(lbH), "+s"(lbE), "+s"(rareH), "+s"(rareL));
        };
        plan(i);
#pragma unroll 1
        for (int n = 0; n < 29; ++n) {
            wait_v<N_LIGHT>(); BAR();
            if (__builtin_expect((rareH | pend) != 0u, 0)) { await(nH, aH); if (pc != 0u) POLL_ISSUE(pc); }
            issueAs(aH, laH);
            issueW(wH, lbH, I0{}, I8{});
            issueW(wE, lbE, I0{}, IS{});
            wait_v<N_HEAVY>(); BAR();
            issueW(wE, lbE, IS{}, I8{});
            if (__builtin_expect((rareL | pend) != 0u, 0)) { if (pc != 0u) read_flags(pc); await(nL, aL); }
            issueAs(aL, laL);
            i += 2;
            plan(i);
        }
        if (mtile == 0) BAR();
        if (lane == 0) red[4 + half] = good ? 0u : 0xffffffffu;
        u64 gdst = 0;
        if (half == 1 && idx == 31) {
            char* pw = a.ws[0];
#pragma unroll
            for (int r = 1; r < NR; ++r) pw = ((lane >> 3) == r) ? a.ws[r] : pw;
            gdst = (u64)(pw + WS_FIN + (lane & 7) * 512 + rank * 64 + xcd * 8);
            asm volatile("" : "+v"(gdst));
        }
        wait_v<0>();
        FENCE(); WAIT_L0(); BAR(); FENCE();
        if (half == 0) {
            const unsigned wgmax = wave_umax((lane >= 4 && lane < 24 && (lane & ~1) != 6) ? red[lane] : 0u);
            if (lane == 0) {
                __hip_atomic_store((u64*)(wsl + WS_GRAN) + xcd * 32 + idx, ((u64)call << 32) | wgmax, __ATOMIC_RELAXED, __HIP_MEMORY_SCOPE_AGENT);
                __hip_atomic_store(seqp, call, __ATOMIC_RELAXED, __HIP_MEMORY_SCOPE_AGENT);
            }
            STAMP(60);
        } else if (idx == 31) {
            const unsigned wgmax = wave_umax((lane >= 4 && lane < 24 && (lane & ~1) != 6) ? red[lane] : 0u);
            const u64 t0 = __builtin_amdgcn_s_memrealtime();
            unsigned polls = 0, gm;
            for (;;) {
                const u64 gv = __hip_atomic_load((const u64*)(wsl + WS_GRAN) + xcd * 32 + (lane & 31), __ATOMIC_RELAXED, __HIP_MEMORY_SCOPE_AGENT);
                const bool self = (lane & 31) == 31;
                gm = self ? wgmax : (unsigned)gv;
                if (__all(self || (unsigned)(gv >> 32) == call)) break;
                if (timed_out(tmo, call, t0, polls)) { gm = 0xffffffffu; break; }
            }
            gm = wave_umax(gm);
            __hip_atomic_store((__attribute__((address_space(1))) u64*)gdst, ((u64)call << 32) | gm, __ATOMIC_RELAXED, __HIP_MEMORY_SCOPE_SYSTEM);
        }
        if (half == 0) { STAMP_FLUSH(16, 24); STAMP_FLUSH(40, 64); }
        return;
    }
    if (wid == 6) {
        PRIO(3);
        STAMP(32);
        const __amdgpu_buffer_rsrc_t xs = __builtin_amdgcn_make_buffer_rsrc((void*)a.x, 0, X_BYTES, SRD3);
        const unsigned po = (unsigned)(idx * 16 + (lane & 15)) * 1024u + (unsigned)(3 + (lane >> 4)) * 128u;
#define OWN_TOUCH() asm volatile("buffer_load_dword v240, %0, %1, %2 offen\n\tbuffer_load_dword v240, %3, %1, %2 offen" \
                         :: "v"(po), "s"(xs), "s"((unsigned)rank * SLOT_BYTES), "v"(po | 896u) : "v240", "memory")
        int nb = 0;
        const int s4 = (int)(((unsigned)(size_t)a.w >> 8) + 3u) & 3, u4 = s4 + (idx >> 1);
        const int fid = ((idx >> 1) - (((u4 + 3) >> 2) - ((s4 + 3) >> 2))) * 2 + mtile;
        if ((u4 & 3) != 0 && fid < 7 * NPART / 8) {
            const int id = fid * 8 + xcd, j = id % 7 + 1, part = id / 7, dst = (rank + j) & 7;
            char* wsd = a.ws[0];
#pragma unroll
            for (int r = 1; r < NR; ++r) wsd = (dst == r) ? a.ws[r] : wsd;
            const __amdgpu_buffer_rsrc_t ds = __builtin_amdgcn_make_buffer_rsrc((void*)wsd, 0, WS_GRAN, SRD3);
            const unsigned piece = part < 16 ? 3u * part : 2u * part + 16u, none = part < 16 ? 0u : OOB;
            const unsigned vsrc = (piece * 8u + (unsigned)(lane >> 3)) * 1024u + (unsigned)(lane & 7) * 16u;
            const unsigned vdst = piece * 1024u + (unsigned)lane * 16u;
            const unsigned fdst = WS_XFLAG + (unsigned)rank * 128u + (unsigned)part * 4u;
            const unsigned vsrc2 = vsrc + 8192u, vsrc3 = vsrc + 16384u + none, vdst3 = vdst + 2048u + none;
            const unsigned ssrc = (unsigned)dst * SLOT_BYTES, sdst = (unsigned)rank * SLOT_BYTES;
            auto xload = [&](auto cc) { x_load<decltype(cc)::value>(vsrc, vsrc2, vsrc3, xs, ssrc); };
            auto xstore = [&](auto cc) { x_store<decltype(cc)::value>(vdst, vdst3, ds, sdst + (unsigned)decltype(cc)::value * 65536u); };
            xload(IC<0>{}); xload(IC<1>{});
            OWN_TOUCH();
            const unsigned call = load_seq(seqp) + 1u;
            auto quiet = [&](int c, int from, int spins) {
                int spin = 0;
#pragma unroll 1
                for (;;) {
                    const bool last = nb >= 7 + 7 * c;
                    if (nb >= from && (c == 0 || last || ((nb ^ mtile) & 1) == 0) && vm_outstanding() == 0u) break;
                    if (!last && (nb < from || ++spin > spins)) { BAR(); ++nb; spin = 0; }
                    else __builtin_amdgcn_s_sleep(1);
                }
                wait_v<0>();
            };
#pragma unroll 1
            for (int c = 0; c < NCHUNK; ++c) {
                const int slot = c == 0 ? 1 + (part & 1) : (c == 1 ? 2 : PUSH_PACE * c) + (part & 2) + mtile;
                if (c <= 2) quiet(c, slot, 0);
                else {
#pragma unroll 1
                    while (nb < slot) { BAR(); ++nb; }
                }
                switch (c) {
                    case 0: xstore(IC<0>{}); break;
                    case 1: xstore(IC<1>{}); xload(IC<2>{}); xload(IC<3>{}); xload(IC<4>{}); xload(IC<5>{}); xload(IC<6>{}); xload(IC<7>{}); break;
                    case 2: xstore(IC<2>{}); break;
                    case 3: xstore(IC<3>{}); break;
                    case 4: xstore(IC<4>{}); break;
                    case 5: xstore(IC<5>{}); break;
                    case 6: xstore(IC<6>{}); break;
                    default: xstore(IC<7>{}); break;
                }
                quiet(c, 0, c == 0 ? 3 : 0);
                FENCE();
                if (lane == 0) __builtin_amdgcn_raw_buffer_store_b32(call, ds, fdst, c * 1024, SYS);
                FENCE();
                if (c == 0) STAMP(34);
            }
        } else OWN_TOUCH();
        STAMP(35);
        STAMP_FLUSH(32, 40);
        while (vm_outstanding() != 0u) BAR();
        return;
    }

    if (wid == 7) {
        const u64 t0 = __builtin_amdgcn_s_memrealtime();
        BAR();
#pragma unroll 1
        for (int i = 0; i < PACE_TO; ++i) {
            const unsigned lim = (unsigned)(PACE_T0 + (i + 1) * PACE_NS);
            while ((unsigned)(__builtin_amdgcn_s_memrealtime() - t0) * 10u < lim) __builtin_amdgcn_s_sleep(1);
            BAR();
        }
        return;
    }

    if (wid == 0) STAMP(0);
    const int wm = wid & 1, wn = wid >> 1;
    const int fr = lane & 15, g = lane >> 4, q = fr >> 2, p = lane & 3;
    unsigned aoff[2], boff[4];
#pragma unroll
    for (int kk = 0; kk < 2; ++kk) aoff[kk] = ldsbase + (unsigned)((wm * 128 + fr) * 128 + (((kk * 4 + g) ^ (fr >> 1)) << 4));
#pragma unroll
    for (int f = 0; f < 4; ++f)
        boff[f] = ldsbase + LDS_B0 + (unsigned)((8 * g + q) * 256 + (((wn * 8 + 2 * p + (f >> 1)) ^ (q | ((g & 1) << 3))) << 4) + ((((f & 1) ^ (p & 1))) << 3));
    asm volatile("" : "+v"(boff[0]), "+v"(boff[1]), "+v"(boff[2]), "+v"(boff[3]), "+v"(aoff[0]), "+v"(aoff[1]));
    const unsigned call = load_seq(seqp) + 1u;
    f32x4 acc[8][4];
#pragma unroll
    for (int m = 0; m < 8; ++m)
#pragma unroll
        for (int f = 0; f < 4; ++f) acc[m][f] = (f32x4){0.f, 0.f, 0.f, 0.f};
    bf16x8 xa[4][2], xb[4][2], wf[4][2];
    bf16x4 tl;
    unsigned bcur[4], acur[2];
#pragma unroll
    for (int f = 0; f < 4; ++f) bcur[f] = boff[f];
    acur[0] = aoff[0]; acur[1] = aoff[1];
#define RD_W(f, r) do { const bf16x4 v_ = __builtin_amdgcn_ds_read_tr16_b64_v4bf16((LAS bf16x4*)(bcur[f] + ((r) >> 1) * 8192 + ((r) & 1) * 1024)); \
        if ((r) & 1) wf[f][(r) >> 1] = __builtin_shufflevector(tl, v_, 0, 1, 2, 3, 4, 5, 6, 7); else tl = v_; } while (0)
#define RE_W(f, r) do { if (!((r) & 1)) asm volatile("" :: "v"(wf[f][(r) >> 1])); RD_W(f, r); } while (0)
#define RD_X(dst, h, r) do { dst[(r) >> 1][(r) & 1] = *(const LAS bf16x8*)(acur[(r) & 1] + ((h) * 4 + ((r) >> 1)) * 2048); } while (0)
#define SLOT(s) const int f = (s) >> 3, j = (f & 1) ? 3 - (((s) >> 1) & 3) : (((s) >> 1) & 3), kk = (s) & 1
    FENCE(); BAR(); FENCE();
    if (wid == 0) { STAMP(2); CSTAMP(30); }
#pragma unroll
    for (int r = 0; r < 4; ++r) RD_W(0, r);
#pragma unroll
    for (int r = 0; r < 8; ++r) RD_X(xa, 0, r);
#pragma unroll
    for (int r = 0; r < 8; ++r) RD_W(1 + (r >> 2), r & 3);
    wf[3][0] = wf[0][0]; wf[3][1] = wf[0][1];
    unsigned oa = A_STAGE, ob = B_STAGE;
    auto stepA = [&]() {
#pragma unroll
        for (int s = 0; s < 32; ++s) {
            SLOT(s);
            acc[j][f] = MFMA(wf[f][kk], xa[j][kk], acc[j][f]);
            SCHED();
            if (s < 4) RE_W(3, s);
            else if (s < 12) RD_X(xb, 1, s - 4);
            else if (s == 23) WAIT_L0();
            else if (s >= 24 && s < 28) bcur[s - 24] = boff[s - 24] + ob;
            else if (s >= 28 && s < 30) acur[s - 28] = aoff[s - 28] + oa;
            SCHED();
        }
    };
    auto stepB = [&]() {
        BAR(); SCHED();
#pragma unroll
        for (int s = 0; s < 32; ++s) {
            SLOT(s);
            acc[4 + j][f] = MFMA(wf[f][kk], xb[j][kk], acc[4 + j][f]);
            SCHED();
            if (s < 8) RD_X(xa, 0, s);
            else if ((s & 7) < 4) RE_W((s >> 3) - 1, s & 3);
            else if (s == 12) { ob = (ob + B_STAGE) & (RB * B_STAGE - 1); asm volatile("" : "+s"(ob)); }
            else if (s == 13) { oa += A_STAGE; asm volatile("" : "+s"(oa)); }
            else if (s == 14) { oa = oa == RA * A_STAGE ? 0u : oa; asm volatile("" : "+s"(oa)); }
            else if (s == 15) WAIT_L0();
            SCHED();
        }
    };
    WAIT_L0();
    stepA();
#pragma unroll 1
    for (int t = 0; t < NT - 1; ++t) { stepB(); stepA(); }

    int mx0 = 0, mx1 = 0;
#define TAKE_MAX(v) do { mx0 = max(max(mx0, __float_as_int((v)[0])), __float_as_int((v)[1])); mx1 = max(max(mx1, __float_as_int((v)[2])), __float_as_int((v)[3])); } while (0)
    SCHED(); BAR(); SCHED();
#pragma unroll
    for (int s = 0; s < 32; ++s) {
        { SLOT(s); acc[4 + j][f] = MFMA(wf[f][kk], xb[j][kk], acc[4 + j][f]); }
        SCHED();
        if (s < 16) TAKE_MAX(acc[s & 3][s >> 2]);
        else if (s < 28) { SLOT(2 * (s - 16)); (void)kk; TAKE_MAX(acc[4 + j][f]); }
        SCHED();
    }
#pragma unroll
    for (int j = 0; j < 4; ++j) TAKE_MAX(acc[7 - j][3]);
    const unsigned wmax = row_umax((unsigned)max(mx0, mx1));
    if (wid == 0) { STAMP(11); CSTAMP(31); }
    if (fr == 15) red[8 + wid * 4 + g] = wmax;
    if (lane == 0) red[32 + wid] = 0u;
    FENCE(); WAIT_L0(); BAR(); FENCE();
    const unsigned wgmax = wave_umax((lane >= 4 && lane < 24 && (lane & ~1) != 6) ? red[lane] : 0u);
    const bool odd = g & 1;
    {
        const unsigned xo = ldsbase + (unsigned)wid * 16384u + (unsigned)lane * 16u, xi = ldsbase + (unsigned)(wid ^ 2) * 16384u + (unsigned)(lane ^ 8) * 16u;
        auto trade = [&](auto base) {
            constexpr int B = decltype(base)::value;
#pragma unroll
            for (int i = 0; i < 16; ++i) *(LAS f32x4*)(xo + i * 1024) = acc[B + (i >> 2)][i & 3];
            FENCE(); WAIT_L0();
            if (lane == 0) *(volatile LAS unsigned*)(red + 32 + wid) = 1u;
            while (__builtin_amdgcn_readfirstlane(*(volatile LAS unsigned*)(red + 32 + (wid ^ 2))) == 0u) {}
            FENCE();
#pragma unroll
            for (int i = 0; i < 16; ++i) acc[B + (i >> 2)][i & 3] = *(const LAS f32x4*)(xi + i * 1024);
        };
        if (wn) trade(IC<0>{}); else trade(IC<4>{});
    }
    const bool lfirst = (fr < 8) == (wn == 0);
    const __amdgpu_buffer_rsrc_t outr = __builtin_amdgcn_make_buffer_rsrc(a.out, 0, MLOC * NDIM, SRD3);
    const unsigned ovr = (unsigned)(mtile * BM + wm * 128 + wn * 64 + (fr & 7)) * NDIM + (unsigned)(ntile * BN + g * 16);
    const unsigned ova = ovr + (lfirst ? 0u : 64u), ovb = ovr + 8u * NDIM + (lfirst ? 64u : 0u);
#define OUT_OFF(m) (((m) & 1 ? ovb : ova) + ((m) >> 1) * 16 * NDIM)
    unsigned used = 0xfffffffeu;
    const char* finp = wsl + WS_FIN + xcd * 512;
    const u64 t0 = __builtin_amdgcn_s_memrealtime();
    unsigned polls = 0;
    for (;;) {
        unsigned gm = wgmax, miss = 0;
        {
            const u64 fv = __hip_atomic_load((const u64*)finp + lane, __ATOMIC_RELAXED, __HIP_MEMORY_SCOPE_SYSTEM);
            const bool ok = (unsigned)(fv >> 32) == call;
            gm = max(gm, wave_umax(ok ? (unsigned)fv : 0u));
            miss = (unsigned)__popcll(__ballot(!ok));
        }
        if (miss != 0u && timed_out(tmo, call, t0, polls)) gm = 0xffffffffu;
        if (gm == 0xffffffffu) {
            const u32x4 o = {0x7f7f7f7fu, 0x7f7f7f7fu, 0x7f7f7f7fu, 0x7f7f7f7fu};
#pragma unroll
            for (int m = 0; m < 8; ++m) __builtin_amdgcn_raw_buffer_store_b128(o, outr, OUT_OFF(m), 0, OUT_AUX);
            break;
        }
        if (miss == 0u && wid == 0) STAMP(13);
        if (gm != used && miss <= SPEC_THR) {
            used = gm;
            const float amax = __uint_as_float(gm);
            const float inv = amax > 0.f ? 0.875f / amax : 0.f;
            const f32x2 sc = {inv, inv};
            auto quant = [&](const f32x4* y) -> u32x4 {
                unsigned d[4];
#pragma unroll
                for (int f = 0; f < 4; ++f) {
                    f32x2 lo = {y[f][0], y[f][1]}, hi = {y[f][2], y[f][3]};
                    asm("v_pk_mul_f32 %0, %1, %2 clamp" : "=v"(lo) : "v"(lo), "v"(sc));
                    asm("v_pk_mul_f32 %0, %1, %2 clamp" : "=v"(hi) : "v"(hi), "v"(sc));
                    s16x2 w = __builtin_bit_cast(s16x2, lo.x);
                    w = __builtin_amdgcn_cvt_scalef32_pk_fp8_f32(w, lo.x, lo.y, 1.0f / 512.0f, false);
                    w = __builtin_amdgcn_cvt_scalef32_pk_fp8_f32(w, hi.x, hi.y, 1.0f / 512.0f, true);
                    d[f] = __builtin_bit_cast(unsigned, w);
                }
                u32x4 o;
                o.x = odd ? d[1] : d[0]; o.y = odd ? d[0] : d[1]; o.z = odd ? d[3] : d[2]; o.w = odd ? d[2] : d[3];
                return o;
            };
#pragma unroll
            for (int j = 0; j < 4; ++j) {
                const u32x4 l = quant(acc[j]), r = quant(acc[4 + j]);
                __builtin_amdgcn_raw_buffer_store_b128(lfirst ? l : r, outr, OUT_OFF(2 * j), 0, OUT_AUX);
                __builtin_amdgcn_raw_buffer_store_b128(lfirst ? r : l, outr, OUT_OFF(2 * j + 1), 0, OUT_AUX);
            }
        }
        if (miss == 0u) break;
        __builtin_amdgcn_s_sleep(1);
    }
}

}

extern "C" void kernel_launch(int rank, int nranks, ncclComm_t comm,
                              hipStream_t stream,
                              void* const* d_in, const int* in_sizes, int n_in,
                              void* d_out, int out_size,
                              const int* in_dtypes, int out_dtype,
                              void* const* d_out_all,
                              void* const* d_ws_all, size_t ws_bytes) {
    using namespace a2ag;
    if (nranks != NR || (unsigned)rank >= (unsigned)NR || n_in < 2 || ws_bytes < WS_NEED
        || in_sizes[0] != KDIM * KSL || in_sizes[1] != KDIM * NDIM || out_size != MLOC * NDIM) {
        fprintf(stderr, "kernel_launch: unexpected problem geometry\n");
        return;
    }
    HIP_CHECK(hipFuncSetAttribute((const void*)a2a_gemm_kernel, hipFuncAttributeMaxDynamicSharedMemorySize, LDS_BYTES));
    Args a;
    a.x = d_in[0]; a.w = d_in[1]; a.out = d_out; a.rank = rank; a.pad = 0;
    a.wsl = (char*)d_ws_all[rank];
    for (int r = 0; r < NR; ++r) a.ws[r] = (char*)d_ws_all[r];
    hipLaunchKernelGGL(a2a_gemm_kernel, dim3(NWG), dim3(NTHREADS), LDS_BYTES, stream, a);
    HIP_CHECK(hipGetLastError());
}
```
